```python
import numpy as np

kernel_name = "gpu_gemm_a2a_m4096_k4096_n16384_bf16_qe4m3"


ABSMAX_RTOL = 0.09

COMPUTE_DTYPE = "bfloat16"
EPILOGUE = "quant_e4m3"
IN_QUANT = None
OUT_QUANT = "e4m3"
IN_DTYPES = {"x": "bfloat16", "w_mat": "bfloat16"}
WORLD_SIZE = 8

def _to_bf16(x):
    u = np.ascontiguousarray(x, dtype=np.float32).view(np.uint32)
    return ((u + 0x7FFF + ((u >> 16) & 1)) & 0xFFFF0000).view(np.float32)

def setup_inputs():

    x = _to_bf16(np.random.randn(4096, 4096).astype(np.float32))
    w_mat = _to_bf16((np.random.randn(4096, 16384) / np.sqrt(4096)).astype(np.float32))
    return {"x": x, "w_mat": w_mat}

def _snap_fp8(x, mant_bits, max_abs, min_normal):
    a = np.abs(x.astype(np.float64))
    _m, _e = np.frexp(a)
    q = np.where(a >= min_normal, np.ldexp(1.0, _e - 1 - mant_bits),
                 min_normal * 2.0 ** -mant_bits)
    snapped = np.minimum(np.round(a / q) * q, max_abs)
    return (np.sign(x) * snapped).astype(np.float32)
def _quant_dequant(y, amax):
    scale = amax / 448.0
    q = _snap_fp8(y / scale, 3, 448.0, 2.0 ** -6)
    return (q * scale).astype(np.float32)
def _epilogue(y):
    return _quant_dequant(y, np.abs(y).max())

def reference(x, w_mat):
    y = (x @ w_mat).astype(np.float32)
    return _epilogue(y)
```

```cpp
#include <hip/hip_runtime.h>
#include <rccl/rccl.h>
#include <cstdio>

#define HIP_CHECK(expr) do { hipError_t _e = (expr); if (_e != hipSuccess) { \
    fprintf(stderr, "HIP error %s at %s:%d\n", hipGetErrorString(_e), __FILE__, __LINE__); \
    return; } } while (0)

#define LAS __attribute__((address_space(3)))
typedef short s16x4 __attribute__((ext_vector_type(4)));
typedef short s16x8 __attribute__((ext_vector_type(8)));
typedef __bf16 bf16x8 __attribute__((ext_vector_type(8)));
typedef float f32x4 __attribute__((ext_vector_type(4)));
typedef unsigned u32x4 __attribute__((ext_vector_type(4)));
typedef unsigned short u16;
typedef unsigned long long u64;

constexpr int NR = 8;
constexpr int M = 512, N = 16384, K = 4096;
constexpr int NS = N / NR;
constexpr int BM = 256, BN = 128, BK = 64, NT = K / BK;
constexpr int NWG = (M / BM) * (N / BN);
constexpr int A_BYTES = BM * BK * 2, B_BYTES = BK * BN * 2;
constexpr int NA = 3, NB = 3;
constexpr int B_BASE = NA * A_BYTES;
constexpr int X_BASE = B_BASE + NB * B_BYTES;
constexpr int LDS_BYTES = X_BASE + 256;

constexpr size_t WS_CNT = 0;
constexpr size_t WS_TMO = WS_CNT + (size_t)NWG * 128;
constexpr size_t WS_GRAN = WS_TMO + 256;
constexpr size_t WS_DONE = WS_GRAN + 2 * (size_t)NR * NWG * 8;
constexpr size_t WS_END = WS_DONE + (size_t)NWG * 128;
constexpr u64 DEADLINE_TICKS = 40000000ull;

struct Params {
  const u16* x; const u16* w;
  unsigned char* out[NR];
  unsigned char* ws[NR];
  int rank; int pad;
};

#define SB() __builtin_amdgcn_sched_barrier(0)
#define BAR() do { asm volatile("" ::: "memory"); __builtin_amdgcn_s_barrier(); asm volatile("" ::: "memory"); } while (0)
#define WAITV(n) asm volatile("s_waitcnt vmcnt(%0)" :: "n"(n) : "memory")
#define WAITL0() asm volatile("s_waitcnt lgkmcnt(0)" ::: "memory")

__device__ __forceinline__ bf16x8 ldA(unsigned addr, int off) { return *(LAS bf16x8*)(addr + off); }
__device__ __forceinline__ s16x4 ldB(unsigned addr, int off) {
  return __builtin_amdgcn_ds_read_tr16_b64_v4i16((LAS s16x4*)(addr + off));
}

__global__ __launch_bounds__(512, 2) void gemm_a2a_q(const Params P) {
  extern __shared__ __attribute__((aligned(16))) char smem[];
  LAS char* lds = (LAS char*)smem;
  const unsigned lbase = (unsigned)(size_t)lds;
  const int tid = threadIdx.x, lane = tid & 63;
  const int wid = __builtin_amdgcn_readfirstlane(tid >> 6);
  const int bid = blockIdx.x;
  const int rank = P.rank;
  const int xcd = bid & 7, idx = bid >> 3;
  const int tn = (idx & 15) * 8 + xcd, tm = idx >> 4;
  unsigned char* const myws = P.ws[rank];
  LAS unsigned* xw = (LAS unsigned*)(lds + X_BASE);

  if (wid >= 4) {
    const int p = wid - 4;
    unsigned ticket = 0;
    if (p == 0 && lane == 0)
      ticket = __hip_atomic_fetch_add((unsigned*)(myws + WS_CNT + (size_t)bid * 128), 1u, __ATOMIC_RELAXED, __HIP_MEMORY_SCOPE_AGENT);
    __amdgpu_buffer_rsrc_t rA = __builtin_amdgcn_make_buffer_rsrc((void*)(P.x + (size_t)tm * BM * K), 0, BM * K * 2, 0x00020000);
    __amdgpu_buffer_rsrc_t rB = __builtin_amdgcn_make_buffer_rsrc((void*)(P.w + (size_t)tn * BN), 0, (K - 1) * N * 2 + BN * 2, 0x00020000);
    const unsigned voA = (unsigned)((lane >> 3) * (K * 2) + (((lane & 7) ^ (((p & 1) << 2) | (lane >> 4))) << 4)) + p * 8 * K * 2;
    const unsigned voB = (unsigned)((lane >> 4) * (N * 2) + (((lane & 15) ^ ((lane >> 4) | (((p >> 1) & 1) << 3))) << 4)) + p * 4 * N * 2;
    auto issue = [&](int kt, int buf) {
#pragma unroll
      for (int j = 0; j < 4; ++j)
        __builtin_amdgcn_raw_ptr_buffer_load_lds(rB, (LAS void*)(lds + B_BASE + buf * B_BYTES + (p + 4 * j) * 1024), 16, voB, (kt * BK + j * 16) * (N * 2), 0, 0);
#pragma unroll
      for (int j = 0; j < 8; ++j)
        __builtin_amdgcn_raw_ptr_buffer_load_lds(rA, (LAS void*)(lds + buf * A_BYTES + (p + 4 * j) * 1024), 16, voA, kt * (BK * 2) + j * 32 * K * 2, 0, 0);
    };
    issue(0, 0); issue(1, 1);
    WAITV(12);
    BAR();
    int buf = 2;
    for (int kt = 0; kt < NT - 2; ++kt) {
      issue(kt + 2, buf);
      WAITV(12);
      buf = (buf == 2) ? 0 : buf + 1;
      BAR();
    }
    WAITV(0);
    BAR();
    const unsigned call = (unsigned)__builtin_amdgcn_readfirstlane((int)ticket) + 1u;
    if (p == 0 && lane == 0) { xw[8] = call; xw[10] = 0u; }
    WAITL0();
    BAR();
    const unsigned callv = xw[8];
    const unsigned char* gp = myws + WS_GRAN + (size_t)(callv & 1) * (NR * NWG * 8) + (size_t)(p * 64 + lane) * 16;
    unsigned* tmo = (unsigned*)(myws + WS_TMO);
    unsigned mx = 0; bool bad = false;
    const u64 t0 = __builtin_amdgcn_s_memrealtime();
    for (;;) {
      u32x4 a, b, c, d;
      asm volatile("global_load_dwordx4 %0, %4, off sc0 sc1\n\t"
                   "global_load_dwordx4 %1, %5, off sc0 sc1\n\t"
                   "global_load_dwordx4 %2, %6, off sc0 sc1\n\t"
                   "global_load_dwordx4 %3, %7, off sc0 sc1\n\ts_waitcnt vmcnt(0)"
                   : "=&v"(a), "=&v"(b), "=&v"(c), "=&v"(d)
                   : "v"(gp), "v"(gp + 4096), "v"(gp + 8192), "v"(gp + 12288) : "memory");
      const bool ok = a[1] == callv && a[3] == callv && b[1] == callv && b[3] == callv &&
                      c[1] == callv && c[3] == callv && d[1] == callv && d[3] == callv;
      if (__all(ok)) {
        mx = max(max(max(a[0], a[2]), max(b[0], b[2])), max(max(c[0], c[2]), max(d[0], d[2])));
        break;
      }
      if (__hip_atomic_load(tmo, __ATOMIC_RELAXED, __HIP_MEMORY_SCOPE_AGENT) == callv ||
          __builtin_amdgcn_s_memrealtime() - t0 > DEADLINE_TICKS) {
        __hip_atomic_store(tmo, callv, __ATOMIC_RELAXED, __HIP_MEMORY_SCOPE_AGENT);
        bad = true; break;
      }
      __builtin_amdgcn_s_sleep(2);
    }
#pragma unroll
    for (int o = 32; o >= 1; o >>= 1) mx = max(mx, (unsigned)__shfl_xor((int)mx, o));
    if (bad || __hip_atomic_load(tmo, __ATOMIC_RELAXED, __HIP_MEMORY_SCOPE_AGENT) == callv) mx = 0x7fc00000u;
    if (lane == 0) xw[4 + p] = mx;
    WAITL0();
    BAR();
    BAR();
    if (p == 0 && lane == 0) {
      const int dst = tn >> 4;
      if (dst != rank)
        __hip_atomic_store((unsigned*)(P.ws[dst] + WS_DONE + (size_t)(rank * 32 + (tn & 15) * 2 + tm) * 128), callv, __ATOMIC_RELAXED, __HIP_MEMORY_SCOPE_SYSTEM);
      if ((bid >> 5) != rank) {
        const unsigned* fl = (const unsigned*)(myws + WS_DONE + (size_t)bid * 128);
        const u64 t1 = __builtin_amdgcn_s_memrealtime();
        while (__hip_atomic_load(fl, __ATOMIC_RELAXED, __HIP_MEMORY_SCOPE_SYSTEM) < callv) {
          if (__hip_atomic_load(tmo, __ATOMIC_RELAXED, __HIP_MEMORY_SCOPE_AGENT) == callv ||
              __builtin_amdgcn_s_memrealtime() - t1 > DEADLINE_TICKS) {
            __hip_atomic_store(tmo, callv, __ATOMIC_RELAXED, __HIP_MEMORY_SCOPE_AGENT);
            P.out[rank][(size_t)rank * M * NS] = 0x7f;
            break;
          }
          __builtin_amdgcn_s_sleep(2);
        }
      }
    }
  } else {
    const int wr = wid >> 1, wc = wid & 1;
    const int g = lane >> 4, c = lane & 15, q = (lane & 15) >> 2, pp = lane & 3;
    unsigned aA[2], aB[4];
#pragma unroll
    for (int ks = 0; ks < 2; ++ks) aA[ks] = lbase + (wr * 128 + c) * 128 + (((4 * ks + g) ^ (c >> 1)) << 4);
#pragma unroll
    for (int b = 0; b < 4; ++b) {
      const int row = 8 * g + q, ch = 8 * wc + 2 * pp + (b >> 1);
      aB[b] = lbase + B_BASE + row * 256 + ((ch ^ ((row & 3) | (row & 8))) << 4) + (((b ^ pp) & 1) << 3);
    }
    f32x4 acc[8][4];
#pragma unroll
    for (int m = 0; m < 8; ++m)
#pragma unroll
      for (int b = 0; b < 4; ++b) acc[m][b] = (f32x4){0.f, 0.f, 0.f, 0.f};
    bf16x8 Af[2][8];
    s16x4 Bl[2][4], Bh[2][4];
    auto readItem = [&](int set, int ks, int i) {
      if (i < 8) {
        const int b = i >> 1;
        if ((i & 1) == 0) Bl[set][b] = ldB(aB[b], ks * 8192);
        else Bh[set][b] = ldB(aB[b], ks * 8192 + 1024);
      } else {
        Af[set][i - 8] = ldA(aA[ks], (i - 8) * 2048);
      }
    };
    auto mma = [&](int set, int i) {
      const int m = i >> 2, b = i & 3;
      s16x8 bv = __builtin_shufflevector(Bl[set][b], Bh[set][b], 0, 1, 2, 3, 4, 5, 6, 7);
      acc[m][b] = __builtin_amdgcn_mfma_f32_16x16x32_bf16(__builtin_bit_cast(bf16x8, bv), Af[set][m], acc[m][b], 0, 0, 0);
    };
    BAR();
#pragma unroll
    for (int i = 0; i < 16; ++i) readItem(0, 0, i);
    int buf = 0;
    for (int kt = 0; kt < NT; ++kt) {
      SB();
#pragma unroll
      for (int i = 0; i < 32; ++i) {
        mma(0, i); SB();
        if (i < 16) { readItem(1, 1, i); SB(); }
      }
      {
        const bool wrap = (buf == 2);
        const int da = wrap ? -2 * A_BYTES : A_BYTES, db = wrap ? -2 * B_BYTES : B_BYTES;
        buf = wrap ? 0 : buf + 1;
        aA[0] += da; aA[1] += da; aB[0] += db; aB[1] += db; aB[2] += db; aB[3] += db;
      }
      WAITL0(); SB();
      BAR();
      SB();
#pragma unroll
      for (int i = 0; i < 32; ++i) {
        mma(1, i); SB();
        if (i < 16) { readItem(0, 0, i); SB(); }
      }
    }
    float am = 0.f;
#pragma unroll
    for (int m = 0; m < 8; ++m)
#pragma unroll
      for (int b = 0; b < 4; ++b) {
        am = fmaxf(am, fmaxf(fabsf(acc[m][b][0]), fabsf(acc[m][b][1])));
        am = fmaxf(am, fmaxf(fabsf(acc[m][b][2]), fabsf(acc[m][b][3])));
      }
    unsigned mu = __float_as_uint(am);
#pragma unroll
    for (int o = 32; o >= 1; o >>= 1) mu = max(mu, (unsigned)__shfl_xor((int)mu, o));
    unsigned arrived = 0;
    if (lane == 0) {
      xw[wid] = mu;
      WAITL0();
      arrived = __hip_atomic_fetch_add(&xw[10], 1u, __ATOMIC_RELAXED, __HIP_MEMORY_SCOPE_WORKGROUP);
    }
    arrived = (unsigned)__builtin_amdgcn_readfirstlane((int)arrived);
    const unsigned callv = xw[8];
    if (arrived == 3) {
      asm volatile("" ::: "memory");
      const unsigned wm = max(max(xw[0], xw[1]), max(xw[2], xw[3]));
      if (lane < NR) {
        u64* dstg = (u64*)(P.ws[lane] + WS_GRAN + (size_t)(callv & 1) * (NR * NWG * 8) + (size_t)(rank * NWG + bid) * 8);
        __hip_atomic_store(dstg, ((u64)callv << 32) | wm, __ATOMIC_RELAXED, __HIP_MEMORY_SCOPE_SYSTEM);
      }
    }
    BAR();
    const unsigned gm = max(max(xw[4], xw[5]), max(xw[6], xw[7]));
    const float inv = 448.0f / __uint_as_float(gm);
    unsigned char* op = P.out[tn >> 4] + (size_t)(rank * M + tm * BM + wr * 128 + c) * NS + (tn & 15) * BN + wc * 64 + 16 * g;
    const bool odd = g & 1;
#pragma unroll
    for (int m = 0; m < 8; ++m) {
      unsigned wv[4];
#pragma unroll
      for (int b = 0; b < 4; ++b) {
        int t = __builtin_amdgcn_cvt_pk_fp8_f32(acc[m][b][0] * inv, acc[m][b][1] * inv, 0, false);
        t = __builtin_amdgcn_cvt_pk_fp8_f32(acc[m][b][2] * inv, acc[m][b][3] * inv, t, true);
        wv[b] = (unsigned)t;
      }
      u32x4 v;
      v[0] = odd ? wv[1] : wv[0]; v[1] = odd ? wv[0] : wv[1]; v[2] = odd ? wv[3] : wv[2]; v[3] = odd ? wv[2] : wv[3];
      asm volatile("global_store_dwordx4 %0, %1, off sc0 sc1\n\ts_nop 1" :: "v"(op + (size_t)m * 16 * NS), "v"(v) : "memory");
    }
    WAITV(0);
    BAR();
  }
}

extern "C" void kernel_launch(int rank, int nranks, ncclComm_t comm, hipStream_t stream,
                              void* const* d_in, const int* in_sizes, int n_in,
                              void* d_out, int out_size, const int* in_dtypes, int out_dtype,
                              void* const* d_out_all, void* const* d_ws_all, size_t ws_bytes) {
  (void)comm; (void)in_sizes; (void)n_in; (void)d_out; (void)out_size; (void)in_dtypes; (void)out_dtype;
  if (nranks != NR || ws_bytes < WS_END || (unsigned)rank >= (unsigned)NR) { fprintf(stderr, "unsupported configuration\n"); return; }
  static bool init = false;
  if (!init) {
    HIP_CHECK(hipFuncSetAttribute((const void*)gemm_a2a_q, hipFuncAttributeMaxDynamicSharedMemorySize, LDS_BYTES));
    init = true;
  }
  Params P;
  P.x = (const u16*)d_in[0]; P.w = (const u16*)d_in[1];
  for (int p = 0; p < NR; ++p) { P.out[p] = (unsigned char*)d_out_all[p]; P.ws[p] = (unsigned char*)d_ws_all[p]; }
  P.rank = rank; P.pad = 0;
  hipLaunchKernelGGL(gemm_a2a_q, dim3(NWG), dim3(512), LDS_BYTES, stream, P);
  HIP_CHECK(hipGetLastError());
}
```

```cpp
#include <hip/hip_runtime.h>
#include <rccl/rccl.h>
#include <cstdio>

#define HIP_CHECK(expr) do { hipError_t _e = (expr); if (_e != hipSuccess) { \
    fprintf(stderr, "HIP error %s at %s:%d\n", hipGetErrorString(_e), __FILE__, __LINE__); \
    return; } } while (0)

#define LAS __attribute__((address_space(3)))
typedef short s16x4 __attribute__((ext_vector_type(4)));
typedef short s16x8 __attribute__((ext_vector_type(8)));
typedef __bf16 bf16x8 __attribute__((ext_vector_type(8)));
typedef float f32x4 __attribute__((ext_vector_type(4)));
typedef unsigned u32x4 __attribute__((ext_vector_type(4)));
typedef unsigned short u16;
typedef unsigned long long u64;

constexpr int NR = 8;
constexpr int M = 512, N = 16384, K = 4096;
constexpr int NS = N / NR;
constexpr int BM = 256, BN = 128, BK = 64, NT = K / BK;
constexpr int NWG = 256;
constexpr int A_BYTES = BM * BK * 2, B_BYTES = BK * BN * 2;
constexpr int B_BASE = 3 * A_BYTES;
constexpr int X_BASE = B_BASE + 3 * B_BYTES;
constexpr int LDS_BYTES = X_BASE + 256;
#define BMODE 0
#define AUXW 2
#define P0 4
#define LG 2

constexpr size_t WS_CNT = 0;
constexpr size_t WS_TMO = WS_CNT + (size_t)NWG * 128;
constexpr size_t WS_GRAN = WS_TMO + 256;
constexpr size_t WS_DONE = WS_GRAN + 2 * (size_t)NR * NWG * 8;
constexpr size_t WS_XF = WS_DONE + (size_t)NR * 32 * 128;
constexpr size_t WS_XS = 1u << 20;
constexpr size_t WS_END = WS_XS + (size_t)NR * NT * A_BYTES;
static_assert(WS_XF + NR * 16 * 128 <= WS_XS, "layout");
constexpr u64 DEADLINE_TICKS = 40000000ull;

struct Params {
  const u16* x; const u16* w;
  unsigned char* out[NR];
  unsigned char* ws[NR];
  int rank; int pad;
};

#define STAMPW(k) do {} while (0)
#define SB() __builtin_amdgcn_sched_barrier(0)
#define BAR() do { asm volatile("" ::: "memory"); __builtin_amdgcn_s_barrier(); asm volatile("" ::: "memory"); } while (0)
#define WAITV(n) asm volatile("s_waitcnt vmcnt(%0)" :: "n"(n) : "memory")
#define WAITL0() asm volatile("s_waitcnt lgkmcnt(0)" ::: "memory")

__device__ __forceinline__ bf16x8 ldA(unsigned addr, int off) { return *(LAS bf16x8*)(size_t)(addr + off); }
__device__ __forceinline__ s16x4 ldB(unsigned addr, int off) {
  return __builtin_amdgcn_ds_read_tr16_b64_v4i16((LAS s16x4*)(size_t)(addr + off));
}
__device__ __forceinline__ unsigned wave_max_u32(unsigned v) {
  v = max(v, (unsigned)__builtin_amdgcn_update_dpp(0, (int)v, 0x111, 0xf, 0xf, false));
  v = max(v, (unsigned)__builtin_amdgcn_update_dpp(0, (int)v, 0x112, 0xf, 0xf, false));
  v = max(v, (unsigned)__builtin_amdgcn_update_dpp(0, (int)v, 0x114, 0xf, 0xf, false));
  v = max(v, (unsigned)__builtin_amdgcn_update_dpp(0, (int)v, 0x118, 0xf, 0xf, false));
  const unsigned a = __builtin_amdgcn_readlane(v, 15), b = __builtin_amdgcn_readlane(v, 31);
  const unsigned c = __builtin_amdgcn_readlane(v, 47), d = __builtin_amdgcn_readlane(v, 63);
  return max(max(a, b), max(c, d));
}

__global__ __launch_bounds__(512, 2) void gemm_a2a_q(const Params P) {
  extern __shared__ __attribute__((aligned(16))) char smem[];
  LAS char* lds = (LAS char*)smem;
  const unsigned lbase = (unsigned)(size_t)lds;
  const int tid = threadIdx.x, lane = tid & 63;
  const int wid = __builtin_amdgcn_readfirstlane(tid >> 6);
  const int bid = blockIdx.x;
  const int rank = P.rank;
  const int xcd = bid & 7, idx = bid >> 3;
  int src, dst, hf, ct;
  { const int tn = (idx & 15) * 8 + xcd; src = rank; dst = tn >> 4; ct = tn & 15; hf = idx >> 4; }
  unsigned char* const myws = P.ws[rank];
  unsigned* const tmo = (unsigned*)(myws + WS_TMO);
  LAS unsigned* xw = (LAS unsigned*)(lds + X_BASE);

  if (wid >= 4) {
    const int p = wid - 4;
    if (p < 2) {
      const bool remote = src != rank;
      const void* base = remote ? (const void*)(myws + WS_XS + (size_t)src * NT * A_BYTES) : (const void*)(P.x + (size_t)hf * BM * K);
      __amdgpu_buffer_rsrc_t rA = __builtin_amdgcn_make_buffer_rsrc((void*)base, 0, BM * K * 2, 0x00020000);
      const unsigned pstr = remote ? 1024u : 8u * K * 2, kstr = remote ? (unsigned)A_BYTES : BK * 2u;
      const unsigned vo = (remote ? (unsigned)lane * 16u
                                  : (unsigned)((lane >> 3) * (K * 2) + (((lane & 7) ^ ((p << 2) | (lane >> 4))) << 4))) + p * pstr;
      auto issue = [&](int kt, int buf) {
#pragma unroll
        for (int j = 0; j < 16; ++j)
          __builtin_amdgcn_raw_ptr_buffer_load_lds(rA, (LAS void*)(lds + buf * A_BYTES + (p + 2 * j) * 1024), 16, vo, kt * kstr + 2 * j * pstr, 0, 17);
      };
      BAR();
      issue(0, 0); issue(1, 1);
      WAITV(16);
      BAR();
      int buf = 2;
      for (int kt = 0; kt < NT - 2; ++kt) {
        issue(kt + 2, buf);
        WAITV(16);
        buf = (buf == 2) ? 0 : buf + 1;
        BAR();
      }
      WAITV(0);
      BAR(); BAR();
    } else if (p == 2) {
      __amdgpu_buffer_rsrc_t rB = __builtin_amdgcn_make_buffer_rsrc((void*)(P.w + (size_t)dst * NS + ct * BN), 0, (K - 1) * N * 2 + BN * 2, 0x00020000);
      const unsigned vo0 = (unsigned)((lane >> 4) * (N * 2) + (((lane & 15) ^ (lane >> 4)) << 4)), vo1 = vo0 ^ 128u;
      auto issue = [&](int kt, int buf) {
#pragma unroll
        for (int j = 0; j < 16; ++j)
          __builtin_amdgcn_raw_ptr_buffer_load_lds(rB, (LAS void*)(lds + B_BASE + buf * B_BYTES + j * 1024), 16, (j & 2) ? vo1 : vo0, (kt * BK + j * 4) * (N * 2), 0, AUXW);
      };
      issue(0, 0); issue(1, 1);
      BAR();
      WAITV(16);
      BAR();
      int buf = 2;
      for (int kt = 0; kt < NT - 2; ++kt) {
        issue(kt + 2, buf);
        WAITV(16);
        buf = (buf == 2) ? 0 : buf + 1;
        BAR();
      }
      WAITV(0);
      BAR(); BAR();
    } else {
      unsigned call;
      if (BMODE && dst != rank) {
        const char* sp = (const char*)(P.x + (size_t)(BM + 16 * ct + (lane >> 3)) * K);
        const char* sp0 = sp + (((lane & 7) ^ (lane >> 4)) << 4);
        const char* sp1 = sp + 8 * K * 2 + (((lane & 7) ^ (4 | (lane >> 4))) << 4);
        unsigned char* dp = P.ws[dst] + WS_XS + (size_t)rank * NT * A_BYTES + ct * 2048 + lane * 16;
        unsigned* fp = (unsigned*)(P.ws[dst] + WS_XF + (rank * 16 + ct) * 128);
        u32x4 r0[8], r1[8];
#pragma unroll
        for (int s = 0; s < 8; ++s) { r0[s] = (u32x4){0u, 0u, 0u, 0u}; r1[s] = (u32x4){0u, 0u, 0u, 0u}; }
#define PLOAD(s) asm volatile("global_load_dwordx4 %0, %2, off offset:%4\n\tglobal_load_dwordx4 %1, %3, off offset:%4" \
                              : "+v"(r0[s]), "+v"(r1[s]) : "v"(sp0), "v"(sp1), "n"((s) * 128) : "memory")
#define PSTORE(s) do { asm volatile("global_store_dwordx4 %0, %1, off sc0 sc1\n\tglobal_store_dwordx4 %0, %2, off offset:1024 sc0 sc1\n\ts_nop 1" \
                                    :: "v"(dp), "v"(r0[s]), "v"(r1[s]) : "memory"); dp += A_BYTES; } while (0)
#define PFLAG() do { ++fv; asm volatile("s_mov_b64 exec, 1\n\tglobal_store_dword %0, %1, off sc0 sc1\n\ts_mov_b64 exec, -1" :: "v"(fp), "v"(fv) : "memory"); } while (0)
        static_assert(P0 > LG && 4 * LG <= 3 * P0 && 5 * (8 - P0) > 5 * LG + 2 && P0 <= 8, "vmcnt literals below");
        unsigned ticket = 0;
        if (lane == 0) asm volatile("global_atomic_add %0, %1, %2, off sc0" : "+v"(ticket) : "v"(myws + WS_CNT + (size_t)bid * 128), "v"(1u) : "memory");
#pragma unroll
        for (int s = 0; s < 8; ++s) PLOAD(s);
        sp0 += 1024; sp1 += 1024;
        BAR();
        asm volatile("s_waitcnt vmcnt(%1)" : "+v"(ticket) : "n"(16 - 2 * P0) : "memory");
        call = (unsigned)__builtin_amdgcn_readfirstlane((int)ticket) + 1u;
        unsigned fv = (call - 1u) * NT;
#pragma unroll
        for (int s = 0; s < P0; ++s) PSTORE(s);
        if (lane == 0) xw[8] = call;
#pragma unroll
        for (int s = 0; s < P0 - LG; ++s) { WAITV(2 * (P0 - 1 - s) + s); PFLAG(); }
#pragma unroll
        for (int s = 0; s < P0; ++s) PLOAD(s);
        WAITL0();
        BAR();
        for (int k8 = 0; k8 < NT - 16; k8 += 8) {
#pragma unroll
          for (int i = 0; i < 8; ++i) {
            const int s = (i + P0) & 7;
            PSTORE(s);
            if (s == 0) { sp0 += 1024; sp1 += 1024; }
            PLOAD(s);
            WAITV(5 * LG + 2);
            PFLAG();
            BAR();
          }
        }
#pragma unroll
        for (int kt = NT - 16; kt < NT; ++kt) {
          const int t = kt + P0, s = t & 7;
          if (t < NT) {
            PSTORE(s);
            if (s == 0) { sp0 += 1024; sp1 += 1024; }
            if (t + 8 < NT) { PLOAD(s); WAITV(5 * LG + 2); } else { WAITV(3 * LG); }
            PFLAG();
          } else if (t < NT + LG) {
            WAITV(3 * (NT + LG - 1 - t) + (t - NT) + 1);
            PFLAG();
          }
          BAR();
        }
        WAITV(0);
      } else {
        unsigned ticket = 0;
        if (lane == 0) ticket = __hip_atomic_fetch_add((unsigned*)(myws + WS_CNT + (size_t)bid * 128), 1u, __ATOMIC_RELAXED, __HIP_MEMORY_SCOPE_AGENT);
        call = (unsigned)__builtin_amdgcn_readfirstlane((int)ticket) + 1u;
        if (lane == 0) xw[8] = call;
        WAITL0();
        if (src != rank) {
          const unsigned* fp = (const unsigned*)(myws + WS_XF + (src * 16 + (lane & 15)) * 128);
          const unsigned base = (call - 1u) * NT;
          unsigned ready = 0;
          bool dead = false;
          for (int k = -2; k < NT; ++k) {
            const unsigned need = (unsigned)min(k == -2 ? 2 : k + 4, NT);
            if (ready < (unsigned)NT && !dead) {
              const u64 t0 = __builtin_amdgcn_s_memrealtime();
              for (;;) {
                if (ready >= need + 2u) break;
                const unsigned v = __hip_atomic_load(fp, __ATOMIC_RELAXED, __HIP_MEMORY_SCOPE_SYSTEM);
                const unsigned d = v - base;
                ready = NT - wave_max_u32(d > (unsigned)NT ? (unsigned)NT : NT - d);
                if (ready >= need) break;
                if (__hip_atomic_load(tmo, __ATOMIC_RELAXED, __HIP_MEMORY_SCOPE_AGENT) == call ||
                    __builtin_amdgcn_s_memrealtime() - t0 > DEADLINE_TICKS) {
                  __hip_atomic_store(tmo, call, __ATOMIC_RELAXED, __HIP_MEMORY_SCOPE_AGENT);
                  dead = true; break;
                }
                __builtin_amdgcn_s_sleep(1);
              }
            }
            BAR();
          }
        } else {
          for (int k = -2; k < NT; ++k) BAR();
        }
      }
      u64* dstg = (u64*)(P.ws[lane & 7] + WS_GRAN + (size_t)(call & 1) * (NR * NWG * 8) + (size_t)(rank * NWG + bid) * 8);
      BAR();
      const unsigned wm = max(max(xw[0], xw[1]), max(xw[2], xw[3]));
      if (lane < NR) __hip_atomic_store(dstg, ((u64)call << 32) | wm, __ATOMIC_RELAXED, __HIP_MEMORY_SCOPE_SYSTEM);
      STAMPW(3);
    }
    if (p < 3) BAR();
    const unsigned callv = xw[8];
    const unsigned char* gp = myws + WS_GRAN + (size_t)(callv & 1) * (NR * NWG * 8) + (size_t)(p * 64 + lane) * 16;
    unsigned mx = 0; bool bad = false;
    const u64 t0 = __builtin_amdgcn_s_memrealtime();
    for (;;) {
      u32x4 a, b, c, d;
      asm volatile("global_load_dwordx4 %0, %4, off sc0 sc1\n\t"
                   "global_load_dwordx4 %1, %5, off sc0 sc1\n\t"
                   "global_load_dwordx4 %2, %6, off sc0 sc1\n\t"
                   "global_load_dwordx4 %3, %7, off sc0 sc1\n\ts_waitcnt vmcnt(0)"
                   : "=&v"(a), "=&v"(b), "=&v"(c), "=&v"(d)
                   : "v"(gp), "v"(gp + 4096), "v"(gp + 8192), "v"(gp + 12288) : "memory");
      const bool ok = a[1] == callv && a[3] == callv && b[1] == callv && b[3] == callv &&
                      c[1] == callv && c[3] == callv && d[1] == callv && d[3] == callv;
      if (__all(ok)) {
        mx = max(max(max(a[0], a[2]), max(b[0], b[2])), max(max(c[0], c[2]), max(d[0], d[2])));
        break;
      }
      if (__hip_atomic_load(tmo, __ATOMIC_RELAXED, __HIP_MEMORY_SCOPE_AGENT) == callv ||
          __builtin_amdgcn_s_memrealtime() - t0 > DEADLINE_TICKS) {
        __hip_atomic_store(tmo, callv, __ATOMIC_RELAXED, __HIP_MEMORY_SCOPE_AGENT);
        bad = true; break;
      }
      __builtin_amdgcn_s_sleep(2);
    }
    mx = wave_max_u32(mx);
    if (bad || __hip_atomic_load(tmo, __ATOMIC_RELAXED, __HIP_MEMORY_SCOPE_AGENT) == callv) mx = 0x7fc00000u;
    if (lane == 0) xw[4 + p] = mx;
    WAITL0();
    BAR();
    BAR();
    if (p == 3 && lane == 0) {
      if (dst != rank)
        __hip_atomic_store((unsigned*)(P.ws[dst] + WS_DONE + (size_t)(BMODE ? rank * 16 + ct : rank * 32 + ct * 2 + hf) * 128), callv, __ATOMIC_RELAXED, __HIP_MEMORY_SCOPE_SYSTEM);
      const int f = BMODE ? bid - 128 : bid;
      if (f >= 0 && (f >> (BMODE ? 4 : 5)) != rank) {
        const unsigned* fl = (const unsigned*)(myws + WS_DONE + (size_t)f * 128);
        const u64 t1 = __builtin_amdgcn_s_memrealtime();
        while (__hip_atomic_load(fl, __ATOMIC_RELAXED, __HIP_MEMORY_SCOPE_SYSTEM) < callv) {
          if (__hip_atomic_load(tmo, __ATOMIC_RELAXED, __HIP_MEMORY_SCOPE_AGENT) == callv ||
              __builtin_amdgcn_s_memrealtime() - t1 > DEADLINE_TICKS) {
            __hip_atomic_store(tmo, callv, __ATOMIC_RELAXED, __HIP_MEMORY_SCOPE_AGENT);
            P.out[rank][(size_t)rank * M * NS] = 0x7f;
            break;
          }
          __builtin_amdgcn_s_sleep(2);
        }
      }
      STAMPW(7);
    }
  } else {
    const int wr = wid >> 1, wc = wid & 1;
    const int g = lane >> 4, c = lane & 15, q = (lane & 15) >> 2, pp = lane & 3;
    if (wid == 0) STAMPW(0);
    unsigned warm = 0;
    if (wid < 2) asm volatile("global_load_dword %0, %1, off" : "+v"(warm) : "v"((const char*)P.x + (size_t)bid * 16384 + (wid * 64 + lane) * 128));
    unsigned aA[2], aB[4];
#pragma unroll
    for (int ks = 0; ks < 2; ++ks) aA[ks] = lbase + (wr * 128 + c) * 128 + (((4 * ks + g) ^ (c >> 1)) << 4);
#pragma unroll
    for (int b = 0; b < 4; ++b) {
      const int row = 8 * g + q, ch = 8 * wc + 2 * pp + (b >> 1);
      aB[b] = lbase + B_BASE + row * 256 + ((ch ^ ((row & 3) | (row & 8))) << 4) + (((b ^ pp) & 1) << 3);
    }
    f32x4 acc[8][4];
#pragma unroll
    for (int m = 0; m < 8; ++m)
#pragma unroll
      for (int b = 0; b < 4; ++b) acc[m][b] = (f32x4){0.f, 0.f, 0.f, 0.f};
    bf16x8 Af[2][8];
    s16x4 Bl[2][4], Bh[2][4];
    auto readItem = [&](int set, int ks, int i) {
      if (i < 8) {
        const int b = i >> 1;
        if ((i & 1) == 0) Bl[set][b] = ldB(aB[b], ks * 8192);
        else Bh[set][b] = ldB(aB[b], ks * 8192 + 1024);
      } else {
        Af[set][i - 8] = ldA(aA[ks], (i - 8) * 2048);
      }
    };
    auto mma = [&](int set, int i) {
      const int m = i >> 2, b = i & 3;
      s16x8 bv = __builtin_shufflevector(Bl[set][b], Bh[set][b], 0, 1, 2, 3, 4, 5, 6, 7);
      acc[m][b] = __builtin_amdgcn_mfma_f32_16x16x32_bf16(__builtin_bit_cast(bf16x8, bv), Af[set][m], acc[m][b], 0, 0, 0);
    };
    BAR();
    BAR();
    if (wid == 0) STAMPW(1);
#pragma unroll
    for (int i = 0; i < 16; ++i) readItem(0, 0, i);
    int buf = 0;
    for (int kt = 0; kt < NT; ++kt) {
      const bool wrap = (buf == 2);
      const int da = wrap ? -2 * A_BYTES : A_BYTES, db = wrap ? -2 * B_BYTES : B_BYTES;
      buf = wrap ? 0 : buf + 1;
      SB();
#pragma unroll
      for (int i = 0; i < 32; ++i) {
        mma(0, i); SB();
        if (i < 16) { readItem(1, 1, i); SB(); }
        if (i == 18) { aA[0] += da; SB(); }
        if (i == 20) { aA[1] += da; SB(); }
        if (i >= 22 && i < 30 && !(i & 1)) { aB[(i - 22) >> 1] += db; SB(); }
      }
      WAITL0(); SB();
      BAR();
      SB();
#pragma unroll
      for (int i = 0; i < 32; ++i) {
        mma(1, i); SB();
        if (i < 16) { readItem(0, 0, i); SB(); }
      }
    }
    if (wid == 0) STAMPW(2);
    float am = 0.f;
#pragma unroll
    for (int m = 0; m < 8; ++m)
#pragma unroll
      for (int b = 0; b < 4; ++b) {
        am = __builtin_fmaxf(__builtin_fmaxf(am, __builtin_fabsf(acc[m][b][0])), __builtin_fabsf(acc[m][b][1]));
        am = __builtin_fmaxf(__builtin_fmaxf(am, __builtin_fabsf(acc[m][b][2])), __builtin_fabsf(acc[m][b][3]));
      }
    const unsigned mu = wave_max_u32(__float_as_uint(am));
    if (lane == 0) xw[wid] = mu;
    asm volatile("s_waitcnt vmcnt(0) lgkmcnt(0)" :: "v"(warm) : "memory");
    BAR();
    unsigned char* op = P.out[dst] + (size_t)(src * M + hf * BM + wr * 128 + c) * NS + ct * BN + wc * 64 + 16 * g;
    const bool odd = g & 1;
    BAR();
    if (wid == 0) STAMPW(4);
    const unsigned gm = max(max(xw[4], xw[5]), max(xw[6], xw[7]));
    const float inv = 448.0f / __uint_as_float(gm);
#pragma unroll
    for (int m = 0; m < 8; ++m) {
      unsigned wv[4];
#pragma unroll
      for (int b = 0; b < 4; ++b) {
        int t = __builtin_amdgcn_cvt_pk_fp8_f32(acc[m][b][0] * inv, acc[m][b][1] * inv, 0, false);
        t = __builtin_amdgcn_cvt_pk_fp8_f32(acc[m][b][2] * inv, acc[m][b][3] * inv, t, true);
        wv[b] = (unsigned)t;
      }
      u32x4 v;
      v[0] = odd ? wv[1] : wv[0]; v[1] = odd ? wv[0] : wv[1]; v[2] = odd ? wv[3] : wv[2]; v[3] = odd ? wv[2] : wv[3];
      asm volatile("global_store_dwordx4 %0, %1, off sc0 sc1\n\ts_nop 1" :: "v"(op + (size_t)m * 16 * NS), "v"(v) : "memory");
      SB();
    }
    if (wid == 0) STAMPW(5);
    WAITV(0);
    if (wid == 0) STAMPW(6);
    BAR();
  }
}

extern "C" void kernel_launch(int rank, int nranks, ncclComm_t comm, hipStream_t stream,
                              void* const* d_in, const int* in_sizes, int n_in,
                              void* d_out, int out_size, const int* in_dtypes, int out_dtype,
                              void* const* d_out_all, void* const* d_ws_all, size_t ws_bytes) {
  (void)comm; (void)in_sizes; (void)n_in; (void)d_out; (void)out_size; (void)in_dtypes; (void)out_dtype;
  if (nranks != NR || ws_bytes < WS_END || (unsigned)rank >= (unsigned)NR) { fprintf(stderr, "unsupported configuration\n"); return; }
  static bool init = false;
  if (!init) {
    HIP_CHECK(hipFuncSetAttribute((const void*)gemm_a2a_q, hipFuncAttributeMaxDynamicSharedMemorySize, LDS_BYTES));
    init = true;
  }
  Params P;
  P.x = (const u16*)d_in[0]; P.w = (const u16*)d_in[1];
  for (int p = 0; p < NR; ++p) { P.out[p] = (unsigned char*)d_out_all[p]; P.ws[p] = (unsigned char*)d_ws_all[p]; }
  P.rank = rank; P.pad = 0;
  hipLaunchKernelGGL(gemm_a2a_q, dim3(NWG), dim3(512), LDS_BYTES, stream, P);
  HIP_CHECK(hipGetLastError());
}
```

```cpp
#include <hip/hip_runtime.h>
#include <rccl/rccl.h>
#include <cstdio>

#define HIP_CHECK(expr) do { hipError_t _e = (expr); if (_e != hipSuccess) { \
    fprintf(stderr, "HIP error %s at %s:%d\n", hipGetErrorString(_e), __FILE__, __LINE__); \
    return; } } while (0)

#define LAS __attribute__((address_space(3)))
typedef short s16x4 __attribute__((ext_vector_type(4)));
typedef short s16x8 __attribute__((ext_vector_type(8)));
typedef __bf16 bf16x8 __attribute__((ext_vector_type(8)));
typedef float f32x4 __attribute__((ext_vector_type(4)));
typedef unsigned u32x4 __attribute__((ext_vector_type(4)));
typedef unsigned short u16;
typedef unsigned long long u64;

constexpr int NR = 8;
constexpr int M = 512, N = 16384, K = 4096;
constexpr int NS = N / NR;
constexpr int BM = 256, BN = 128, BK = 64, NT = K / BK;
constexpr int NWG = 256;
constexpr int XT_BYTES = BM * BK * 2;
constexpr int A_BYTES = 288 * BK * 2, B_BYTES = BK * BN * 2;
constexpr int B_BASE = 3 * A_BYTES;
constexpr int X_BASE = B_BASE + 3 * B_BYTES;
constexpr int LDS_BYTES = X_BASE + 256;
#define AUXW 2
#define CLG 1

constexpr size_t WS_CNT = 0;
constexpr size_t WS_TMO = WS_CNT + (size_t)NWG * 128;
constexpr size_t WS_GRAN = WS_TMO + 256;
constexpr size_t WS_DONE = WS_GRAN + 2 * (size_t)NR * NWG * 8;
constexpr size_t WS_XF = WS_DONE + (size_t)NR * 16 * 128;
constexpr size_t WS_XS = 1u << 20;
constexpr size_t WS_END = WS_XS + (size_t)NR * NT * XT_BYTES;
static_assert(WS_XF + NR * 16 * 128 <= WS_XS, "layout");
constexpr u64 DEADLINE_TICKS = 40000000ull;

struct Params {
  const u16* x; const u16* w;
  unsigned char* out[NR];
  unsigned char* ws[NR];
  int rank; int pad;
};

#define STAMPW(k) do {} while (0)
#define SB() __builtin_amdgcn_sched_barrier(0)
#define BAR() do { asm volatile("" ::: "memory"); __builtin_amdgcn_s_barrier(); asm volatile("" ::: "memory"); } while (0)
#define WAITV(n) asm volatile("s_waitcnt vmcnt(%0)" :: "n"(n) : "memory")
#define WAITL0() asm volatile("s_waitcnt lgkmcnt(0)" ::: "memory")
#define DI __device__ __forceinline__

DI bf16x8 ldA(unsigned addr, int off) { return *(LAS bf16x8*)(size_t)(addr + off); }
DI s16x4 ldB(unsigned addr, int off) { return __builtin_amdgcn_ds_read_tr16_b64_v4i16((LAS s16x4*)(size_t)(addr + off)); }
DI unsigned wave_max_u32(unsigned v) {
  v = max(v, (unsigned)__builtin_amdgcn_update_dpp(0, (int)v, 0x111, 0xf, 0xf, false));
  v = max(v, (unsigned)__builtin_amdgcn_update_dpp(0, (int)v, 0x112, 0xf, 0xf, false));
  v = max(v, (unsigned)__builtin_amdgcn_update_dpp(0, (int)v, 0x114, 0xf, 0xf, false));
  v = max(v, (unsigned)__builtin_amdgcn_update_dpp(0, (int)v, 0x118, 0xf, 0xf, false));
  const unsigned a = __builtin_amdgcn_readlane(v, 15), b = __builtin_amdgcn_readlane(v, 31);
  const unsigned c = __builtin_amdgcn_readlane(v, 47), d = __builtin_amdgcn_readlane(v, 63);
  return max(max(a, b), max(c, d));
}
DI bool give_up(unsigned* tmo, unsigned call, u64 t0, unsigned& spins) {
  if ((++spins & 7u) != 0u) return false;
  if (__hip_atomic_load(tmo, __ATOMIC_RELAXED, __HIP_MEMORY_SCOPE_AGENT) == call || __builtin_amdgcn_s_memrealtime() - t0 > DEADLINE_TICKS) {
    __hip_atomic_store(tmo, call, __ATOMIC_RELAXED, __HIP_MEMORY_SCOPE_AGENT);
    return true;
  }
  return false;
}

struct Tile { int src, dst, ct, grp; };

template <bool TALL>
DI void loader(const Params& P, const Tile T, LAS char* lds, const int p, const int lane) {
  const int rank = P.rank;
  const bool remote = TALL && T.src != rank;
  const void* base = remote ? (const void*)(P.ws[rank] + WS_XS + (size_t)T.src * NT * XT_BYTES) : (const void*)(P.x + (size_t)(TALL ? BM : 0) * K);
  __amdgpu_buffer_rsrc_t rA = __builtin_amdgcn_make_buffer_rsrc((void*)base, 0, BM * K * 2, 0x00020000);
  __amdgpu_buffer_rsrc_t rE = __builtin_amdgcn_make_buffer_rsrc((void*)(P.x + (size_t)T.grp * 32 * K), 0, 32 * K * 2, 0x00020000);
  __amdgpu_buffer_rsrc_t rB = __builtin_amdgcn_make_buffer_rsrc((void*)(P.w + (size_t)T.dst * NS + T.ct * BN), 0, (K - 1) * N * 2 + BN * 2, 0x00020000);
  const unsigned pstr = remote ? 1024u : 8u * K * 2, kstr = remote ? (unsigned)XT_BYTES : BK * 2u;
  const unsigned voL = (unsigned)(((lane >> 3) + 8 * p) * (K * 2) + (((lane & 7) ^ ((p << 2) | (lane >> 4))) << 4));
  const unsigned voA = remote ? (unsigned)lane * 16u + p * 1024u : voL;
  const unsigned voB = (unsigned)(((lane >> 4) + 8 * p) * (N * 2) + (((lane & 15) ^ ((lane >> 4) | (p << 3))) << 4));
  auto issueA = [&](int kt, int buf) {
#pragma unroll
    for (int j = 0; j < 16; ++j)
      __builtin_amdgcn_raw_ptr_buffer_load_lds(rA, (LAS void*)(lds + buf * A_BYTES + (p + 2 * j + 2 * (j >> 3)) * 1024), 16, voA, kt * kstr + 2 * j * pstr, 0, 17);
    if (TALL) {
#pragma unroll
      for (int j = 0; j < 2; ++j)
        __builtin_amdgcn_raw_ptr_buffer_load_lds(rE, (LAS void*)(lds + buf * A_BYTES + (16 + p + 18 * j) * 1024), 16, voL, kt * (BK * 2) + j * 16 * K * 2, 0, 0);
    }
  };
  auto issueB = [&](int kt, int buf) {
#pragma unroll
    for (int j = 0; j < 8; ++j) {
      const int pc = (j & 1) + 4 * (j >> 1);
      __builtin_amdgcn_raw_ptr_buffer_load_lds(rB, (LAS void*)(lds + B_BASE + buf * B_BYTES + (pc + 2 * p) * 1024), 16, voB, (kt * BK + pc * 4) * (N * 2), 0, TALL ? 0 : AUXW);
    }
  };
  constexpr int NA = TALL ? 18 : 16;
  issueB(0, 0); issueB(1, 1);
  BAR();
  issueA(0, 0); issueA(1, 1);
  WAITV(NA);
  BAR();
  int buf = 2;
  for (int kt = 0; kt < NT - 2; ++kt) {
    issueB(kt + 2, buf); issueA(kt + 2, buf);
    WAITV(NA + 8);
    buf = (buf == 2) ? 0 : buf + 1;
    BAR();
  }
  WAITV(0);
  BAR(); BAR();
}

template <int MB>
DI void consumer(const Params& P, const Tile T, LAS char* lds, const int wid, const int lane) {
  const unsigned lbase = (unsigned)(size_t)lds;
  LAS unsigned* xw = (LAS unsigned*)(lds + X_BASE);
  const int wr = wid >> 1, wc = wid & 1;
  const int g = lane >> 4, c = lane & 15, q = (lane & 15) >> 2, pp = lane & 3;
  unsigned aA[2], aB[4];
#pragma unroll
  for (int ks = 0; ks < 2; ++ks) aA[ks] = lbase + (wr * 144 + c) * 128 + (((4 * ks + g) ^ (c >> 1)) << 4);
#pragma unroll
  for (int b = 0; b < 4; ++b) {
    const int row = 8 * g + q, ch = 8 * wc + 2 * pp + (b >> 1);
    aB[b] = lbase + B_BASE + row * 256 + ((ch ^ ((row & 3) | (row & 8))) << 4) + (((b ^ pp) & 1) << 3);
  }
  f32x4 acc[MB][4];
#pragma unroll
  for (int m = 0; m < MB; ++m)
#pragma unroll
    for (int b = 0; b < 4; ++b) acc[m][b] = (f32x4){0.f, 0.f, 0.f, 0.f};
  bf16x8 Af[2][MB];
  s16x4 Bl[2][4], Bh[2][4];
  constexpr int NRD = 8 + MB, NM = 4 * MB;
  auto readItem = [&](int set, int ks, int i) {
    if (i < 8) {
      const int b = i >> 1;
      if ((i & 1) == 0) Bl[set][b] = ldB(aB[b], ks * 8192);
      else Bh[set][b] = ldB(aB[b], ks * 8192 + 1024);
    } else {
      Af[set][i - 8] = ldA(aA[ks], (i - 8) * 2048);
    }
  };
  auto mma = [&](int set, int i) {
    const int m = i >> 2, b = i & 3;
    s16x8 bv = __builtin_shufflevector(Bl[set][b], Bh[set][b], 0, 1, 2, 3, 4, 5, 6, 7);
    acc[m][b] = __builtin_amdgcn_mfma_f32_16x16x32_bf16(__builtin_bit_cast(bf16x8, bv), Af[set][m], acc[m][b], 0, 0, 0);
  };
  BAR();
  BAR();
  if (wid == 0) STAMPW(1);
#pragma unroll
  for (int i = 0; i < NRD; ++i) readItem(0, 0, i);
  int buf = 0;
  for (int kt = 0; kt < NT; ++kt) {
    const bool wrap = (buf == 2);
    const int da = wrap ? -2 * A_BYTES : A_BYTES, db = wrap ? -2 * B_BYTES : B_BYTES;
    buf = wrap ? 0 : buf + 1;
    SB();
#pragma unroll
    for (int i = 0; i < NM; ++i) {
      mma(0, i); SB();
      if (i < NRD) { readItem(1, 1, i); SB(); }
      if (i == 18) { aA[0] += da; SB(); }
      if (i == 20) { aA[1] += da; SB(); }
      if (i >= 22 && i < 30 && !(i & 1)) { aB[(i - 22) >> 1] += db; SB(); }
    }
    WAITL0(); SB();
    BAR();
    SB();
#pragma unroll
    for (int i = 0; i < NM; ++i) {
      mma(1, i); SB();
      if (i < NRD) { readItem(0, 0, i); SB(); }
    }
  }
  if (wid == 0) STAMPW(2);
  float am = 0.f;
#pragma unroll
  for (int m = 0; m < MB; ++m)
#pragma unroll
    for (int b = 0; b < 4; ++b) {
      am = __builtin_fmaxf(__builtin_fmaxf(am, __builtin_fabsf(acc[m][b][0])), __builtin_fabsf(acc[m][b][1]));
      am = __builtin_fmaxf(__builtin_fmaxf(am, __builtin_fabsf(acc[m][b][2])), __builtin_fabsf(acc[m][b][3]));
    }
  const unsigned mu = wave_max_u32(__float_as_uint(am));
  if (lane == 0) xw[wid] = mu;
  WAITL0();
  BAR();
  unsigned char* ob = P.out[T.dst] + T.ct * BN + wc * 64 + 16 * g;
  unsigned char* op = ob + (size_t)(T.src * M + (MB == 9 ? BM : 0) + wr * 128 + c) * NS;
  unsigned char* oe = ob + (size_t)(P.rank * M + T.grp * 32 + wr * 16 + c) * NS;
  const bool odd = g & 1;
  BAR();
  if (wid == 0) STAMPW(4);
  const unsigned gm = max(max(xw[4], xw[5]), max(xw[6], xw[7]));
  const float inv = 448.0f / __uint_as_float(gm);
#pragma unroll
  for (int m = 0; m < MB; ++m) {
    unsigned wv[4];
#pragma unroll
    for (int b = 0; b < 4; ++b) {
      int t = __builtin_amdgcn_cvt_pk_fp8_f32(acc[m][b][0] * inv, acc[m][b][1] * inv, 0, false);
      t = __builtin_amdgcn_cvt_pk_fp8_f32(acc[m][b][2] * inv, acc[m][b][3] * inv, t, true);
      wv[b] = (unsigned)t;
    }
    u32x4 v;
    v[0] = odd ? wv[1] : wv[0]; v[1] = odd ? wv[0] : wv[1]; v[2] = odd ? wv[3] : wv[2]; v[3] = odd ? wv[2] : wv[3];
    asm volatile("global_store_dwordx4 %0, %1, off sc0 sc1\n\ts_nop 1" :: "v"(m < 8 ? op + (size_t)m * 16 * NS : oe), "v"(v) : "memory");
    SB();
  }
  if (wid == 0) STAMPW(5);
  WAITV(0);
  if (wid == 0) STAMPW(6);
  BAR();
}

__global__ __launch_bounds__(512, 2) void gemm_a2a_q(const Params P) {
  extern __shared__ __attribute__((aligned(16))) char smem[];
  LAS char* lds = (LAS char*)smem;
  const int tid = threadIdx.x, lane = tid & 63;
  const int wid = __builtin_amdgcn_readfirstlane(tid >> 6);
  const int bid = blockIdx.x;
  const int rank = P.rank;
  const int xcd = bid & 7, idx = bid >> 3;
  unsigned char* const myws = P.ws[rank];
  unsigned* const tmo = (unsigned*)(myws + WS_TMO);
  unsigned* const cnt = (unsigned*)(myws + WS_CNT + (size_t)bid * 128);
  if (wid == 0) STAMPW(0);

  if (idx < 2) {
    const int cw = xcd * 2 + idx;
    LAS unsigned* xw = (LAS unsigned*)(lds + X_BASE);
    if (wid < 7) {
      const int pd = (rank + 1 + wid) & 7;
      const char* sp = (const char*)(P.x + (size_t)(BM + 16 * cw + (lane >> 3)) * K);
      const char* sp0 = sp + (((lane & 7) ^ (lane >> 4)) << 4);
      const char* sp1 = sp + 8 * K * 2 + (((lane & 7) ^ (4 | (lane >> 4))) << 4);
      unsigned char* dp = P.ws[pd] + WS_XS + (size_t)rank * NT * XT_BYTES + cw * 2048 + lane * 16;
      unsigned* fp = (unsigned*)(P.ws[pd] + WS_XF + (rank * 16 + cw) * 128);
      u32x4 r0[8], r1[8];
#pragma unroll
      for (int s = 0; s < 8; ++s) { r0[s] = (u32x4){0u, 0u, 0u, 0u}; r1[s] = (u32x4){0u, 0u, 0u, 0u}; }
#define CLOAD(s) asm volatile("global_load_dwordx4 %0, %2, off offset:%4\n\tglobal_load_dwordx4 %1, %3, off offset:%4" \
                              : "+v"(r0[s]), "+v"(r1[s]) : "v"(sp0), "v"(sp1), "n"((s) * 128) : "memory")
#define CSTORE(s) do { asm volatile("global_store_dwordx4 %0, %1, off sc0 sc1\n\tglobal_store_dwordx4 %0, %2, off offset:1024 sc0 sc1\n\ts_nop 1" \
                                    :: "v"(dp), "v"(r0[s]), "v"(r1[s]) : "memory"); dp += XT_BYTES; } while (0)
#define CFLAG() do { ++fv; asm volatile("s_mov_b64 exec, 1\n\tglobal_store_dword %0, %1, off sc0 sc1\n\ts_mov_b64 exec, -1" :: "v"(fp), "v"(fv) : "memory"); } while (0)
      static_assert(CLG == 1, "the start-up below is written for CLG == 1");
      CLOAD(0); CLOAD(1); CLOAD(2);
      WAITV(4); CSTORE(0);
      WAITV(4); CSTORE(1);
#pragma unroll
      for (int s = 3; s < 8; ++s) CLOAD(s);
      sp0 += 1024; sp1 += 1024;
      CLOAD(0); CLOAD(1);
      BAR();
      unsigned fv = (xw[8] - 1u) * NT;
      WAITV(16); CFLAG();
      CSTORE(2); CLOAD(2); WAITV(19); CFLAG(); BAR();
      WAITV(10);
#pragma unroll
      for (int t = 3; t < 8; ++t) { CSTORE(t); CLOAD(t); WAITV(5 * CLG + 2); CFLAG();
        BAR(); }
      for (int k8 = 8; k8 < NT - 8; k8 += 8) {
        sp0 += 1024; sp1 += 1024;
#pragma unroll
        for (int i = 0; i < 8; ++i) { CSTORE(i); CLOAD(i); WAITV(5 * CLG + 2); CFLAG(); BAR(); }
      }
#pragma unroll
      for (int i = 0; i < 8; ++i) { CSTORE(i); WAITV(3 * CLG); CFLAG(); BAR(); }
      WAITV(0);
      fv += CLG - 1; CFLAG();
      if (wid == 0) { STAMPW(1); STAMPW(2); }
    } else {
      unsigned ticket = 0;
      if (lane == 0) ticket = __hip_atomic_fetch_add(cnt, 1u, __ATOMIC_RELAXED, __HIP_MEMORY_SCOPE_AGENT);
      const unsigned call = (unsigned)__builtin_amdgcn_readfirstlane((int)ticket) + 1u;
      if (lane == 0) xw[8] = call;
      WAITL0();
      BAR();
      u64* dstg = (u64*)(P.ws[lane & 7] + WS_GRAN + (size_t)(call & 1) * (NR * NWG * 8) + (size_t)(rank * NWG + bid) * 8);
      if (lane < NR) __hip_atomic_store(dstg, (u64)call << 32, __ATOMIC_RELAXED, __HIP_MEMORY_SCOPE_SYSTEM);
      for (int k = 2; k < NT; ++k) BAR();
    }
    return;
  }

  const bool tall = idx >= 16;
  Tile T;
  if (!tall) { const int u = idx - 2; T.src = rank; T.dst = (rank + 1 + u % 7) & 7; T.ct = xcd * 2 + u / 7; T.grp = 0; }
  else { const int u = idx - 16; T.grp = (xcd >> 2) * 4 + (u >> 2); T.src = (rank + T.grp) & 7; T.dst = rank; T.ct = (xcd & 3) * 4 + (u & 3); }

  if (wid < 4) {
    if (tall) consumer<9>(P, T, lds, wid, lane); else consumer<8>(P, T, lds, wid, lane);
    return;
  }
  LAS unsigned* xw = (LAS unsigned*)(lds + X_BASE);
  const int p = wid - 4;
  if (p < 2) {
    if (tall) loader<true>(P, T, lds, p, lane); else loader<false>(P, T, lds, p, lane);
    BAR();
  } else if (p == 2) {
    unsigned warm = 0;
    asm volatile("global_load_dword %0, %1, off\n\tglobal_load_dword %0, %2, off" : "+v"(warm)
                 : "v"((const char*)P.x + (size_t)bid * 16384 + lane * 128), "v"((const char*)P.x + (size_t)bid * 16384 + 8192 + lane * 128) : "memory");
    if (idx >= 30)
      asm volatile("global_load_dword %0, %1, off\n\tglobal_load_dword %0, %2, off" : "+v"(warm)
                   : "v"((const char*)P.x + (size_t)(bid - 240) * 16384 + lane * 128), "v"((const char*)P.x + (size_t)(bid - 240) * 16384 + 8192 + lane * 128) : "memory");
    for (int k = -2; k < NT; ++k) BAR();
    asm volatile("s_waitcnt vmcnt(0)" :: "v"(warm) : "memory");
    BAR();
  } else {
    unsigned ticket = 0;
    if (lane == 0) ticket = __hip_atomic_fetch_add(cnt, 1u, __ATOMIC_RELAXED, __HIP_MEMORY_SCOPE_AGENT);
    if (T.src == rank) BAR();
    const unsigned call = (unsigned)__builtin_amdgcn_readfirstlane((int)ticket) + 1u;
    if (lane == 0) xw[8] = call;
    WAITL0();
    if (T.src != rank) {
      const unsigned* fp = (const unsigned*)(myws + WS_XF + (T.src * 16 + (lane & 15)) * 128);
      const unsigned base = (call - 1u) * NT;
      unsigned ready = 0, spins = 0;
      for (int k = -2; k < NT; ++k) {
        const unsigned need = (unsigned)min(k + 4, NT);
        if (ready < min(need + 2u, (unsigned)NT)) {
          const u64 t0 = __builtin_amdgcn_s_memrealtime();
          for (;;) {
            const unsigned d = __hip_atomic_load(fp, __ATOMIC_RELAXED, __HIP_MEMORY_SCOPE_SYSTEM) - base;
            ready = NT - wave_max_u32(d > (unsigned)NT ? (unsigned)NT : NT - d);
            if (ready >= need) break;
            if (give_up(tmo, call, t0, spins)) { ready = NT; break; }
            __builtin_amdgcn_s_sleep(1);
          }
        }
        BAR();
      }
    } else {
      for (int k = -1; k < NT; ++k) BAR();
    }
    u64* dstg = (u64*)(P.ws[lane & 7] + WS_GRAN + (size_t)(call & 1) * (NR * NWG * 8) + (size_t)(rank * NWG + bid) * 8);
    BAR();
    const unsigned wm = max(max(xw[0], xw[1]), max(xw[2], xw[3]));
    if (lane < NR) __hip_atomic_store(dstg, ((u64)call << 32) | wm, __ATOMIC_RELAXED, __HIP_MEMORY_SCOPE_SYSTEM);
    STAMPW(3);
  }
  const unsigned callv = xw[8];
  const unsigned char* gp = myws + WS_GRAN + (size_t)(callv & 1) * (NR * NWG * 8) + (size_t)(p * 64 + lane) * 16;
  unsigned mx = 0, spins = 0;
  const u64 t0 = __builtin_amdgcn_s_memrealtime();
  for (;;) {
    u32x4 a, b, c, d;
    asm volatile("global_load_dwordx4 %0, %4, off sc0 sc1\n\t"
                 "global_load_dwordx4 %1, %5, off sc0 sc1\n\t"
                 "global_load_dwordx4 %2, %6, off sc0 sc1\n\t"
                 "global_load_dwordx4 %3, %7, off sc0 sc1\n\ts_waitcnt vmcnt(0)"
                 : "=&v"(a), "=&v"(b), "=&v"(c), "=&v"(d)
                 : "v"(gp), "v"(gp + 4096), "v"(gp + 8192), "v"(gp + 12288) : "memory");
    const bool ok = a[1] == callv && a[3] == callv && b[1] == callv && b[3] == callv &&
                    c[1] == callv && c[3] == callv && d[1] == callv && d[3] == callv;
    if (__all(ok)) {
      mx = max(max(max(a[0], a[2]), max(b[0], b[2])), max(max(c[0], c[2]), max(d[0], d[2])));
      break;
    }
    if (give_up(tmo, callv, t0, spins)) break;
    __builtin_amdgcn_s_sleep(2);
  }
  mx = wave_max_u32(mx);
  if (__hip_atomic_load(tmo, __ATOMIC_RELAXED, __HIP_MEMORY_SCOPE_AGENT) == callv) mx = 0x7fc00000u;
  if (lane == 0) xw[4 + p] = mx;
  WAITL0();
  BAR();
  BAR();
  if (p == 3 && lane == 0) {
    if (T.dst != rank) {
      __hip_atomic_store((unsigned*)(P.ws[T.dst] + WS_DONE + (size_t)(rank * 16 + T.ct) * 128), callv, __ATOMIC_RELAXED, __HIP_MEMORY_SCOPE_SYSTEM);
    } else if (((bid - 128) >> 4) != rank) {
      const unsigned* fl = (const unsigned*)(myws + WS_DONE + (size_t)(bid - 128) * 128);
      const u64 t1 = __builtin_amdgcn_s_memrealtime();
      while (__hip_atomic_load(fl, __ATOMIC_RELAXED, __HIP_MEMORY_SCOPE_SYSTEM) < callv) {
        if (give_up(tmo, callv, t1, spins)) { P.out[rank][(size_t)rank * M * NS] = 0x7f; break; }
        __builtin_amdgcn_s_sleep(2);
      }
    }
    STAMPW(7);
  }
}

extern "C" void kernel_launch(int rank, int nranks, ncclComm_t comm, hipStream_t stream,
                              void* const* d_in, const int* in_sizes, int n_in,
                              void* d_out, int out_size, const int* in_dtypes, int out_dtype,
                              void* const* d_out_all, void* const* d_ws_all, size_t ws_bytes) {
  (void)comm; (void)in_sizes; (void)n_in; (void)d_out; (void)out_size; (void)in_dtypes; (void)out_dtype;
  if (nranks != NR || ws_bytes < WS_END || (unsigned)rank >= (unsigned)NR) { fprintf(stderr, "unsupported configuration\n"); return; }
  static bool init = false;
  if (!init) {
    HIP_CHECK(hipFuncSetAttribute((const void*)gemm_a2a_q, hipFuncAttributeMaxDynamicSharedMemorySize, LDS_BYTES));
    init = true;
  }
  Params P;
  P.x = (const u16*)d_in[0]; P.w = (const u16*)d_in[1];
  for (int p = 0; p < NR; ++p) { P.out[p] = (unsigned char*)d_out_all[p]; P.ws[p] = (unsigned char*)d_ws_all[p]; }
  P.rank = rank; P.pad = 0;
  hipLaunchKernelGGL(gemm_a2a_q, dim3(NWG), dim3(512), LDS_BYTES, stream, P);
  HIP_CHECK(hipGetLastError());
}
```

```cpp
#include <hip/hip_runtime.h>
#include <rccl/rccl.h>
#include <cstdio>

#define HIP_CHECK(expr) do { hipError_t _e = (expr); if (_e != hipSuccess) { \
    fprintf(stderr, "HIP error %s at %s:%d\n", hipGetErrorString(_e), __FILE__, __LINE__); \
    return; } } while (0)

#define LAS __attribute__((address_space(3)))
typedef short s16x4 __attribute__((ext_vector_type(4)));
typedef short s16x8 __attribute__((ext_vector_type(8)));
typedef __bf16 bf16x8 __attribute__((ext_vector_type(8)));
typedef float f32x4 __attribute__((ext_vector_type(4)));
typedef unsigned u32x4 __attribute__((ext_vector_type(4)));
typedef unsigned short u16;
typedef unsigned long long u64;

constexpr int NR = 8;
constexpr int M = 512, N = 16384, K = 4096;
constexpr int NS = N / NR;
constexpr int BM = 256, BN = 128, BK = 64, NT = K / BK;
constexpr int NWG = 256;
constexpr int XT_BYTES = BM * BK * 2;
constexpr int A_BYTES = 288 * BK * 2, B_BYTES = BK * BN * 2;
constexpr int B_BASE = 3 * A_BYTES;
constexpr int X_BASE = B_BASE + 3 * B_BYTES;
constexpr int LDS_BYTES = X_BASE + 256;
#define AUXW 2
#define NA1 28
#define PA1 72
#define PA2 94
#define NB1 22
#define PB1 70
#define PB2 80
#define GSLEEP 2
#define CLG 1

constexpr size_t WS_CNT = 0;
constexpr size_t WS_TMO = WS_CNT + (size_t)NWG * 128;
constexpr size_t WS_G1 = WS_TMO + 256;
constexpr size_t WS_G2 = WS_G1 + 8 * 128 * 8;
constexpr size_t WS_DONE = WS_G2 + 2 * NR * 8 * 8;
constexpr size_t WS_XF = WS_DONE + (size_t)NR * 16 * 128;
constexpr size_t WS_XS = 1u << 20;
constexpr size_t XS_STRIDE = (size_t)NT * XT_BYTES;
constexpr size_t WS_END = WS_XS + (size_t)NR * XS_STRIDE;
static_assert(WS_XF + NR * 16 * 128 <= WS_XS, "layout");
constexpr u64 DEADLINE_TICKS = 40000000ull;

struct Params {
  const u16* x; const u16* w;
  unsigned char* out[NR];
  unsigned char* ws[NR];
  int rank; int pad;
};

#define STAMPW(k) do {} while (0)
#define SB() __builtin_amdgcn_sched_barrier(0)
#define BAR() do { asm volatile("" ::: "memory"); __builtin_amdgcn_s_barrier(); asm volatile("" ::: "memory"); } while (0)
#define WAITV(n) asm volatile("s_waitcnt vmcnt(%0)" :: "n"(n) : "memory")
#define WAITL0() asm volatile("s_waitcnt lgkmcnt(0)" ::: "memory")
#define DI __device__ __forceinline__

DI bf16x8 ldA(unsigned addr, int off) { return *(LAS bf16x8*)(size_t)(addr + off); }
DI s16x4 ldB(unsigned addr, int off) { return __builtin_amdgcn_ds_read_tr16_b64_v4i16((LAS s16x4*)(size_t)(addr + off)); }
DI unsigned wave_max_u32(unsigned v) {
  v = max(v, (unsigned)__builtin_amdgcn_update_dpp(0, (int)v, 0x111, 0xf, 0xf, false));
  v = max(v, (unsigned)__builtin_amdgcn_update_dpp(0, (int)v, 0x112, 0xf, 0xf, false));
  v = max(v, (unsigned)__builtin_amdgcn_update_dpp(0, (int)v, 0x114, 0xf, 0xf, false));
  v = max(v, (unsigned)__builtin_amdgcn_update_dpp(0, (int)v, 0x118, 0xf, 0xf, false));
  const unsigned a = __builtin_amdgcn_readlane(v, 15), b = __builtin_amdgcn_readlane(v, 31);
  const unsigned c = __builtin_amdgcn_readlane(v, 47), d = __builtin_amdgcn_readlane(v, 63);
  return max(max(a, b), max(c, d));
}
DI bool give_up(unsigned* tmo, unsigned call, u64 t0, unsigned& spins) {
  if ((++spins & 7u) != 0u) return false;
  if (__hip_atomic_load(tmo, __ATOMIC_RELAXED, __HIP_MEMORY_SCOPE_AGENT) == call || __builtin_amdgcn_s_memrealtime() - t0 > DEADLINE_TICKS) {
    __hip_atomic_store(tmo, call, __ATOMIC_RELAXED, __HIP_MEMORY_SCOPE_AGENT);
    return true;
  }
  return false;
}

struct Tile { int src, dst, ct, grp; };

template <bool TALL>
DI void loader(const Params& P, const Tile T, LAS char* lds, const int p, const int lane) {
  const int rank = P.rank;
  const bool remote = TALL && T.src != rank;
  const void* base = remote ? (const void*)(P.ws[rank] + WS_XS + (size_t)T.src * XS_STRIDE) : (const void*)(P.x + (size_t)(TALL ? BM : 0) * K);
  __amdgpu_buffer_rsrc_t rA = __builtin_amdgcn_make_buffer_rsrc((void*)base, 0, BM * K * 2, 0x00020000);
  __amdgpu_buffer_rsrc_t rE = __builtin_amdgcn_make_buffer_rsrc((void*)(P.x + (size_t)T.grp * 32 * K), 0, 32 * K * 2, 0x00020000);
  __amdgpu_buffer_rsrc_t rB = __builtin_amdgcn_make_buffer_rsrc((void*)(P.w + (size_t)T.dst * NS + T.ct * BN), 0, (K - 1) * N * 2 + BN * 2, 0x00020000);
  const unsigned pstr = remote ? 1024u : 8u * K * 2, kstr = remote ? (unsigned)XT_BYTES : BK * 2u;
  const unsigned voL = (unsigned)(((lane >> 3) + 8 * p) * (K * 2) + (((lane & 7) ^ ((p << 2) | (lane >> 4))) << 4));
  const unsigned voA = remote ? (unsigned)lane * 16u + p * 1024u : voL;
  const unsigned voB = (unsigned)(((lane >> 4) + 8 * p) * (N * 2) + (((lane & 15) ^ ((lane >> 4) | (p << 3))) << 4));
  auto issueA = [&](int kt, int buf) {
#pragma unroll
    for (int j = 0; j < 16; ++j)
      __builtin_amdgcn_raw_ptr_buffer_load_lds(rA, (LAS void*)(lds + buf * A_BYTES + (p + 2 * j + 2 * (j >> 3)) * 1024), 16, voA, kt * kstr + 2 * j * pstr, 0, 17);
    if (TALL) {
#pragma unroll
      for (int j = 0; j < 2; ++j)
        __builtin_amdgcn_raw_ptr_buffer_load_lds(rE, (LAS void*)(lds + buf * A_BYTES + (16 + p + 18 * j) * 1024), 16, voL, kt * (BK * 2) + j * 16 * K * 2, 0, 0);
    }
  };
  auto issueB = [&](int kt, int buf) {
#pragma unroll
    for (int j = 0; j < 8; ++j) {
      const int pc = (j & 1) + 4 * (j >> 1);
      __builtin_amdgcn_raw_ptr_buffer_load_lds(rB, (LAS void*)(lds + B_BASE + buf * B_BYTES + (pc + 2 * p) * 1024), 16, voB, (kt * BK + pc * 4) * (N * 2), 0, TALL ? 0 : AUXW);
    }
  };
  constexpr int NA = TALL ? 18 : 16;
  issueB(0, 0); issueB(1, 1);
  BAR();
  issueA(0, 0); issueA(1, 1);
  WAITV(NA);
  BAR();
  int buf = 2;
  for (int kt = 0; kt < NT - 2; ++kt) {
    issueB(kt + 2, buf); issueA(kt + 2, buf);
    WAITV(NA + 8);
    buf = (buf == 2) ? 0 : buf + 1;
    BAR();
  }
  WAITV(0);
  BAR(); BAR();
}

template <int MB>
DI void consumer(const Params& P, const Tile T, LAS char* lds, const int wid, const int lane) {
  const unsigned lbase = (unsigned)(size_t)lds;
  LAS unsigned* xw = (LAS unsigned*)(lds + X_BASE);
  const int wr = wid >> 1, wc = wid & 1;
  const int g = lane >> 4, c = lane & 15, q = (lane & 15) >> 2, pp = lane & 3;
  unsigned aA[2], aB[4];
#pragma unroll
  for (int ks = 0; ks < 2; ++ks) aA[ks] = lbase + (wr * 144 + c) * 128 + (((4 * ks + g) ^ (c >> 1)) << 4);
#pragma unroll
  for (int b = 0; b < 4; ++b) {
    const int row = 8 * g + q, ch = 8 * wc + 2 * pp + (b >> 1);
    aB[b] = lbase + B_BASE + row * 256 + ((ch ^ ((row & 3) | (row & 8))) << 4) + (((b ^ pp) & 1) << 3);
  }
  f32x4 acc[MB][4];
#pragma unroll
  for (int m = 0; m < MB; ++m)
#pragma unroll
    for (int b = 0; b < 4; ++b) acc[m][b] = (f32x4){0.f, 0.f, 0.f, 0.f};
  bf16x8 Af[2][MB];
  s16x4 Bl[2][4], Bh[2][4];
  constexpr int NRD = 8 + MB, NM = 4 * MB;
  auto readItem = [&](int set, int ks, int i) {
    if (i < 8) {
      const int b = i >> 1;
      if ((i & 1) == 0) Bl[set][b] = ldB(aB[b], ks * 8192);
      else Bh[set][b] = ldB(aB[b], ks * 8192 + 1024);
    } else {
      Af[set][i - 8] = ldA(aA[ks], (i - 8) * 2048);
    }
  };
  auto mma = [&](int set, int i) {
    const int m = i >> 2, b = i & 3;
    s16x8 bv = __builtin_shufflevector(Bl[set][b], Bh[set][b], 0, 1, 2, 3, 4, 5, 6, 7);
    acc[m][b] = __builtin_amdgcn_mfma_f32_16x16x32_bf16(__builtin_bit_cast(bf16x8, bv), Af[set][m], acc[m][b], 0, 0, 0);
  };
  BAR();
  BAR();
  if (wid == 0) STAMPW(1);
#pragma unroll
  for (int i = 0; i < NRD; ++i) readItem(0, 0, i);
  int buf = 0;
  for (int kt = 0; kt < NT; ++kt) {
    const bool wrap = (buf == 2);
    const int da = wrap ? -2 * A_BYTES : A_BYTES, db = wrap ? -2 * B_BYTES : B_BYTES;
    buf = wrap ? 0 : buf + 1;
    SB();
#pragma unroll
    for (int i = 0; i < NM; ++i) {
      mma(0, i); SB();
      if (i < NRD) { readItem(1, 1, i); SB(); }
      if (i == 18) { aA[0] += da; SB(); }
      if (i == 20) { aA[1] += da; SB(); }
      if (i >= 22 && i < 30 && !(i & 1)) { aB[(i - 22) >> 1] += db; SB(); }
    }
    WAITL0(); SB();
    BAR();
    SB();
#pragma unroll
    for (int i = 0; i < NM; ++i) {
      mma(1, i); SB();
      if (i < NRD) { readItem(0, 0, i); SB(); }
    }
  }
  if (wid == 0) STAMPW(2);
  float am = 0.f;
#pragma unroll
  for (int m = 0; m < MB; ++m)
#pragma unroll
    for (int b = 0; b < 4; ++b) {
      am = __builtin_fmaxf(__builtin_fmaxf(am, __builtin_fabsf(acc[m][b][0])), __builtin_fabsf(acc[m][b][1]));
      am = __builtin_fmaxf(__builtin_fmaxf(am, __builtin_fabsf(acc[m][b][2])), __builtin_fabsf(acc[m][b][3]));
    }
  const unsigned mu = wave_max_u32(__float_as_uint(am));
  if (lane == 0)
    __hip_atomic_store((u64*)(P.ws[P.rank] + WS_G1) + (blockIdx.x & 7) * 128 + (blockIdx.x >> 3) * 4 + wid, ((u64)xw[1] << 32) | mu, __ATOMIC_RELAXED, __HIP_MEMORY_SCOPE_SYSTEM);
  if (wid == 0) STAMPW(3);
  unsigned char* ob = P.out[T.dst] + T.ct * BN + wc * 64 + 16 * g;
  unsigned char* op = ob + (size_t)(T.src * M + (MB == 9 ? BM : 0) + wr * 128 + c) * NS;
  unsigned char* oe = ob + (size_t)(P.rank * M + T.grp * 32 + wr * 16 + c) * NS;
  const bool odd = g & 1;
  BAR();
  if (wid == 0) STAMPW(4);
  const float inv = 448.0f / __uint_as_float(xw[0]);
#pragma unroll
  for (int m = 0; m < MB; ++m) {
    unsigned wv[4];
#pragma unroll
    for (int b = 0; b < 4; ++b) {
      int t = __builtin_amdgcn_cvt_pk_fp8_f32(acc[m][b][0] * inv, acc[m][b][1] * inv, 0, false);
      t = __builtin_amdgcn_cvt_pk_fp8_f32(acc[m][b][2] * inv, acc[m][b][3] * inv, t, true);
      wv[b] = (unsigned)t;
    }
    u32x4 v;
    v[0] = odd ? wv[1] : wv[0]; v[1] = odd ? wv[0] : wv[1]; v[2] = odd ? wv[3] : wv[2]; v[3] = odd ? wv[2] : wv[3];
    asm volatile("global_store_dwordx4 %0, %1, off sc0 sc1\n\ts_nop 1" :: "v"(m < 8 ? op + (size_t)m * 16 * NS : oe), "v"(v) : "memory");
    SB();
  }
  if (wid == 0) STAMPW(5);
  WAITV(0);
  if (wid == 0) STAMPW(6);
  BAR();
}

__global__ __launch_bounds__(512, 2) void gemm_a2a_q(const Params P) {
  extern __shared__ __attribute__((aligned(16))) char smem[];
  LAS char* lds = (LAS char*)smem;
  const int tid = threadIdx.x, lane = tid & 63;
  const int wid = __builtin_amdgcn_readfirstlane(tid >> 6);
  const int bid = blockIdx.x;
  const int rank = P.rank;
  const int xcd = bid & 7, idx = bid >> 3;
  unsigned char* const myws = P.ws[rank];
  unsigned* const tmo = (unsigned*)(myws + WS_TMO);
  unsigned* const cnt = (unsigned*)(myws + WS_CNT + (size_t)bid * 128);
  if (wid == 0) STAMPW(0);

  if (idx < 2) {
    const int cw = xcd * 2 + idx;
    LAS unsigned* xw = (LAS unsigned*)(lds + X_BASE);
    if (wid < 7) {
      const int pd = (rank + 1 + wid) & 7;
      const char* sp = (const char*)(P.x + (size_t)(BM + 16 * cw + (lane >> 3)) * K);
      const char* sp0 = sp + (((lane & 7) ^ (lane >> 4)) << 4);
      const char* sp1 = sp + 8 * K * 2 + (((lane & 7) ^ (4 | (lane >> 4))) << 4);
      unsigned char* dp = P.ws[pd] + WS_XS + (size_t)rank * XS_STRIDE + cw * 2048 + lane * 16;
      unsigned* fp = (unsigned*)(P.ws[pd] + WS_XF + (rank * 16 + cw) * 128);
      u32x4 r0[8], r1[8];
#pragma unroll
      for (int s = 0; s < 8; ++s) { r0[s] = (u32x4){0u, 0u, 0u, 0u}; r1[s] = (u32x4){0u, 0u, 0u, 0u}; }
#define CLOAD(s) asm volatile("global_load_dwordx4 %0, %2, off offset:%4\n\tglobal_load_dwordx4 %1, %3, off offset:%4" \
                              : "+v"(r0[s]), "+v"(r1[s]) : "v"(sp0), "v"(sp1), "n"((s) * 128) : "memory")
#define CSTORE(s) do { asm volatile("global_store_dwordx4 %0, %1, off sc0 sc1\n\tglobal_store_dwordx4 %0, %2, off offset:1024 sc0 sc1\n\ts_nop 1" \
                                    :: "v"(dp), "v"(r0[s]), "v"(r1[s]) : "memory"); dp += XT_BYTES; } while (0)
#define CFLAG() do { ++fv; asm volatile("s_mov_b64 exec, 1\n\tglobal_store_dword %0, %1, off sc0 sc1\n\ts_mov_b64 exec, -1" :: "v"(fp), "v"(fv) : "memory"); } while (0)
      static_assert(CLG == 1, "the start-up below is written for CLG == 1");
      CLOAD(0); CLOAD(1); CLOAD(2);
      WAITV(4); CSTORE(0);
      WAITV(4); CSTORE(1);
#pragma unroll
      for (int s = 3; s < 8; ++s) CLOAD(s);
      sp0 += 1024; sp1 += 1024;
      CLOAD(0); CLOAD(1);
      BAR();
      unsigned fv = (xw[1] - 1u) * NT;
      WAITV(16); CFLAG();
      CSTORE(2); CLOAD(2); WAITV(19); CFLAG(); BAR();
      WAITV(10);
#pragma unroll
      for (int t = 3; t < 8; ++t) { CSTORE(t); CLOAD(t); WAITV(5 * CLG + 2); CFLAG();
        BAR(); }
      for (int k8 = 8; k8 < NT - 8; k8 += 8) {
        sp0 += 1024; sp1 += 1024;
#pragma unroll
        for (int i = 0; i < 8; ++i) { CSTORE(i); CLOAD(i); WAITV(5 * CLG + 2); CFLAG(); BAR(); }
      }
#pragma unroll
      for (int i = 0; i < 8; ++i) { CSTORE(i); WAITV(3 * CLG); CFLAG(); BAR(); }
      WAITV(0);
      fv += CLG - 1; CFLAG();
      if (wid == 0) { STAMPW(1); STAMPW(2); }
    } else {
      unsigned ticket = 0;
      if (lane == 0) ticket = __hip_atomic_fetch_add(cnt, 1u, __ATOMIC_RELAXED, __HIP_MEMORY_SCOPE_AGENT);
      const unsigned call = (unsigned)__builtin_amdgcn_readfirstlane((int)ticket) + 1u;
      if (lane == 0) xw[1] = call;
      WAITL0();
      BAR();
      for (int k = 2; k < NT; ++k) BAR();
      if (idx == 0) {
        const unsigned char* gp = myws + WS_G1 + xcd * 1024 + lane * 16;
        u64* dstg = (u64*)(P.ws[lane & 7] + WS_G2) + (call & 1) * (NR * 8) + rank * 8 + xcd;
        unsigned spins = 0;
        const u64 t0 = __builtin_amdgcn_s_memrealtime();
        for (;;) {
          u32x4 a;
          asm volatile("global_load_dwordx4 %0, %1, off sc0 sc1\n\ts_waitcnt vmcnt(0)" : "=&v"(a) : "v"(gp) : "memory");
          if (__all(lane < 4 || (a[1] == call && a[3] == call))) {
            const unsigned mx = wave_max_u32(lane < 4 ? 0u : max(a[0], a[2]));
            if (lane < NR) __hip_atomic_store(dstg, ((u64)call << 32) | mx, __ATOMIC_RELAXED, __HIP_MEMORY_SCOPE_SYSTEM);
            break;
          }
          if (give_up(tmo, call, t0, spins)) break;
          __builtin_amdgcn_s_sleep(1);
        }
      }
    }
    return;
  }

  const bool tall = idx >= 16;
  Tile T;
  if (!tall) { const int u = idx - 2; T.src = rank; T.dst = (rank + 1 + u % 7) & 7; T.ct = xcd * 2 + u / 7; T.grp = 0; }
  else { const int u = idx - 16; T.grp = (xcd >> 2) * 4 + (u >> 2); T.src = (rank + T.grp) & 7; T.dst = rank; T.ct = (xcd & 3) * 4 + (u & 3); }

  if (wid < 4) {
    if (tall) consumer<9>(P, T, lds, wid, lane); else consumer<8>(P, T, lds, wid, lane);
    return;
  }
  if (wid < 6) {
    if (tall) loader<true>(P, T, lds, wid - 4, lane); else loader<false>(P, T, lds, wid - 4, lane);
    return;
  }
  const bool remote = T.src != rank;
  if (wid == 6) {
    unsigned warm = 0;
    asm volatile("global_load_dword %0, %1, off\n\tglobal_load_dword %0, %2, off" : "+v"(warm)
                 : "v"((const char*)P.x + (size_t)bid * 16384 + lane * 128), "v"((const char*)P.x + (size_t)bid * 16384 + 8192 + lane * 128) : "memory");
    if (idx >= 30)
      asm volatile("global_load_dword %0, %1, off\n\tglobal_load_dword %0, %2, off" : "+v"(warm)
                   : "v"((const char*)P.x + (size_t)(bid - 240) * 16384 + lane * 128), "v"((const char*)P.x + (size_t)(bid - 240) * 16384 + 8192 + lane * 128) : "memory");
    BAR(); BAR();
    const u64 tb = __builtin_amdgcn_s_memrealtime();
    const int n1 = remote ? NB1 : NA1, p1 = remote ? PB1 : PA1, p2 = remote ? PB2 : PA2;
    for (int k = 1; k <= NT; ++k) {
      const int tgt = k <= n1 ? k * p1 : n1 * p1 + (k - n1) * p2;
      while (__builtin_amdgcn_s_memrealtime() - tb < (u64)tgt) __builtin_amdgcn_s_sleep(1);
      BAR();
    }
    asm volatile("s_waitcnt vmcnt(0)" :: "v"(warm) : "memory");
    return;
  }
  LAS unsigned* xw = (LAS unsigned*)(lds + X_BASE);
  unsigned ticket = 0, spins = 0;
  if (lane == 0) ticket = __hip_atomic_fetch_add(cnt, 1u, __ATOMIC_RELAXED, __HIP_MEMORY_SCOPE_AGENT);
  if (!remote) BAR();
  const unsigned call = (unsigned)__builtin_amdgcn_readfirstlane((int)ticket) + 1u;
  if (lane == 0) xw[1] = call;
  WAITL0();
  if (remote) {
    const unsigned* fp = (const unsigned*)(myws + WS_XF + (T.src * 16 + (lane & 15)) * 128);
    const unsigned base = (call - 1u) * NT;
    unsigned ready = 0;
    for (int k = -2; k < NT; ++k) {
      const unsigned need = (unsigned)min(k + 4, NT);
      if (ready < min(need + 2u, (unsigned)NT)) {
        const u64 t0 = __builtin_amdgcn_s_memrealtime();
        for (;;) {
          const unsigned d = __hip_atomic_load(fp, __ATOMIC_RELAXED, __HIP_MEMORY_SCOPE_SYSTEM) - base;
          ready = NT - wave_max_u32(d > (unsigned)NT ? (unsigned)NT : NT - d);
          if (ready >= need) break;
          if (give_up(tmo, call, t0, spins)) { ready = NT; break; }
          __builtin_amdgcn_s_sleep(1);
        }
      }
      BAR();
    }
  } else {
    for (int k = -1; k < NT; ++k) BAR();
  }
  {
    const u64* gp = (const u64*)(myws + WS_G2) + (call & 1) * (NR * 8) + lane;
    unsigned mx;
    const u64 t0 = __builtin_amdgcn_s_memrealtime();
    for (;;) {
      u64 a; unsigned tm;
      asm volatile("global_load_dwordx2 %0, %2, off sc0 sc1\n\tglobal_load_dword %1, %3, off sc1\n\ts_waitcnt vmcnt(0)"
                   : "=&v"(a), "=&v"(tm) : "v"(gp), "v"(tmo) : "memory");
      if (tm == call) { mx = 0x7fc00000u; break; }
      if (__all((unsigned)(a >> 32) == call)) { mx = wave_max_u32((unsigned)a); break; }
      if (__builtin_amdgcn_s_memrealtime() - t0 > DEADLINE_TICKS) __hip_atomic_store(tmo, call, __ATOMIC_RELAXED, __HIP_MEMORY_SCOPE_AGENT);
      __builtin_amdgcn_s_sleep(GSLEEP);
    }
    if (lane == 0) xw[0] = mx;
    WAITL0();
  }
  BAR();
  BAR();
  if (lane == 0) {
    if (T.dst != rank) {
      __hip_atomic_store((unsigned*)(P.ws[T.dst] + WS_DONE + (size_t)(rank * 16 + T.ct) * 128), call, __ATOMIC_RELAXED, __HIP_MEMORY_SCOPE_SYSTEM);
    } else if (((bid - 128) >> 4) != rank) {
      const unsigned* fl = (const unsigned*)(myws + WS_DONE + (size_t)(bid - 128) * 128);
      const u64 t1 = __builtin_amdgcn_s_memrealtime();
      while (__hip_atomic_load(fl, __ATOMIC_RELAXED, __HIP_MEMORY_SCOPE_SYSTEM) < call) {
        if (give_up(tmo, call, t1, spins)) { P.out[rank][(size_t)rank * M * NS] = 0x7f; break; }
        __builtin_amdgcn_s_sleep(1);
      }
    }
    STAMPW(7);
  }
}

extern "C" void kernel_launch(int rank, int nranks, ncclComm_t comm, hipStream_t stream,
                              void* const* d_in, const int* in_sizes, int n_in,
                              void* d_out, int out_size, const int* in_dtypes, int out_dtype,
                              void* const* d_out_all, void* const* d_ws_all, size_t ws_bytes) {
  (void)comm; (void)in_sizes; (void)n_in; (void)d_out; (void)out_size; (void)in_dtypes; (void)out_dtype;
  if (nranks != NR || ws_bytes < WS_END || (unsigned)rank >= (unsigned)NR) { fprintf(stderr, "unsupported configuration\n"); return; }
  static bool init = false;
  if (!init) {
    HIP_CHECK(hipFuncSetAttribute((const void*)gemm_a2a_q, hipFuncAttributeMaxDynamicSharedMemorySize, LDS_BYTES));
    init = true;
  }
  Params P;
  P.x = (const u16*)d_in[0]; P.w = (const u16*)d_in[1];
  for (int p = 0; p < NR; ++p) { P.out[p] = (unsigned char*)d_out_all[p]; P.ws[p] = (unsigned char*)d_ws_all[p]; }
  P.rank = rank; P.pad = 0;
  hipLaunchKernelGGL(gemm_a2a_q, dim3(NWG), dim3(512), LDS_BYTES, stream, P);
  HIP_CHECK(hipGetLastError());
}
```

```cpp
#include <hip/hip_runtime.h>
#include <rccl/rccl.h>
#include <cstdio>

#define HIP_CHECK(expr) do { hipError_t _e = (expr); if (_e != hipSuccess) { \
    fprintf(stderr, "HIP error %s at %s:%d\n", hipGetErrorString(_e), __FILE__, __LINE__); \
    return; } } while (0)

#define LAS __attribute__((address_space(3)))
typedef short s16x4 __attribute__((ext_vector_type(4)));
typedef short s16x8 __attribute__((ext_vector_type(8)));
typedef __bf16 bf16x8 __attribute__((ext_vector_type(8)));
typedef float f32x4 __attribute__((ext_vector_type(4)));
typedef unsigned u32x4 __attribute__((ext_vector_type(4)));
typedef unsigned short u16;
typedef unsigned long long u64;

constexpr int NR = 8;
constexpr int M = 512, N = 16384, K = 4096;
constexpr int NS = N / NR;
constexpr int BM = 256, BN = 128, BK = 64, NT = K / BK;
constexpr int NWG = 256;
constexpr int XT_BYTES = BM * BK * 2;
constexpr int A_BYTES = 288 * BK * 2, B_BYTES = BK * BN * 2;
constexpr int B_BASE = 3 * A_BYTES;
constexpr int X_BASE = B_BASE + 3 * B_BYTES;
constexpr int LDS_BYTES = X_BASE + 256;
#define AUXW 2
#define QT1 1600
#define QT2 3600
#define QA 84
#define QB 120
#define NE 3
#define ESLEEP 16
#define GSLEEP 2
#define CLG 1

constexpr size_t WS_CNT = 0;
constexpr size_t WS_TMO = WS_CNT + (size_t)NWG * 128;
constexpr size_t WS_G1 = WS_TMO + 256;
constexpr size_t WS_G2 = WS_G1 + 8 * 128 * 8;
constexpr size_t WS_DONE = WS_G2 + 2 * NR * 8 * 8;
constexpr size_t WS_XF = WS_DONE + (size_t)NR * 16 * 128;
constexpr size_t WS_EF = WS_XF + (size_t)NR * 16 * 128;
constexpr size_t WS_XS = 1u << 20;
constexpr size_t XS_STRIDE = (size_t)NT * XT_BYTES;
constexpr size_t WS_ES = WS_XS + (size_t)NR * XS_STRIDE;
constexpr size_t WS_END = WS_ES + (size_t)2 * NR * 16 * 4 * (NE ? NE : 1) * 2048;
static_assert(WS_EF + NR * 16 * 4 * 128 <= WS_XS, "layout");
constexpr u64 DEADLINE_TICKS = 40000000ull;

struct Params {
  const u16* x; const u16* w;
  unsigned char* out[NR];
  unsigned char* ws[NR];
  int rank; int pad;
};

#define STAMPW(k) do {} while (0)
#define SB() __builtin_amdgcn_sched_barrier(0)
#define BAR() do { asm volatile("" ::: "memory"); __builtin_amdgcn_s_barrier(); asm volatile("" ::: "memory"); } while (0)
#define WAITV(n) asm volatile("s_waitcnt vmcnt(%0)" :: "n"(n) : "memory")
#define WAITL0() asm volatile("s_waitcnt lgkmcnt(0)" ::: "memory")
#define DI __device__ __forceinline__

DI bf16x8 ldA(unsigned addr, int off) { return *(LAS bf16x8*)(size_t)(addr + off); }
DI s16x4 ldB(unsigned addr, int off) { return __builtin_amdgcn_ds_read_tr16_b64_v4i16((LAS s16x4*)(size_t)(addr + off)); }
DI unsigned wave_max_u32(unsigned v) {
  v = max(v, (unsigned)__builtin_amdgcn_update_dpp(0, (int)v, 0x111, 0xf, 0xf, false));
  v = max(v, (unsigned)__builtin_amdgcn_update_dpp(0, (int)v, 0x112, 0xf, 0xf, false));
  v = max(v, (unsigned)__builtin_amdgcn_update_dpp(0, (int)v, 0x114, 0xf, 0xf, false));
  v = max(v, (unsigned)__builtin_amdgcn_update_dpp(0, (int)v, 0x118, 0xf, 0xf, false));
  const unsigned a = __builtin_amdgcn_readlane(v, 15), b = __builtin_amdgcn_readlane(v, 31);
  const unsigned c = __builtin_amdgcn_readlane(v, 47), d = __builtin_amdgcn_readlane(v, 63);
  return max(max(a, b), max(c, d));
}
DI bool give_up(unsigned* tmo, unsigned call, u64 t0, unsigned& spins) {
  if ((++spins & 7u) != 0u) return false;
  if (__hip_atomic_load(tmo, __ATOMIC_RELAXED, __HIP_MEMORY_SCOPE_AGENT) == call || __builtin_amdgcn_s_memrealtime() - t0 > DEADLINE_TICKS) {
    __hip_atomic_store(tmo, call, __ATOMIC_RELAXED, __HIP_MEMORY_SCOPE_AGENT);
    return true;
  }
  return false;
}

DI unsigned pkbf(float a, float b) { unsigned r; asm("v_cvt_pk_bf16_f32 %0, %1, %2" : "=v"(r) : "v"(a), "v"(b)); return r; }
DI unsigned q4(float a, float b, float c, float d, float inv) {
  const int t = __builtin_amdgcn_cvt_pk_fp8_f32(a * inv, b * inv, 0, false);
  return (unsigned)__builtin_amdgcn_cvt_pk_fp8_f32(c * inv, d * inv, t, true);
}
DI void store_row16(unsigned char* p, unsigned w0, unsigned w1, unsigned w2, unsigned w3, bool odd) {
  const u32x4 v = {odd ? w1 : w0, odd ? w0 : w1, odd ? w3 : w2, odd ? w2 : w3};
  asm volatile("global_store_dwordx4 %0, %1, off sc0 sc1\n\ts_nop 1" :: "v"(p), "v"(v) : "memory");
}

struct Tile { int src, dst, ct, grp; };

template <bool TALL>
DI void loader(const Params& P, const Tile T, LAS char* lds, const int p, const int lane) {
  const int rank = P.rank;
  const bool remote = TALL && T.src != rank;
  const void* base = remote ? (const void*)(P.ws[rank] + WS_XS + (size_t)T.src * XS_STRIDE) : (const void*)(P.x + (size_t)(TALL ? BM : 0) * K);
  __amdgpu_buffer_rsrc_t rA = __builtin_amdgcn_make_buffer_rsrc((void*)base, 0, BM * K * 2, 0x00020000);
  __amdgpu_buffer_rsrc_t rE = __builtin_amdgcn_make_buffer_rsrc((void*)(P.x + (size_t)T.grp * 32 * K), 0, 32 * K * 2, 0x00020000);
  __amdgpu_buffer_rsrc_t rB = __builtin_amdgcn_make_buffer_rsrc((void*)(P.w + (size_t)T.dst * NS + T.ct * BN), 0, (K - 1) * N * 2 + BN * 2, 0x00020000);
  const unsigned pstr = remote ? 1024u : 8u * K * 2, kstr = remote ? (unsigned)XT_BYTES : BK * 2u;
  const unsigned voL = (unsigned)(((lane >> 3) + 8 * p) * (K * 2) + (((lane & 7) ^ ((p << 2) | (lane >> 4))) << 4));
  const unsigned voA = remote ? (unsigned)lane * 16u + p * 1024u : voL;
  const unsigned voB = (unsigned)(((lane >> 4) + 8 * p) * (N * 2) + (((lane & 15) ^ ((lane >> 4) | (p << 3))) << 4));
  auto issueA = [&](int kt, int buf) {
#pragma unroll
    for (int j = 0; j < 16; ++j)
      __builtin_amdgcn_raw_ptr_buffer_load_lds(rA, (LAS void*)(lds + buf * A_BYTES + (p + 2 * j + 2 * (j >> 3)) * 1024), 16, voA, kt * kstr + 2 * j * pstr, 0, 17);
    if (TALL) {
#pragma unroll
      for (int j = 0; j < 2; ++j)
        __builtin_amdgcn_raw_ptr_buffer_load_lds(rE, (LAS void*)(lds + buf * A_BYTES + (16 + p + 18 * j) * 1024), 16, voL, kt * (BK * 2) + j * 16 * K * 2, 0, 0);
    }
  };
  auto issueB = [&](int kt, int buf) {
#pragma unroll
    for (int j = 0; j < 8; ++j) {
      const int pc = (j & 1) + 4 * (j >> 1);
      __builtin_amdgcn_raw_ptr_buffer_load_lds(rB, (LAS void*)(lds + B_BASE + buf * B_BYTES + (pc + 2 * p) * 1024), 16, voB, (kt * BK + pc * 4) * (N * 2), 0, TALL ? 0 : AUXW);
    }
  };
  constexpr int NA = TALL ? 18 : 16;
  issueB(0, 0); issueB(1, 1);
  BAR();
  issueA(0, 0); issueA(1, 1);
  WAITV(NA);
  BAR();
  int buf = 2;
  for (int kt = 0; kt < NT - 2; ++kt) {
    issueB(kt + 2, buf); issueA(kt + 2, buf);
    WAITV(NA + 8);
    buf = (buf == 2) ? 0 : buf + 1;
    BAR();
  }
  WAITV(0);
  BAR(); BAR();
}

template <int MB>
DI void consumer(const Params& P, const Tile T, LAS char* lds, const int wid, const int lane) {
  const unsigned lbase = (unsigned)(size_t)lds;
  LAS unsigned* xw = (LAS unsigned*)(lds + X_BASE);
  const int wr = wid >> 1, wc = wid & 1;
  const int g = lane >> 4, c = lane & 15, q = (lane & 15) >> 2, pp = lane & 3;
  unsigned aA[2], aB[4];
#pragma unroll
  for (int ks = 0; ks < 2; ++ks) aA[ks] = lbase + (wr * 144 + c) * 128 + (((4 * ks + g) ^ (c >> 1)) << 4);
#pragma unroll
  for (int b = 0; b < 4; ++b) {
    const int row = 8 * g + q, ch = 8 * wc + 2 * pp + (b >> 1);
    aB[b] = lbase + B_BASE + row * 256 + ((ch ^ ((row & 3) | (row & 8))) << 4) + (((b ^ pp) & 1) << 3);
  }
  f32x4 acc[MB][4];
#pragma unroll
  for (int m = 0; m < MB; ++m)
#pragma unroll
    for (int b = 0; b < 4; ++b) acc[m][b] = (f32x4){0.f, 0.f, 0.f, 0.f};
  bf16x8 Af[2][MB];
  s16x4 Bl[2][4], Bh[2][4];
  constexpr int NRD = 8 + MB, NM = 4 * MB;
  auto readItem = [&](int set, int ks, int i) {
    if (i < 8) {
      const int b = i >> 1;
      if ((i & 1) == 0) Bl[set][b] = ldB(aB[b], ks * 8192);
      else Bh[set][b] = ldB(aB[b], ks * 8192 + 1024);
    } else {
      Af[set][i - 8] = ldA(aA[ks], (i - 8) * 2048);
    }
  };
  auto mma = [&](int set, int i) {
    const int m = i >> 2, b = i & 3;
    s16x8 bv = __builtin_shufflevector(Bl[set][b], Bh[set][b], 0, 1, 2, 3, 4, 5, 6, 7);
    acc[m][b] = __builtin_amdgcn_mfma_f32_16x16x32_bf16(__builtin_bit_cast(bf16x8, bv), Af[set][m], acc[m][b], 0, 0, 0);
  };
  BAR();
  BAR();
  if (wid == 0) STAMPW(1);
#pragma unroll
  for (int i = 0; i < NRD; ++i) readItem(0, 0, i);
  int buf = 0;
  for (int kt = 0; kt < NT; ++kt) {
    const bool wrap = (buf == 2);
    const int da = wrap ? -2 * A_BYTES : A_BYTES, db = wrap ? -2 * B_BYTES : B_BYTES;
    buf = wrap ? 0 : buf + 1;
    SB();
#pragma unroll
    for (int i = 0; i < NM; ++i) {
      mma(0, i); SB();
      if (i < NRD) { readItem(1, 1, i); SB(); }
      if (i == 18) { aA[0] += da; SB(); }
      if (i == 20) { aA[1] += da; SB(); }
      if (i >= 22 && i < 30 && !(i & 1)) { aB[(i - 22) >> 1] += db; SB(); }
    }
    WAITL0(); SB();
    BAR();
    SB();
#pragma unroll
    for (int i = 0; i < NM; ++i) {
      mma(1, i); SB();
      if (i < NRD) { readItem(0, 0, i); SB(); }
    }
  }
  if (wid == 0) STAMPW(2);
  float am = 0.f;
#pragma unroll
  for (int m = 0; m < MB; ++m)
#pragma unroll
    for (int b = 0; b < 4; ++b) {
      am = __builtin_fmaxf(__builtin_fmaxf(am, __builtin_fabsf(acc[m][b][0])), __builtin_fabsf(acc[m][b][1]));
      am = __builtin_fmaxf(__builtin_fmaxf(am, __builtin_fabsf(acc[m][b][2])), __builtin_fabsf(acc[m][b][3]));
    }
  const unsigned mu = wave_max_u32(__float_as_uint(am));
  const unsigned call = xw[1];
  if (lane == 0)
    __hip_atomic_store((u64*)(P.ws[P.rank] + WS_G1) + (blockIdx.x & 7) * 128 + (blockIdx.x >> 3) * 4 + wid, ((u64)call << 32) | mu, __ATOMIC_RELAXED, __HIP_MEMORY_SCOPE_SYSTEM);
  if (wid == 0) STAMPW(3);
  constexpr int M0 = MB == 8 ? NE : 0;
  if (M0 > 0) {
    const int slot = (P.rank * 16 + T.ct) * 4 + wid;
    unsigned char* ep = P.ws[T.dst] + WS_ES + ((size_t)(call & 1) * (NR * 64) + slot) * (NE * 2048) + lane * 16;
#pragma unroll
    for (int m = 0; m < M0; ++m)
#pragma unroll
      for (int b = 0; b < 4; b += 2) {
        const u32x4 v = {pkbf(acc[m][b][0], acc[m][b][1]), pkbf(acc[m][b][2], acc[m][b][3]), pkbf(acc[m][b + 1][0], acc[m][b + 1][1]), pkbf(acc[m][b + 1][2], acc[m][b + 1][3])};
        asm volatile("global_store_dwordx4 %0, %1, off sc0 sc1\n\ts_nop 1" :: "v"(ep + m * 2048 + b * 512), "v"(v) : "memory");
        WAITV(0);
        if (ESLEEP) __builtin_amdgcn_s_sleep(ESLEEP);
      }
    if (lane == 0) __hip_atomic_store((unsigned*)(P.ws[T.dst] + WS_EF + (size_t)slot * 128), call, __ATOMIC_RELAXED, __HIP_MEMORY_SCOPE_SYSTEM);
    if (wid == 0) STAMPW(5);
  }
  unsigned char* ob = P.out[T.dst] + T.ct * BN + wc * 64 + 16 * g;
  unsigned char* op = ob + (size_t)(T.src * M + (MB == 9 ? BM : 0) + wr * 128 + c) * NS;
  unsigned char* oe = ob + (size_t)(P.rank * M + T.grp * 32 + wr * 16 + c) * NS;
  BAR();
  if (wid == 0) STAMPW(4);
  const float inv = 448.0f / __uint_as_float(xw[0]);
#pragma unroll
  for (int m = M0; m < MB; ++m) {
    unsigned w[4];
#pragma unroll
    for (int b = 0; b < 4; ++b) w[b] = q4(acc[m][b][0], acc[m][b][1], acc[m][b][2], acc[m][b][3], inv);
    store_row16(m < 8 ? op + (size_t)m * 16 * NS : oe, w[0], w[1], w[2], w[3], g & 1);
    SB();
  }
  if (MB == 8) {
    WAITV(0);
    if (wid == 0) STAMPW(6);
    BAR();
  }
}

struct Early { u32x4 v[NE ? 2 * NE : 1]; };
DI void early_fetch(const Params& P, Early& E, const int f, const int w, const int lane, const unsigned call) {
  unsigned char* const myws = P.ws[P.rank];
  const unsigned* fl = (const unsigned*)(myws + WS_EF + (size_t)(f * 4 + w) * 128);
  unsigned spins = 0;
  const u64 t0 = __builtin_amdgcn_s_memrealtime();
  while (__hip_atomic_load(fl, __ATOMIC_RELAXED, __HIP_MEMORY_SCOPE_SYSTEM) < call) {
    if (give_up((unsigned*)(myws + WS_TMO), call, t0, spins)) break;
    __builtin_amdgcn_s_sleep(2);
  }
  const unsigned char* ep = myws + WS_ES + ((size_t)(call & 1) * (NR * 64) + f * 4 + w) * (NE * 2048) + lane * 16;
#pragma unroll
  for (int m = 0; m < NE; ++m)
    asm volatile("global_load_dwordx4 %0, %2, off sc0 sc1\n\tglobal_load_dwordx4 %1, %2, off offset:1024 sc0 sc1" : "=&v"(E.v[2 * m]), "=&v"(E.v[2 * m + 1]) : "v"(ep + m * 2048) : "memory");
  WAITV(0);
#pragma unroll
  for (int m = 0; m < 2 * NE; ++m) asm volatile("" : "+v"(E.v[m]));
}
DI void early_store(const Params& P, const Early& E, const int f, const int w, const int lane, const float inv) {
  unsigned char* op = P.out[P.rank] + (size_t)((f >> 4) * M + (w >> 1) * 128 + (lane & 15)) * NS + (f & 15) * BN + (w & 1) * 64 + (lane & 48);
#pragma unroll
  for (int m = 0; m < NE; ++m) {
    unsigned q[4];
#pragma unroll
    for (int b = 0; b < 4; ++b) {
      const unsigned lo = E.v[2 * m + (b >> 1)][(b & 1) * 2], hi = E.v[2 * m + (b >> 1)][(b & 1) * 2 + 1];
      q[b] = q4(__uint_as_float(lo << 16), __uint_as_float(lo & 0xffff0000u), __uint_as_float(hi << 16), __uint_as_float(hi & 0xffff0000u), inv);
    }
    store_row16(op + (size_t)m * 16 * NS, q[0], q[1], q[2], q[3], (lane >> 4) & 1);
  }
}

__global__ __launch_bounds__(512, 2) void gemm_a2a_q(const Params P) {
  extern __shared__ __attribute__((aligned(16))) char smem[];
  LAS char* lds = (LAS char*)smem;
  const int tid = threadIdx.x, lane = tid & 63;
  const int wid = __builtin_amdgcn_readfirstlane(tid >> 6);
  const int bid = blockIdx.x;
  const int rank = P.rank;
  const int xcd = bid & 7, idx = bid >> 3;
  unsigned char* const myws = P.ws[rank];
  unsigned* const tmo = (unsigned*)(myws + WS_TMO);
  unsigned* const cnt = (unsigned*)(myws + WS_CNT + (size_t)bid * 128);
  if (wid == 0) STAMPW(0);

  if (idx < 2) {
    const int cw = xcd * 2 + idx;
    LAS unsigned* xw = (LAS unsigned*)(lds + X_BASE);
    if (wid < 7) {
      const int pd = (rank + 1 + wid) & 7;
      const char* sp = (const char*)(P.x + (size_t)(BM + 16 * cw + (lane >> 3)) * K);
      const char* sp0 = sp + (((lane & 7) ^ (lane >> 4)) << 4);
      const char* sp1 = sp + 8 * K * 2 + (((lane & 7) ^ (4 | (lane >> 4))) << 4);
      unsigned char* dp = P.ws[pd] + WS_XS + (size_t)rank * XS_STRIDE + cw * 2048 + lane * 16;
      unsigned* fp = (unsigned*)(P.ws[pd] + WS_XF + (rank * 16 + cw) * 128);
      u32x4 r0[8], r1[8];
#pragma unroll
      for (int s = 0; s < 8; ++s) { r0[s] = (u32x4){0u, 0u, 0u, 0u}; r1[s] = (u32x4){0u, 0u, 0u, 0u}; }
#define CLOAD(s) asm volatile("global_load_dwordx4 %0, %2, off offset:%4\n\tglobal_load_dwordx4 %1, %3, off offset:%4" \
                              : "+v"(r0[s]), "+v"(r1[s]) : "v"(sp0), "v"(sp1), "n"((s) * 128) : "memory")
#define CSTORE(s) do { asm volatile("global_store_dwordx4 %0, %1, off sc0 sc1\n\tglobal_store_dwordx4 %0, %2, off offset:1024 sc0 sc1\n\ts_nop 1" \
                                    :: "v"(dp), "v"(r0[s]), "v"(r1[s]) : "memory"); dp += XT_BYTES; } while (0)
#define CFLAG() do { ++fv; asm volatile("s_mov_b64 exec, 1\n\tglobal_store_dword %0, %1, off sc0 sc1\n\ts_mov_b64 exec, -1" :: "v"(fp), "v"(fv) : "memory"); } while (0)
      static_assert(CLG == 1, "the start-up below is written for CLG == 1");
      CLOAD(0); CLOAD(1); CLOAD(2);
      WAITV(4); CSTORE(0);
      WAITV(4); CSTORE(1);
#pragma unroll
      for (int s = 3; s < 8; ++s) CLOAD(s);
      sp0 += 1024; sp1 += 1024;
      CLOAD(0); CLOAD(1);
      BAR();
      unsigned fv = (xw[1] - 1u) * NT;
      WAITV(16); CFLAG();
      CSTORE(2); CLOAD(2); WAITV(19); CFLAG(); BAR();
      WAITV(10);
#pragma unroll
      for (int t = 3; t < 8; ++t) { CSTORE(t); CLOAD(t); WAITV(5 * CLG + 2); CFLAG();
        BAR(); }
      for (int k8 = 8; k8 < NT - 8; k8 += 8) {
        sp0 += 1024; sp1 += 1024;
#pragma unroll
        for (int i = 0; i < 8; ++i) { CSTORE(i); CLOAD(i); WAITV(5 * CLG + 2); CFLAG(); BAR(); }
      }
#pragma unroll
      for (int i = 0; i < 8; ++i) { CSTORE(i); WAITV(3 * CLG); CFLAG(); BAR(); }
      WAITV(0);
      fv += CLG - 1; CFLAG();
      if (wid == 0) { STAMPW(1); STAMPW(2); }
    } else {
      unsigned ticket = 0;
      if (lane == 0) ticket = __hip_atomic_fetch_add(cnt, 1u, __ATOMIC_RELAXED, __HIP_MEMORY_SCOPE_AGENT);
      const unsigned call = (unsigned)__builtin_amdgcn_readfirstlane((int)ticket) + 1u;
      if (lane == 0) xw[1] = call;
      WAITL0();
      BAR();
      for (int k = 2; k < NT; ++k) BAR();
      if (idx == 0) {
        const unsigned char* gp = myws + WS_G1 + xcd * 1024 + lane * 16;
        u64* dstg = (u64*)(P.ws[lane & 7] + WS_G2) + (call & 1) * (NR * 8) + rank * 8 + xcd;
        unsigned spins = 0;
        const u64 t0 = __builtin_amdgcn_s_memrealtime();
        for (;;) {
          u32x4 a;
          asm volatile("global_load_dwordx4 %0, %1, off sc0 sc1\n\ts_waitcnt vmcnt(0)" : "=&v"(a) : "v"(gp) : "memory");
          if (__all(lane < 4 || (a[1] == call && a[3] == call))) {
            const unsigned mx = wave_max_u32(lane < 4 ? 0u : max(a[0], a[2]));
            if (lane < NR) __hip_atomic_store(dstg, ((u64)call << 32) | mx, __ATOMIC_RELAXED, __HIP_MEMORY_SCOPE_SYSTEM);
            break;
          }
          if (give_up(tmo, call, t0, spins)) break;
          __builtin_amdgcn_s_sleep(1);
        }
      }
    }
    return;
  }

  const bool tall = idx >= 16;
  Tile T;
  if (!tall) { const int u = idx - 2; T.src = rank; T.dst = (rank + 1 + u % 7) & 7; T.ct = xcd * 2 + u / 7; T.grp = 0; }
  else { const int u = idx - 16; T.grp = (xcd >> 2) * 4 + (u >> 2); T.src = (rank + T.grp) & 7; T.dst = rank; T.ct = (xcd & 3) * 4 + (u & 3); }

  if (wid < 4) {
    if (tall) consumer<9>(P, T, lds, wid, lane); else consumer<8>(P, T, lds, wid, lane);
    return;
  }
  LAS unsigned* xw = (LAS unsigned*)(lds + X_BASE);
  const int f = bid - 128;
  const bool incoming = tall && (f >> 4) != rank;
  const bool remote = T.src != rank;
  Early E;
  if (wid < 6) {
    if (tall) loader<true>(P, T, lds, wid - 4, lane); else loader<false>(P, T, lds, wid - 4, lane);
  } else if (wid == 6) {
    const u64 tb = __builtin_amdgcn_s_memrealtime();
    unsigned warm = 0;
    asm volatile("global_load_dword %0, %1, off\n\tglobal_load_dword %0, %2, off" : "+v"(warm)
                 : "v"((const char*)P.x + (size_t)bid * 16384 + lane * 128), "v"((const char*)P.x + (size_t)bid * 16384 + 8192 + lane * 128) : "memory");
    if (idx >= 30)
      asm volatile("global_load_dword %0, %1, off\n\tglobal_load_dword %0, %2, off" : "+v"(warm)
                   : "v"((const char*)P.x + (size_t)(bid - 240) * 16384 + lane * 128), "v"((const char*)P.x + (size_t)(bid - 240) * 16384 + 8192 + lane * 128) : "memory");
    BAR(); BAR();
    const int hold = !tall ? QA : remote ? QB : 0;
    int tl = (int)(__builtin_amdgcn_s_memrealtime() - tb);
    for (int k = 1; k <= NT; ++k) {
      if (tl >= QT1 && tl < QT2) {
        tl += hold;
        while ((int)(__builtin_amdgcn_s_memrealtime() - tb) < tl) __builtin_amdgcn_s_sleep(1);
      }
      BAR();
      tl = max(tl, (int)(__builtin_amdgcn_s_memrealtime() - tb) - 40);
    }
    asm volatile("s_waitcnt vmcnt(0)" :: "v"(warm) : "memory");
  }
  if (wid < 7) {
    if (NE > 0 && incoming) {
      early_fetch(P, E, f, wid - 4, lane, xw[1]);
      const bool dead = __hip_atomic_load(tmo, __ATOMIC_RELAXED, __HIP_MEMORY_SCOPE_AGENT) == xw[1];
      BAR();
      early_store(P, E, f, wid - 4, lane, dead ? __uint_as_float(0x7fc00000u) : 448.0f / __uint_as_float(xw[0]));
    }
    return;
  }
  unsigned ticket = 0, spins = 0;
  if (lane == 0) ticket = __hip_atomic_fetch_add(cnt, 1u, __ATOMIC_RELAXED, __HIP_MEMORY_SCOPE_AGENT);
  if (!remote) BAR();
  const unsigned call = (unsigned)__builtin_amdgcn_readfirstlane((int)ticket) + 1u;
  if (lane == 0) xw[1] = call;
  WAITL0();
  if (remote) {
    const unsigned* fp = (const unsigned*)(myws + WS_XF + (T.src * 16 + (lane & 15)) * 128);
    const unsigned base = (call - 1u) * NT;
    unsigned ready = 0;
    for (int k = -2; k < NT; ++k) {
      const unsigned need = (unsigned)min(k + 4, NT);
      if (ready < min(need + 2u, (unsigned)NT)) {
        const u64 t0 = __builtin_amdgcn_s_memrealtime();
        for (;;) {
          const unsigned d = __hip_atomic_load(fp, __ATOMIC_RELAXED, __HIP_MEMORY_SCOPE_SYSTEM) - base;
          ready = NT - wave_max_u32(d > (unsigned)NT ? (unsigned)NT : NT - d);
          if (ready >= need) break;
          if (give_up(tmo, call, t0, spins)) { ready = NT; break; }
          __builtin_amdgcn_s_sleep(1);
        }
      }
      BAR();
    }
  } else {
    for (int k = -1; k < NT; ++k) BAR();
  }
  if (NE > 0 && incoming) early_fetch(P, E, f, 3, lane, call);
  unsigned mx;
  {
    const u64* gp = (const u64*)(myws + WS_G2) + (call & 1) * (NR * 8) + lane;
    const u64 t0 = __builtin_amdgcn_s_memrealtime();
    for (;;) {
      u64 a; unsigned tm;
      asm volatile("global_load_dwordx2 %0, %2, off sc0 sc1\n\tglobal_load_dword %1, %3, off sc1\n\ts_waitcnt vmcnt(0)"
                   : "=&v"(a), "=&v"(tm) : "v"(gp), "v"(tmo) : "memory");
      if (tm == call) { mx = 0x7fc00000u; break; }
      if (__all((unsigned)(a >> 32) == call)) { mx = wave_max_u32((unsigned)a); break; }
      if (__builtin_amdgcn_s_memrealtime() - t0 > DEADLINE_TICKS) __hip_atomic_store(tmo, call, __ATOMIC_RELAXED, __HIP_MEMORY_SCOPE_AGENT);
      __builtin_amdgcn_s_sleep(GSLEEP);
    }
    if (lane == 0) xw[0] = mx;
    WAITL0();
  }
  BAR();
  if (!tall) {
    BAR();
    if (lane == 0) __hip_atomic_store((unsigned*)(P.ws[T.dst] + WS_DONE + (size_t)(rank * 16 + T.ct) * 128), call, __ATOMIC_RELAXED, __HIP_MEMORY_SCOPE_SYSTEM);
  } else if (incoming) {
    if (NE > 0) early_store(P, E, f, 3, lane, 448.0f / __uint_as_float(mx));
    const unsigned* fl = (const unsigned*)(myws + WS_DONE + (size_t)f * 128);
    const u64 t1 = __builtin_amdgcn_s_memrealtime();
    while (__hip_atomic_load(fl, __ATOMIC_RELAXED, __HIP_MEMORY_SCOPE_SYSTEM) < call) {
      if (give_up(tmo, call, t1, spins)) { if (lane == 0) P.out[rank][(size_t)rank * M * NS] = 0x7f; break; }
      __builtin_amdgcn_s_sleep(1);
    }
  }
  STAMPW(7);
}

extern "C" void kernel_launch(int rank, int nranks, ncclComm_t comm, hipStream_t stream,
                              void* const* d_in, const int* in_sizes, int n_in,
                              void* d_out, int out_size, const int* in_dtypes, int out_dtype,
                              void* const* d_out_all, void* const* d_ws_all, size_t ws_bytes) {
  (void)comm; (void)in_sizes; (void)n_in; (void)d_out; (void)out_size; (void)in_dtypes; (void)out_dtype;
  if (nranks != NR || ws_bytes < WS_END || (unsigned)rank >= (unsigned)NR) { fprintf(stderr, "unsupported configuration\n"); return; }
  static bool init = false;
  if (!init) {
    HIP_CHECK(hipFuncSetAttribute((const void*)gemm_a2a_q, hipFuncAttributeMaxDynamicSharedMemorySize, LDS_BYTES));
    init = true;
  }
  Params P;
  P.x = (const u16*)d_in[0]; P.w = (const u16*)d_in[1];
  for (int p = 0; p < NR; ++p) { P.out[p] = (unsigned char*)d_out_all[p]; P.ws[p] = (unsigned char*)d_ws_all[p]; }
  P.rank = rank; P.pad = 0;
  hipLaunchKernelGGL(gemm_a2a_q, dim3(NWG), dim3(512), LDS_BYTES, stream, P);
  HIP_CHECK(hipGetLastError());
}
```

```cpp
#include <hip/hip_runtime.h>
#include <rccl/rccl.h>
#include <cstdio>

#define HIP_CHECK(expr) do { hipError_t _e = (expr); if (_e != hipSuccess) { \
    fprintf(stderr, "HIP error %s at %s:%d\n", hipGetErrorString(_e), __FILE__, __LINE__); \
    return; } } while (0)

#define LAS __attribute__((address_space(3)))
#define GAS __attribute__((address_space(1)))
typedef short s16x4 __attribute__((ext_vector_type(4)));
typedef short s16x8 __attribute__((ext_vector_type(8)));
typedef __bf16 bf16x8 __attribute__((ext_vector_type(8)));
typedef float f32x4 __attribute__((ext_vector_type(4)));
typedef unsigned u32x4 __attribute__((ext_vector_type(4)));
typedef unsigned short u16;
typedef unsigned long long u64;
typedef unsigned char u8;

constexpr int NR = 8;
constexpr int M = 512, N = 16384, K = 4096;
constexpr int NS = N / NR;
constexpr int BM = 256, BN = 128, BK = 64, NT = K / BK;
constexpr int NWG = 256;
constexpr int XT_BYTES = BM * BK * 2;
constexpr int A_BYTES = 288 * BK * 2, B_BYTES = BK * BN * 2;
constexpr int B_BASE = 3 * A_BYTES;
constexpr int X_BASE = B_BASE + 3 * B_BYTES;
constexpr int LDS_BYTES = X_BASE + 256;
#define AUXW 2
#define QT1 1750
#define QT2 3550
#define QA 82
#define QB 114
#define NE 3
#define ESLEEP 16

constexpr size_t WS_CNT = 0;
constexpr size_t WS_TMO = WS_CNT + (size_t)NWG * 128;
constexpr size_t WS_G1 = WS_TMO + 256;
constexpr size_t WS_G2 = WS_G1 + 8 * 128 * 8;
constexpr size_t WS_DONE = WS_G2 + 2 * NR * 8 * 8;
constexpr size_t WS_XF = WS_DONE + (size_t)NR * 16 * 128;
constexpr size_t WS_EF = WS_XF + (size_t)NR * 16 * 128;
constexpr size_t WS_XS = 1u << 20;
constexpr size_t XS_STRIDE = (size_t)NT * XT_BYTES;
constexpr size_t WS_ES = WS_XS + (size_t)NR * XS_STRIDE;
constexpr size_t WS_END = WS_ES + (size_t)2 * NR * 64 * 8192;
static_assert(WS_EF + NR * 64 * 128 <= WS_XS && NE >= 0 && NE <= 4, "layout");
constexpr u64 DEADLINE_TICKS = 40000000ull;

struct Params {
  const u16* x; const u16* w;
  int rank; int pad;
  u8* ws[NR];
  u8* out[NR];
};
struct Ctx { const u16* x; const u16* w; u8* ws; u8* out; u8* pws; u8* pout; int rank; };
struct Tile { int src, dst, ct, grp; };

#define STAMPW(k) do {} while (0)
#define SB() __builtin_amdgcn_sched_barrier(0)
#define BAR() do { asm volatile("" ::: "memory"); __builtin_amdgcn_s_barrier(); asm volatile("" ::: "memory"); } while (0)
#define WAITV(n) asm volatile("s_waitcnt vmcnt(%0)" :: "n"(n) : "memory")
#define WAITL0() asm volatile("s_waitcnt lgkmcnt(0)" ::: "memory")
#define DI __device__ __forceinline__

#define SYS __HIP_MEMORY_SCOPE_SYSTEM
#define AGT __HIP_MEMORY_SCOPE_AGENT
template <int SC> DI unsigned ld32(const void* p) { return __hip_atomic_load((const GAS unsigned*)(size_t)p, __ATOMIC_RELAXED, SC); }
template <int SC> DI void st32(void* p, unsigned v) { __hip_atomic_store((GAS unsigned*)(size_t)p, v, __ATOMIC_RELAXED, SC); }
DI void st64(void* p, unsigned tag, unsigned v) { __hip_atomic_store((GAS u64*)(size_t)p, ((u64)tag << 32) | v, __ATOMIC_RELAXED, SYS); }
DI unsigned draw(void* p) { return __hip_atomic_fetch_add((GAS unsigned*)(size_t)p, 1u, __ATOMIC_RELAXED, AGT); }
DI bf16x8 ldA(unsigned addr, int off) { return *(LAS bf16x8*)(size_t)(addr + off); }
DI s16x4 ldB(unsigned addr, int off) { return __builtin_amdgcn_ds_read_tr16_b64_v4i16((LAS s16x4*)(size_t)(addr + off)); }
DI unsigned wave_max_u32(unsigned v) {
  v = max(v, (unsigned)__builtin_amdgcn_update_dpp(0, (int)v, 0x111, 0xf, 0xf, false));
  v = max(v, (unsigned)__builtin_amdgcn_update_dpp(0, (int)v, 0x112, 0xf, 0xf, false));
  v = max(v, (unsigned)__builtin_amdgcn_update_dpp(0, (int)v, 0x114, 0xf, 0xf, false));
  v = max(v, (unsigned)__builtin_amdgcn_update_dpp(0, (int)v, 0x118, 0xf, 0xf, false));
  const unsigned a = __builtin_amdgcn_readlane(v, 15), b = __builtin_amdgcn_readlane(v, 31);
  const unsigned c = __builtin_amdgcn_readlane(v, 47), d = __builtin_amdgcn_readlane(v, 63);
  return max(max(a, b), max(c, d));
}
DI bool give_up(unsigned* tmo, unsigned call, u64 t0, unsigned& spins) {
  if ((++spins & 7u) != 0u) return false;
  if (ld32<AGT>(tmo) == call || __builtin_amdgcn_s_memrealtime() - t0 > DEADLINE_TICKS) {
    st32<AGT>(tmo, call);
    return true;
  }
  return false;
}
DI unsigned pkbf(float a, float b) { unsigned r; asm("v_cvt_pk_bf16_f32 %0, %1, %2" : "=v"(r) : "v"(a), "v"(b)); return r; }
DI unsigned q4(float a, float b, float c, float d, float inv) {
  const int t = __builtin_amdgcn_cvt_pk_fp8_f32(a * inv, b * inv, 0, false);
  return (unsigned)__builtin_amdgcn_cvt_pk_fp8_f32(c * inv, d * inv, t, true);
}
DI void store16(u8* p, const u32x4 v) { asm volatile("global_store_dwordx4 %0, %1, off sc0 sc1\n\ts_nop 1" :: "v"(p), "v"(v) : "memory"); }
DI void store_row16(u8* p, unsigned w0, unsigned w1, unsigned w2, unsigned w3, bool odd) {
  store16(p, (u32x4){odd ? w1 : w0, odd ? w0 : w1, odd ? w3 : w2, odd ? w2 : w3});
}

template <bool TALL>
DI void loader(const Ctx& C, const Tile T, LAS char* lds, const int p, const int lane) {
  const bool remote = TALL && T.grp != 0;
  const void* base = remote ? (const void*)(C.ws + WS_XS + (size_t)T.src * XS_STRIDE) : (const void*)(C.x + (size_t)(TALL ? BM : 0) * K);
  __amdgpu_buffer_rsrc_t rA = __builtin_amdgcn_make_buffer_rsrc((void*)base, 0, BM * K * 2, 0x00020000);
  __amdgpu_buffer_rsrc_t rE = __builtin_amdgcn_make_buffer_rsrc((void*)(C.x + (size_t)T.grp * 32 * K), 0, 32 * K * 2, 0x00020000);
  __amdgpu_buffer_rsrc_t rB = __builtin_amdgcn_make_buffer_rsrc((void*)(C.w + (size_t)T.dst * NS + T.ct * BN), 0, (K - 1) * N * 2 + BN * 2, 0x00020000);
  const unsigned pstr = remote ? 1024u : 8u * K * 2, kstr = remote ? (unsigned)XT_BYTES : BK * 2u;
  const unsigned voL = (unsigned)(((lane >> 3) + 8 * p) * (K * 2) + (((lane & 7) ^ ((p << 2) | (lane >> 4))) << 4));
  const unsigned voA = remote ? (unsigned)lane * 16u + p * 1024u : voL;
  const unsigned voB = (unsigned)(((lane >> 4) + 8 * p) * (N * 2) + (((lane & 15) ^ ((lane >> 4) | (p << 3))) << 4));
  auto issueA = [&](int kt, int buf) {
#pragma unroll
    for (int j = 0; j < 16; ++j)
      __builtin_amdgcn_raw_ptr_buffer_load_lds(rA, (LAS void*)(lds + buf * A_BYTES + (p + 2 * j + 2 * (j >> 3)) * 1024), 16, voA, kt * kstr + 2 * j * pstr, 0, TALL ? 17 : 0);
    if (TALL) {
#pragma unroll
      for (int j = 0; j < 2; ++j)
        __builtin_amdgcn_raw_ptr_buffer_load_lds(rE, (LAS void*)(lds + buf * A_BYTES + (16 + p + 18 * j) * 1024), 16, voL, kt * (BK * 2) + j * 16 * K * 2, 0, 0);
    }
  };
  auto issueB = [&](int kt, int buf) {
#pragma unroll
    for (int j = 0; j < 8; ++j) {
      const int pc = (j & 1) + 4 * (j >> 1);
      __builtin_amdgcn_raw_ptr_buffer_load_lds(rB, (LAS void*)(lds + B_BASE + buf * B_BYTES + (pc + 2 * p) * 1024), 16, voB, (kt * BK + pc * 4) * (N * 2), 0, TALL ? 0 : AUXW);
    }
  };
  constexpr int NA = TALL ? 18 : 16;
  if (remote) {
    issueB(0, 0); issueB(1, 1);
    BAR();
    issueA(0, 0);
    BAR();
    issueA(1, 1);
    WAITV(NA);
  } else {
    issueB(0, 0); issueA(0, 0); issueB(1, 1); issueA(1, 1);
    WAITV(NA + 8);
  }
  BAR();
  int buf = 2;
  for (int kt = 0; kt < NT - 2; ++kt) {
    issueB(kt + 2, buf); issueA(kt + 2, buf);
    WAITV(NA + 8);
    buf = (buf == 2) ? 0 : buf + 1;
    BAR();
  }
  WAITV(0);
  BAR(); BAR();
}

template <int MB>
DI void consumer(const Ctx& C, const Tile T, LAS char* lds, const int wid, const int lane) {
  const unsigned lbase = (unsigned)(size_t)lds;
  LAS unsigned* xw = (LAS unsigned*)(lds + X_BASE);
  const int wr = wid >> 1, wc = wid & 1;
  const int g = lane >> 4, c = lane & 15, q = (lane & 15) >> 2, pp = lane & 3;
  unsigned aA[2], aB[4];
#pragma unroll
  for (int ks = 0; ks < 2; ++ks) aA[ks] = lbase + (wr * 144 + c) * 128 + (((4 * ks + g) ^ (c >> 1)) << 4);
#pragma unroll
  for (int b = 0; b < 4; ++b) {
    const int row = 8 * g + q, ch = 8 * wc + 2 * pp + (b >> 1);
    aB[b] = lbase + B_BASE + row * 256 + ((ch ^ ((row & 3) | (row & 8))) << 4) + (((b ^ pp) & 1) << 3);
  }
  f32x4 acc[MB][4];
  bf16x8 Af[MB][2];
  s16x4 Bq[2][4][2][2];
  auto rdA = [&](int m, int ks) { Af[m][ks] = ldA(aA[ks], m * 2048); SB(); };
  auto rdB = [&](int set, int i) { Bq[set][i >> 2][(i >> 1) & 1][i & 1] = ldB(aB[i >> 2], ((i >> 1) & 1) * 8192 + (i & 1) * 1024); SB(); };
  auto mma = [&](int set, int m, int i) {
    const int b = (m & 1) ? 3 - (i >> 1) : (i >> 1), ks = ((i >> 1) ^ i) & 1;
    s16x8 bv = __builtin_shufflevector(Bq[set][b][ks][0], Bq[set][b][ks][1], 0, 1, 2, 3, 4, 5, 6, 7);
    acc[m][b] = __builtin_amdgcn_mfma_f32_16x16x32_bf16(__builtin_bit_cast(bf16x8, bv), Af[m][ks], acc[m][b], 0, 0, 0);
    SB();
  };
  int buf = 0;
  auto ktile = [&](int set) {
    int da = 0, db = 0;
#pragma unroll
    for (int m = 0; m < MB - 4; ++m)
#pragma unroll
      for (int i = 0; i < 8; ++i) {
        mma(set, m, i);
        if (m == 0 && i < 6) rdA(2 + (i >> 1), i & 1);
        if (m > 0 && i < 2) rdA(m + 4, i);
        if (m == 0 && i == 6) { db = buf == 2 ? -2 * B_BYTES : B_BYTES; asm volatile("" : "+s"(db)); SB(); }
        if (m == 1 && i >= 3 && i < 7) { aB[i - 3] += db; SB(); }
        if (m == 2 && i == 3) { da = buf == 2 ? -2 * A_BYTES : A_BYTES; asm volatile("" : "+s"(da)); SB(); }
        if (m == 2 && i == 5) { buf = buf == 2 ? -1 : buf; asm volatile("" : "+s"(buf)); SB(); }
        if (m == 2 && i == 6) { ++buf; asm volatile("" : "+s"(buf)); SB(); }
        if (m == MB - 5 && (i == 3 || i == 4)) { aA[i - 3] += da; SB(); }
      }
    __builtin_amdgcn_s_waitcnt(0xc07f);
    SB();
    BAR();
    SB();
#pragma unroll
    for (int m = MB - 4; m < MB; ++m)
#pragma unroll
      for (int i = 0; i < 8; ++i) {
        const int j = (m - (MB - 4)) * 8 + i;
        mma(set, m, i);
        if (j < 16) rdB(set ^ 1, j);
        if (j >= 16 && j < 20) rdA((j - 16) >> 1, j & 1);
      }
  };
#pragma unroll
  for (int m = 0; m < MB; ++m)
#pragma unroll
    for (int b = 0; b < 4; ++b) { acc[m][b] = (f32x4){0.f, 0.f, 0.f, 0.f}; asm volatile("" : "+v"(acc[m][b])); }
  SB();
  if (MB == 9 && T.grp != 0) { BAR(); BAR(); }
  BAR();
  if (wid == 0) STAMPW(1);
#pragma unroll
  for (int i = 0; i < 16; ++i) rdB(0, i);
#pragma unroll
  for (int i = 0; i < 4; ++i) rdA(i >> 1, i & 1);
#pragma unroll 1
  for (int kt = 0; kt < NT; kt += 2) { ktile(0); ktile(1); }
  if (wid == 0) STAMPW(2);
  float am = 0.f;
#pragma unroll
  for (int m = 0; m < MB; ++m)
#pragma unroll
    for (int b = 0; b < 4; ++b) {
      am = __builtin_fmaxf(__builtin_fmaxf(am, __builtin_fabsf(acc[m][b][0])), __builtin_fabsf(acc[m][b][1]));
      am = __builtin_fmaxf(__builtin_fmaxf(am, __builtin_fabsf(acc[m][b][2])), __builtin_fabsf(acc[m][b][3]));
    }
  const unsigned mu = wave_max_u32(__float_as_uint(am));
  const unsigned call = xw[1];
  if (lane == 0)
    st64((u64*)(C.ws + WS_G1) + (blockIdx.x & 7) * 128 + (blockIdx.x >> 3) * 4 + wid, call, mu);
  if (wid == 0) STAMPW(3);
  constexpr int M0 = MB == 8 ? NE : 0;
  if (M0 > 0) {
    const int slot = (C.rank * 16 + T.ct) * 4 + wid;
    u8* ep = C.pws + WS_ES + ((size_t)(call & 1) * (NR * 64) + slot) * 8192 + lane * 16;
#pragma unroll
    for (int m = 0; m < M0; ++m)
#pragma unroll
      for (int b = 0; b < 4; b += 2) {
        store16(ep + m * 2048 + b * 512, (u32x4){pkbf(acc[m][b][0], acc[m][b][1]), pkbf(acc[m][b][2], acc[m][b][3]),
                                                 pkbf(acc[m][b + 1][0], acc[m][b + 1][1]), pkbf(acc[m][b + 1][2], acc[m][b + 1][3])});
        if (__builtin_amdgcn_readfirstlane(__hip_atomic_load(&xw[2], __ATOMIC_RELAXED, __HIP_MEMORY_SCOPE_WORKGROUP)) == 0) { WAITV(0); __builtin_amdgcn_s_sleep(ESLEEP); }
      }
    WAITV(0);
    if (lane == 0) st32<SYS>(C.pws + WS_EF + (size_t)slot * 128, call);
    if (wid == 0) STAMPW(5);
  }
  u8* ob = C.pout + T.ct * BN + wc * 64 + 16 * g;
  u8* op = ob + (size_t)(T.src * M + (MB == 9 ? BM : 0) + wr * 128 + c) * NS;
  u8* oe = ob + (size_t)(C.rank * M + T.grp * 32 + wr * 16 + c) * NS;
  BAR();
  if (wid == 0) STAMPW(4);
  const float inv = 448.0f / __uint_as_float(xw[0]);
#pragma unroll
  for (int m = M0; m < MB; ++m) {
    unsigned w[4];
#pragma unroll
    for (int b = 0; b < 4; ++b) w[b] = q4(acc[m][b][0], acc[m][b][1], acc[m][b][2], acc[m][b][3], inv);
    store_row16(m < 8 ? op + (size_t)m * 16 * NS : oe, w[0], w[1], w[2], w[3], g & 1);
    SB();
  }
  if (MB == 8) {
    WAITV(0);
    if (wid == 0) STAMPW(6);
    BAR();
  }
}

struct Early { u32x4 v[8]; };
DI void early_fetch(const Ctx& C, Early& E, const int f, const int w, const int lane, const unsigned call) {
  const unsigned* fl = (const unsigned*)(C.ws + WS_EF + (size_t)(f * 4 + w) * 128);
  unsigned spins = 0;
  const u64 t0 = __builtin_amdgcn_s_memrealtime();
  while (ld32<SYS>(fl) < call) {
    if (give_up((unsigned*)(C.ws + WS_TMO), call, t0, spins)) break;
    __builtin_amdgcn_s_sleep(2);
  }
  const u8* ep = C.ws + WS_ES + ((size_t)(call & 1) * (NR * 64) + f * 4 + w) * 8192 + lane * 16;
  asm volatile("global_load_dwordx4 %0, %8, off sc0 sc1\n\tglobal_load_dwordx4 %1, %8, off offset:1024 sc0 sc1\n\t"
               "global_load_dwordx4 %2, %8, off offset:2048 sc0 sc1\n\tglobal_load_dwordx4 %3, %8, off offset:3072 sc0 sc1\n\t"
               "global_load_dwordx4 %4, %9, off sc0 sc1\n\tglobal_load_dwordx4 %5, %9, off offset:1024 sc0 sc1\n\t"
               "global_load_dwordx4 %6, %10, off sc0 sc1\n\tglobal_load_dwordx4 %7, %10, off offset:1024 sc0 sc1\n\ts_waitcnt vmcnt(0)"
               : "=&v"(E.v[0]), "=&v"(E.v[1]), "=&v"(E.v[2]), "=&v"(E.v[3]), "=&v"(E.v[4]), "=&v"(E.v[5]), "=&v"(E.v[6]), "=&v"(E.v[7])
               : "v"(ep), "v"(ep + (NE > 2 ? 4096 : 0)), "v"(ep + (NE > 3 ? 6144 : 0)) : "memory");
}
DI void early_store(const Ctx& C, const Early& E, const int f, const int w, const int lane, const float inv) {
  u8* op = C.out + (size_t)((f >> 4) * M + (w >> 1) * 128 + (lane & 15)) * NS + (f & 15) * BN + (w & 1) * 64 + (lane & 48);
#pragma unroll
  for (int m = 0; m < NE; ++m) {
    unsigned q[4];
#pragma unroll
    for (int b = 0; b < 4; ++b) {
      const unsigned lo = E.v[2 * m + (b >> 1)][(b & 1) * 2], hi = E.v[2 * m + (b >> 1)][(b & 1) * 2 + 1];
      q[b] = q4(__uint_as_float(lo << 16), __uint_as_float(lo & 0xffff0000u), __uint_as_float(hi << 16), __uint_as_float(hi & 0xffff0000u), inv);
    }
    store_row16(op + (size_t)m * 16 * NS, q[0], q[1], q[2], q[3], (lane >> 4) & 1);
  }
}

__global__ __launch_bounds__(512, 2) void gemm_a2a_q(const Params P) {
  extern __shared__ __attribute__((aligned(16))) char smem[];
  LAS char* lds = (LAS char*)smem;
  const int tid = threadIdx.x, lane = tid & 63;
  const int wid = __builtin_amdgcn_readfirstlane(tid >> 6);
  const int bid = blockIdx.x;
  const int xcd = bid & 7, idx = bid >> 3;
  const int pj = (idx < 2 ? wid + 1 : idx < 16 ? 1 + (idx - 2) % 7 : 0) & 7;
  Ctx C = {P.x, P.w, P.ws[0], P.out[0], P.ws[pj], P.out[pj], P.rank};
  asm volatile("" : "+s"(C.x), "+s"(C.w), "+s"(C.ws), "+s"(C.out), "+s"(C.pws), "+s"(C.pout), "+s"(C.rank));
  const int rank = C.rank;
  unsigned* const tmo = (unsigned*)(C.ws + WS_TMO);
  unsigned* const cnt = (unsigned*)(C.ws + WS_CNT + (size_t)bid * 128);
  LAS unsigned* xw = (LAS unsigned*)(lds + X_BASE);
  if (wid == 0) STAMPW(0);

  if (idx < 2) {
    const int cw = xcd * 2 + idx;
    if (wid < 7) {
      const char* sp = (const char*)(C.x + (size_t)(BM + 16 * cw + (lane >> 3)) * K);
      const char* sp0 = sp + (((lane & 7) ^ (lane >> 4)) << 4);
      const char* sp1 = sp + 8 * K * 2 + (((lane & 7) ^ (4 | (lane >> 4))) << 4);
      u8* dp = C.pws + WS_XS + (size_t)rank * XS_STRIDE + cw * 2048 + lane * 16;
      unsigned* fp = (unsigned*)(C.pws + WS_XF + (rank * 16 + cw) * 128);
      u32x4 r0[8], r1[8];
#pragma unroll
      for (int s = 0; s < 8; ++s) { r0[s] = (u32x4){0u, 0u, 0u, 0u}; r1[s] = (u32x4){0u, 0u, 0u, 0u}; }
#define CLOAD(s) asm volatile("global_load_dwordx4 %0, %2, off offset:%4\n\tglobal_load_dwordx4 %1, %3, off offset:%4" \
                              : "+v"(r0[s]), "+v"(r1[s]) : "v"(sp0), "v"(sp1), "n"((s) * 128) : "memory")
#define CSTORE(s) do { asm volatile("global_store_dwordx4 %0, %1, off sc0 sc1\n\tglobal_store_dwordx4 %0, %2, off offset:1024 sc0 sc1\n\ts_nop 1" \
                                    :: "v"(dp), "v"(r0[s]), "v"(r1[s]) : "memory"); dp += XT_BYTES; } while (0)
#define CFLAG() do { ++fv; asm volatile("s_mov_b64 exec, 1\n\tglobal_store_dword %0, %1, off sc0 sc1\n\ts_mov_b64 exec, -1" :: "v"(fp), "v"(fv) : "memory"); } while (0)
      CLOAD(0); CLOAD(1); CLOAD(2);
      WAITV(4); CSTORE(0);
      WAITV(4); CSTORE(1);
#pragma unroll
      for (int s = 3; s < 8; ++s) CLOAD(s);
      sp0 += 1024; sp1 += 1024;
      CLOAD(0); CLOAD(1);
      BAR();
      unsigned fv = (xw[1] - 1u) * NT;
      WAITV(16); CFLAG();
      CSTORE(2); CLOAD(2); WAITV(19); CFLAG(); BAR();
      WAITV(10);
#pragma unroll
      for (int t = 3; t < 8; ++t) { CSTORE(t); CLOAD(t); WAITV(7); CFLAG();
        BAR(); }
      for (int k8 = 8; k8 < NT - 8; k8 += 8) {
        sp0 += 1024; sp1 += 1024;
#pragma unroll
        for (int i = 0; i < 8; ++i) { CSTORE(i); CLOAD(i); WAITV(7); CFLAG(); BAR(); }
      }
#pragma unroll
      for (int i = 0; i < 8; ++i) { CSTORE(i); WAITV(3); CFLAG(); BAR(); }
      WAITV(0);
      CFLAG();
      if (wid == 0) { STAMPW(1); STAMPW(2); }
    } else {
      unsigned ticket = 0;
      if (lane == 0) ticket = draw(cnt);
      const unsigned call = (unsigned)__builtin_amdgcn_readfirstlane((int)ticket) + 1u;
      if (lane == 0) xw[1] = call;
      WAITL0();
      BAR();
      for (int k = 2; k < NT; ++k) BAR();
      if (idx == 0) {
        const u8* gp = C.ws + WS_G1 + xcd * 1024 + lane * 16;
        u64* dstg = (u64*)(P.ws[lane & 7] + WS_G2) + (call & 1) * (NR * 8) + rank * 8 + xcd;
        asm volatile("" : "+v"(dstg));
        unsigned spins = 0;
        const u64 t0 = __builtin_amdgcn_s_memrealtime();
        for (;;) {
          u32x4 a;
          asm volatile("global_load_dwordx4 %0, %1, off sc0 sc1\n\ts_waitcnt vmcnt(0)" : "=&v"(a) : "v"(gp) : "memory");
          if (__all(lane < 4 || (a[1] == call && a[3] == call))) {
            const unsigned mx = wave_max_u32(lane < 4 ? 0u : max(a[0], a[2]));
            if (lane < NR) st64(dstg, call, mx);
            break;
          }
          if (give_up(tmo, call, t0, spins)) break;
          __builtin_amdgcn_s_sleep(1);
        }
      }
    }
    return;
  }

  const bool tall = idx >= 16;
  Tile T;
  if (!tall) { T.src = rank; T.dst = (rank + pj) & 7; T.ct = xcd * 2 + (idx - 2) / 7; T.grp = 0; }
  else { const int u = idx - 16; T.grp = (xcd >> 2) * 4 + (u >> 2); T.src = (rank + T.grp) & 7; T.dst = rank; T.ct = (xcd & 3) * 4 + (u & 3); }

  if (wid < 4) {
    if (tall) consumer<9>(C, T, lds, wid, lane); else consumer<8>(C, T, lds, wid, lane);
    return;
  }
  const int f = bid - 128;
  const bool incoming = tall && (f >> 4) != rank;
  const bool remote = T.grp != 0;
  Early E;
  if (wid < 6) {
    if (tall) loader<true>(C, T, lds, wid - 4, lane); else loader<false>(C, T, lds, wid - 4, lane);
  } else if (wid == 6) {
    const u64 tb = __builtin_amdgcn_s_memrealtime();
    unsigned warm = 0;
    if (tall)
      asm volatile("global_load_dword %0, %1, off\n\tglobal_load_dword %0, %2, off\n\tglobal_load_dword %0, %3, off\n\tglobal_load_dword %0, %4, off" : "+v"(warm)
                   : "v"((const char*)C.x + (size_t)f * 16384 + lane * 128), "v"((const char*)C.x + (size_t)f * 16384 + 8192 + lane * 128),
                     "v"((const char*)C.x + (size_t)bid * 16384 + lane * 128), "v"((const char*)C.x + (size_t)bid * 16384 + 8192 + lane * 128) : "memory");
    if (remote) { BAR(); BAR(); }
    BAR();
    const int hold = !tall ? QA : remote ? QB : 0;
    int tl = (int)(__builtin_amdgcn_s_memrealtime() - tb);
    for (int k = 1; k <= NT; ++k) {
      if (tl >= QT1 && tl < QT2) {
        tl += hold;
        while ((int)(__builtin_amdgcn_s_memrealtime() - tb) < tl) __builtin_amdgcn_s_sleep(1);
      }
      BAR();
      tl = max(tl, (int)(__builtin_amdgcn_s_memrealtime() - tb) - 40);
    }
    asm volatile("s_waitcnt vmcnt(0)" :: "v"(warm) : "memory");
  }
  if (wid < 7) {
    if (NE > 0 && incoming) {
      const unsigned call = xw[1];
      early_fetch(C, E, f, wid - 4, lane, call);
      const bool dead = ld32<AGT>(tmo) == call;
      BAR();
      early_store(C, E, f, wid - 4, lane, dead ? __uint_as_float(0x7fc00000u) : 448.0f / __uint_as_float(xw[0]));
    }
    return;
  }
  unsigned ticket = 0, spins = 0;
  if (lane == 0) ticket = draw(cnt);
  const unsigned call = (unsigned)__builtin_amdgcn_readfirstlane((int)ticket) + 1u;
  if (lane == 0) { xw[1] = call; xw[2] = 0u; }
  WAITL0();
  if (remote) {
    const unsigned* fp = (const unsigned*)(C.ws + WS_XF + (T.src * 16 + (lane & 15)) * 128);
    const unsigned base = (call - 1u) * NT;
    unsigned ready = 0;
    for (int k = -3; k < NT; ++k) {
      const unsigned need = (unsigned)min(k + 4, NT);
      if (ready < (k < 0 ? need : min(need + 2u, (unsigned)NT))) {
        const u64 t0 = __builtin_amdgcn_s_memrealtime();
        for (;;) {
          const unsigned d = ld32<SYS>(fp) - base;
          ready = NT - wave_max_u32(d > (unsigned)NT ? (unsigned)NT : NT - d);
          if (ready >= need) break;
          if (give_up(tmo, call, t0, spins)) { ready = NT; break; }
          __builtin_amdgcn_s_sleep(1);
        }
      }
      BAR();
    }
  } else {
    for (int k = -1; k < NT; ++k) BAR();
  }
  if (NE > 0 && incoming) early_fetch(C, E, f, 3, lane, call);
  unsigned mx;
  {
    const u64* gp = (const u64*)(C.ws + WS_G2) + (call & 1) * (NR * 8) + lane;
    const u64 t0 = __builtin_amdgcn_s_memrealtime();
    for (;;) {
      u64 a; unsigned tm;
      asm volatile("global_load_dwordx2 %0, %2, off sc0 sc1\n\tglobal_load_dword %1, %3, off sc1\n\ts_waitcnt vmcnt(0)"
                   : "=&v"(a), "=&v"(tm) : "v"(gp), "v"(tmo) : "memory");
      if (tm == call) { mx = 0x7fc00000u; break; }
      if (__all((unsigned)(a >> 32) == call)) { mx = wave_max_u32((unsigned)a); break; }
      if (__builtin_amdgcn_s_memrealtime() - t0 > DEADLINE_TICKS) st32<AGT>(tmo, call);
      __builtin_amdgcn_s_sleep(2);
    }
    if (lane == 0) { xw[0] = mx; __hip_atomic_store(&xw[2], 1u, __ATOMIC_RELAXED, __HIP_MEMORY_SCOPE_WORKGROUP); }
    WAITL0();
  }
  BAR();
  if (!tall) {
    BAR();
    if (lane == 0) st32<SYS>(C.pws + WS_DONE + (size_t)(rank * 16 + T.ct) * 128, call);
  } else if (incoming) {
    if (NE > 0) early_store(C, E, f, 3, lane, 448.0f / __uint_as_float(mx));
    const unsigned* fl = (const unsigned*)(C.ws + WS_DONE + (size_t)f * 128);
    const u64 t1 = __builtin_amdgcn_s_memrealtime();
    while (ld32<SYS>(fl) < call) {
      if (give_up(tmo, call, t1, spins)) { if (lane == 0) *(GAS u8*)(size_t)(C.out + (size_t)rank * M * NS) = 0x7f; break; }
      __builtin_amdgcn_s_sleep(1);
    }
  }
  STAMPW(7);
}

extern "C" void kernel_launch(int rank, int nranks, ncclComm_t comm, hipStream_t stream,
                              void* const* d_in, const int* in_sizes, int n_in,
                              void* d_out, int out_size, const int* in_dtypes, int out_dtype,
                              void* const* d_out_all, void* const* d_ws_all, size_t ws_bytes) {
  (void)comm; (void)in_sizes; (void)n_in; (void)d_out; (void)out_size; (void)in_dtypes; (void)out_dtype;
  if (nranks != NR || ws_bytes < WS_END || (unsigned)rank >= (unsigned)NR) { fprintf(stderr, "unsupported configuration\n"); return; }
  static bool init = false;
  if (!init) {
    HIP_CHECK(hipFuncSetAttribute((const void*)gemm_a2a_q, hipFuncAttributeMaxDynamicSharedMemorySize, LDS_BYTES));
    init = true;
  }
  Params P;
  P.x = (const u16*)d_in[0]; P.w = (const u16*)d_in[1];
  for (int j = 0; j < NR; ++j) { P.out[j] = (u8*)d_out_all[(rank + j) % NR]; P.ws[j] = (u8*)d_ws_all[(rank + j) % NR]; }
  P.rank = rank; P.pad = 0;
  hipLaunchKernelGGL(gemm_a2a_q, dim3(NWG), dim3(512), LDS_BYTES, stream, P);
  HIP_CHECK(hipGetLastError());
}
```

```cpp
#include <hip/hip_runtime.h>
#include <rccl/rccl.h>
#include <cstdio>

#define HIP_CHECK(expr) do { hipError_t _e = (expr); if (_e != hipSuccess) { \
    fprintf(stderr, "HIP error %s at %s:%d\n", hipGetErrorString(_e), __FILE__, __LINE__); \
    return; } } while (0)

#define LAS __attribute__((address_space(3)))
#define GAS __attribute__((address_space(1)))
typedef short s16x4 __attribute__((ext_vector_type(4)));
typedef short s16x8 __attribute__((ext_vector_type(8)));
typedef __bf16 bf16x8 __attribute__((ext_vector_type(8)));
typedef float f32x4 __attribute__((ext_vector_type(4)));
typedef unsigned u32x4 __attribute__((ext_vector_type(4)));
typedef unsigned short u16;
typedef unsigned long long u64;
typedef unsigned char u8;

constexpr int NR = 8;
constexpr int M = 512, N = 16384, K = 4096;
constexpr int NS = N / NR;
constexpr int BM = 256, BN = 128, BK = 64, NT = K / BK;
constexpr int NWG = 256;
constexpr int XT_BYTES = BM * BK * 2;
constexpr int A_BYTES = 288 * BK * 2, B_BYTES = BK * BN * 2;
constexpr int B_BASE = 3 * A_BYTES;
constexpr int X_BASE = B_BASE + 3 * B_BYTES;
constexpr int LDS_BYTES = X_BASE + 256;
#define AUXW 2
#define QT1 1600
#define QT2 3620
#define QAF 71
#define QAS 108
#define QB 126
#define QL 100
#define DLY 30
#define NEF 8
#define NES 3
#define EWIN 1
#define ESLEEP 8
#define RETILE 1
#define FASTM(ct) (!((ct) & 1))

constexpr size_t WS_CNT = 0;
constexpr size_t WS_TMO = WS_CNT + (size_t)NWG * 128;
constexpr size_t WS_G1 = WS_TMO + 256;
constexpr size_t WS_G2 = WS_G1 + 8 * 128 * 8;
constexpr size_t WS_DONE = WS_G2 + 2 * NR * 8 * 8;
constexpr size_t WS_XF = WS_DONE + (size_t)NR * 16 * 128;
constexpr size_t WS_EF = WS_XF + (size_t)NR * 16 * 128;
constexpr size_t WS_XC = WS_EF + (size_t)NR * 64 * 128;
constexpr size_t WS_XS = 1u << 20;
constexpr size_t XS_STRIDE = (size_t)NT * XT_BYTES;
constexpr size_t WS_ES = WS_XS + (size_t)NR * XS_STRIDE;
constexpr size_t WS_END = WS_ES + (size_t)2 * NR * 64 * 16384;
static_assert(WS_XC + 8 * 128 <= WS_XS && NES <= NEF && NEF <= 8, "layout");
constexpr u64 DEADLINE_TICKS = 40000000ull;

struct Params {
  const u16* x; const u16* w;
  int rank; int pad;
  u8* ws[NR];
  u8* out[NR];
};
struct Ctx { const u16* x; const u16* w; u8* ws; u8* out; u8* pws; u8* pout; int rank; };
struct Tile { int src, dst, ct, grp; };

#define STAMPW(k) do {} while (0)
#define SB() __builtin_amdgcn_sched_barrier(0)
#define BAR() do { asm volatile("" ::: "memory"); __builtin_amdgcn_s_barrier(); asm volatile("" ::: "memory"); } while (0)
#define WAITV(n) asm volatile("s_waitcnt vmcnt(%0)" :: "n"(n) : "memory")
#define WAITL0() asm volatile("s_waitcnt lgkmcnt(0)" ::: "memory")
#define DI __device__ __forceinline__

#define SYS __HIP_MEMORY_SCOPE_SYSTEM
#define AGT __HIP_MEMORY_SCOPE_AGENT
template <int SC> DI unsigned ld32(const void* p) { return __hip_atomic_load((const GAS unsigned*)(size_t)p, __ATOMIC_RELAXED, SC); }
template <int SC> DI void st32(void* p, unsigned v) { __hip_atomic_store((GAS unsigned*)(size_t)p, v, __ATOMIC_RELAXED, SC); }
DI void st64(void* p, unsigned tag, unsigned v) { __hip_atomic_store((GAS u64*)(size_t)p, ((u64)tag << 32) | v, __ATOMIC_RELAXED, SYS); }
DI unsigned draw(void* p) { return __hip_atomic_fetch_add((GAS unsigned*)(size_t)p, 1u, __ATOMIC_RELAXED, AGT); }
DI bf16x8 ldA(unsigned addr, int off) { return *(LAS bf16x8*)(size_t)(addr + off); }
DI s16x4 ldB(unsigned addr, int off) { return __builtin_amdgcn_ds_read_tr16_b64_v4i16((LAS s16x4*)(size_t)(addr + off)); }
DI unsigned wave_max_u32(unsigned v) {
  v = max(v, (unsigned)__builtin_amdgcn_update_dpp(0, (int)v, 0x111, 0xf, 0xf, false));
  v = max(v, (unsigned)__builtin_amdgcn_update_dpp(0, (int)v, 0x112, 0xf, 0xf, false));
  v = max(v, (unsigned)__builtin_amdgcn_update_dpp(0, (int)v, 0x114, 0xf, 0xf, false));
  v = max(v, (unsigned)__builtin_amdgcn_update_dpp(0, (int)v, 0x118, 0xf, 0xf, false));
  const unsigned a = __builtin_amdgcn_readlane(v, 15), b = __builtin_amdgcn_readlane(v, 31);
  const unsigned c = __builtin_amdgcn_readlane(v, 47), d = __builtin_amdgcn_readlane(v, 63);
  return max(max(a, b), max(c, d));
}
DI bool give_up(unsigned* tmo, unsigned call, u64 t0, unsigned& spins) {
  if ((++spins & 7u) != 0u) return false;
  if (ld32<AGT>(tmo) == call || __builtin_amdgcn_s_memrealtime() - t0 > DEADLINE_TICKS) {
    st32<AGT>(tmo, call);
    return true;
  }
  return false;
}
DI unsigned pkbf(float a, float b) { unsigned r; asm("v_cvt_pk_bf16_f32 %0, %1, %2" : "=v"(r) : "v"(a), "v"(b)); return r; }
DI unsigned q4(float a, float b, float c, float d, float inv) {
  const int t = __builtin_amdgcn_cvt_pk_fp8_f32(a * inv, b * inv, 0, false);
  return (unsigned)__builtin_amdgcn_cvt_pk_fp8_f32(c * inv, d * inv, t, true);
}
DI void store16(u8* p, const u32x4 v) { asm volatile("global_store_dwordx4 %0, %1, off sc0 sc1\n\ts_nop 1" :: "v"(p), "v"(v) : "memory"); }
DI void store_row16(u8* p, unsigned w0, unsigned w1, unsigned w2, unsigned w3, bool odd) {
  store16(p, (u32x4){odd ? w1 : w0, odd ? w0 : w1, odd ? w3 : w2, odd ? w2 : w3});
}

template <bool TALL>
DI void loader(const Ctx& C, const Tile T, LAS char* lds, const int p, const int lane) {
  const bool remote = TALL && T.grp != 0;
  const void* base = remote ? (const void*)(C.ws + WS_XS + (size_t)T.src * XS_STRIDE) : (const void*)(C.x + (size_t)(TALL ? BM : 0) * K);
  __amdgpu_buffer_rsrc_t rA = __builtin_amdgcn_make_buffer_rsrc((void*)base, 0, BM * K * 2, 0x00020000);
  __amdgpu_buffer_rsrc_t rE = __builtin_amdgcn_make_buffer_rsrc((void*)(C.x + (size_t)T.grp * 32 * K), 0, 32 * K * 2, 0x00020000);
  __amdgpu_buffer_rsrc_t rB = __builtin_amdgcn_make_buffer_rsrc((void*)(C.w + (size_t)T.dst * NS + T.ct * BN), 0, (K - 1) * N * 2 + BN * 2, 0x00020000);
  const unsigned pstr = remote ? 1024u : 8u * K * 2, kstr = remote ? (unsigned)XT_BYTES : BK * 2u;
  const unsigned voL = (unsigned)(((lane >> 3) + 8 * p) * (K * 2) + (((lane & 7) ^ ((p << 2) | (lane >> 4))) << 4));
  const unsigned voA = remote ? (unsigned)lane * 16u + p * 1024u : voL;
  const unsigned voB = (unsigned)(((lane >> 4) + 8 * p) * (N * 2) + (((lane & 15) ^ ((lane >> 4) | (p << 3))) << 4));
  __amdgpu_buffer_rsrc_t rX = __builtin_amdgcn_make_buffer_rsrc((void*)(C.ws + WS_XS + (size_t)C.rank * XS_STRIDE), 0, BM * K * 2, 0x00020000);
  auto issueA = [&](int kt, int buf) {
    if (RETILE && !TALL && kt > 8 && ((kt & 7) == 3 || kt == NT - 1)) {
#pragma unroll
      for (int j = 0; j < 16; ++j)
        __builtin_amdgcn_raw_ptr_buffer_load_lds(rX, (LAS void*)(lds + buf * A_BYTES + (p + 2 * j + 2 * (j >> 3)) * 1024), 16, (unsigned)lane * 16u + p * 1024u, kt * XT_BYTES + 2 * j * 1024, 0, 17);
      return;
    }
#pragma unroll
    for (int j = 0; j < 16; ++j)
      __builtin_amdgcn_raw_ptr_buffer_load_lds(rA, (LAS void*)(lds + buf * A_BYTES + (p + 2 * j + 2 * (j >> 3)) * 1024), 16, voA, kt * kstr + 2 * j * pstr, 0, TALL ? 17 : 0);
    if (TALL) {
#pragma unroll
      for (int j = 0; j < 2; ++j)
        __builtin_amdgcn_raw_ptr_buffer_load_lds(rE, (LAS void*)(lds + buf * A_BYTES + (16 + p + 18 * j) * 1024), 16, voL, kt * (BK * 2) + j * 16 * K * 2, 0, 0);
    }
  };
  auto issueB = [&](int kt, int buf) {
#pragma unroll
    for (int j = 0; j < 8; ++j) {
      const int pc = (j & 1) + 4 * (j >> 1);
      __builtin_amdgcn_raw_ptr_buffer_load_lds(rB, (LAS void*)(lds + B_BASE + buf * B_BYTES + (pc + 2 * p) * 1024), 16, voB, (kt * BK + pc * 4) * (N * 2), 0, TALL ? 0 : AUXW);
    }
  };
  constexpr int NA = TALL ? 18 : 16;
  if (remote) {
    if (DLY) __builtin_amdgcn_s_sleep(DLY);
    issueB(0, 0); issueB(1, 1);
    BAR();
    issueA(0, 0);
    BAR();
    issueA(1, 1);
    WAITV(NA);
  } else {
    issueB(0, 0); issueA(0, 0); issueB(1, 1); issueA(1, 1);
    WAITV(NA + 8);
  }
  BAR();
  int buf = 2;
  for (int kt = 0; kt < NT - 2; ++kt) {
    issueB(kt + 2, buf); issueA(kt + 2, buf);
    WAITV(NA + 8);
    buf = (buf == 2) ? 0 : buf + 1;
    BAR();
  }
  WAITV(0);
  BAR(); BAR();
}

template <int MB>
DI void consumer(const Ctx& C, const Tile T, LAS char* lds, const int wid, const int lane) {
  const unsigned lbase = (unsigned)(size_t)lds;
  LAS unsigned* xw = (LAS unsigned*)(lds + X_BASE);
  const int wr = wid >> 1, wc = wid & 1;
  const int g = lane >> 4, c = lane & 15, q = (lane & 15) >> 2, pp = lane & 3;
  unsigned aA[2], aB[4];
#pragma unroll
  for (int ks = 0; ks < 2; ++ks) aA[ks] = lbase + (wr * 144 + c) * 128 + (((4 * ks + g) ^ (c >> 1)) << 4);
#pragma unroll
  for (int b = 0; b < 4; ++b) {
    const int row = 8 * g + q, ch = 8 * wc + 2 * pp + (b >> 1);
    aB[b] = lbase + B_BASE + row * 256 + ((ch ^ ((row & 3) | (row & 8))) << 4) + (((b ^ pp) & 1) << 3);
  }
  f32x4 acc[MB][4];
  bf16x8 Af[MB][2];
  s16x4 Bq[2][4][2][2];
  auto rdA = [&](int m, int ks) { Af[m][ks] = ldA(aA[ks], m * 2048); SB(); };
  auto rdB = [&](int set, int i) { Bq[set][i >> 2][(i >> 1) & 1][i & 1] = ldB(aB[i >> 2], ((i >> 1) & 1) * 8192 + (i & 1) * 1024); SB(); };
  auto mma = [&](int set, int m, int i) {
    const int b = (m & 1) ? 3 - (i >> 1) : (i >> 1), ks = ((i >> 1) ^ i) & 1;
    s16x8 bv = __builtin_shufflevector(Bq[set][b][ks][0], Bq[set][b][ks][1], 0, 1, 2, 3, 4, 5, 6, 7);
    acc[m][b] = __builtin_amdgcn_mfma_f32_16x16x32_bf16(__builtin_bit_cast(bf16x8, bv), Af[m][ks], acc[m][b], 0, 0, 0);
    SB();
  };
  int buf = 0;
  auto ktile = [&](int set) {
    int da = 0, db = 0;
#pragma unroll
    for (int m = 0; m < MB - 4; ++m)
#pragma unroll
      for (int i = 0; i < 8; ++i) {
        mma(set, m, i);
        if (m == 0 && i < 6) rdA(2 + (i >> 1), i & 1);
        if (m > 0 && i < 2) rdA(m + 4, i);
        if (m == 0 && i == 6) { db = buf == 2 ? -2 * B_BYTES : B_BYTES; asm volatile("" : "+s"(db)); SB(); }
        if (m == 1 && i >= 3 && i < 7) { aB[i - 3] += db; SB(); }
        if (m == 2 && i == 3) { da = buf == 2 ? -2 * A_BYTES : A_BYTES; asm volatile("" : "+s"(da)); SB(); }
        if (m == 2 && i == 5) { buf = buf == 2 ? -1 : buf; asm volatile("" : "+s"(buf)); SB(); }
        if (m == 2 && i == 6) { ++buf; asm volatile("" : "+s"(buf)); SB(); }
        if (m == MB - 5 && (i == 3 || i == 4)) { aA[i - 3] += da; SB(); }
      }
    __builtin_amdgcn_s_waitcnt(0xc07f);
    SB();
    BAR();
    SB();
#pragma unroll
    for (int m = MB - 4; m < MB; ++m)
#pragma unroll
      for (int i = 0; i < 8; ++i) {
        const int j = (m - (MB - 4)) * 8 + i;
        mma(set, m, i);
        if (j < 16) rdB(set ^ 1, j);
        if (j >= 16 && j < 20) rdA((j - 16) >> 1, j & 1);
      }
  };
#pragma unroll
  for (int m = 0; m < MB; ++m)
#pragma unroll
    for (int b = 0; b < 4; ++b) { acc[m][b] = (f32x4){0.f, 0.f, 0.f, 0.f}; asm volatile("" : "+v"(acc[m][b])); }
  SB();
  if (RETILE && MB == 9 && wid < 2) {
    const int id = (blockIdx.x - 128) * 2 + wid, bi = id >> 5, pc = id & 31, kt = bi < 7 ? 11 + 8 * bi : NT - 1;
    u32x4 v;
    if (DLY && T.grp != 0) __builtin_amdgcn_s_sleep(DLY);
    asm volatile("global_load_dwordx4 %0, %1, off\n\ts_waitcnt vmcnt(0)" : "=&v"(v)
                 : "v"((const char*)(C.x + (size_t)(8 * pc + (lane >> 3)) * K + kt * BK) + (((lane & 7) ^ (((pc & 1) << 2) | (lane >> 4))) << 4)) : "memory");
    store16(C.ws + WS_XS + (size_t)C.rank * XS_STRIDE + kt * XT_BYTES + pc * 1024 + lane * 16, v);
    WAITV(0);
    if (lane == 0) __hip_atomic_fetch_add((GAS unsigned*)(size_t)(C.ws + WS_XC + bi * 128), 1u, __ATOMIC_RELAXED, SYS);
  }
  if (MB == 9 && T.grp != 0) { BAR(); BAR(); }
  BAR();
  if (wid == 0) STAMPW(1);
#pragma unroll
  for (int i = 0; i < 16; ++i) rdB(0, i);
#pragma unroll
  for (int i = 0; i < 4; ++i) rdA(i >> 1, i & 1);
#pragma unroll 1
  for (int kt = 0; kt < NT; kt += 2) { ktile(0); ktile(1); }
  if (wid == 0) STAMPW(2);
  float am = 0.f;
#pragma unroll
  for (int m = 0; m < MB; ++m)
#pragma unroll
    for (int b = 0; b < 4; ++b) {
      am = __builtin_fmaxf(__builtin_fmaxf(am, __builtin_fabsf(acc[m][b][0])), __builtin_fabsf(acc[m][b][1]));
      am = __builtin_fmaxf(__builtin_fmaxf(am, __builtin_fabsf(acc[m][b][2])), __builtin_fabsf(acc[m][b][3]));
    }
  const unsigned mu = wave_max_u32(__float_as_uint(am));
  const unsigned call = xw[1];
  if (lane == 0)
    st64((u64*)(C.ws + WS_G1) + (blockIdx.x & 7) * 128 + (blockIdx.x >> 3) * 4 + wid, call, mu);
  if (wid == 0) STAMPW(3);
  const int ne = MB == 9 ? 0 : FASTM(T.ct) ? NEF : NES;
  if (MB == 8) {
    const int slot = (C.rank * 16 + T.ct) * 4 + wid;
    u8* ep = C.pws + WS_ES + ((size_t)(call & 1) * (NR * 64) + slot) * 16384 + lane * 16;
#pragma unroll
    for (int m = 0; m < NEF; ++m)
      if (m < ne) {
#pragma unroll
        for (int b = 0; b < 4; b += 2) {
          store16(ep + m * 2048 + b * 512, (u32x4){pkbf(acc[m][b][0], acc[m][b][1]), pkbf(acc[m][b][2], acc[m][b][3]),
                                                   pkbf(acc[m][b + 1][0], acc[m][b + 1][1]), pkbf(acc[m][b + 1][2], acc[m][b + 1][3])});
          if (__builtin_amdgcn_readfirstlane(__hip_atomic_load(&xw[2], __ATOMIC_RELAXED, __HIP_MEMORY_SCOPE_WORKGROUP)) == 0) { WAITV(EWIN); __builtin_amdgcn_s_sleep(ESLEEP); }
        }
      }
    WAITV(0);
    if (lane == 0) st32<SYS>(C.pws + WS_EF + (size_t)slot * 128, call);
    if (wid == 0) STAMPW(5);
  }
  u8* ob = C.pout + T.ct * BN + wc * 64 + 16 * g;
  u8* op = ob + (size_t)(T.src * M + (MB == 9 ? BM : 0) + wr * 128 + c) * NS;
  u8* oe = ob + (size_t)(C.rank * M + T.grp * 32 + wr * 16 + c) * NS;
  BAR();
  if (wid == 0) STAMPW(4);
  const float inv = 448.0f / __uint_as_float(xw[0]);
#pragma unroll
  for (int m = MB - 1; m >= (MB == 8 ? NES : 0); --m)
    if (m >= ne) {
      unsigned w[4];
#pragma unroll
      for (int b = 0; b < 4; ++b) w[b] = q4(acc[m][b][0], acc[m][b][1], acc[m][b][2], acc[m][b][3], inv);
      store_row16(m < 8 ? op + (size_t)m * 16 * NS : oe, w[0], w[1], w[2], w[3], g & 1);
      SB();
    }
  if (MB == 8) {
    WAITV(0);
    if (wid == 0) STAMPW(6);
    BAR();
  }
}

struct Early { u32x4 v[16]; };
#define LD8(o, p) asm volatile("global_load_dwordx4 %0, %8, off sc0 sc1\n\tglobal_load_dwordx4 %1, %8, off offset:1024 sc0 sc1\n\t" \
               "global_load_dwordx4 %2, %8, off offset:2048 sc0 sc1\n\tglobal_load_dwordx4 %3, %8, off offset:3072 sc0 sc1\n\t" \
               "global_load_dwordx4 %4, %9, off sc0 sc1\n\tglobal_load_dwordx4 %5, %9, off offset:1024 sc0 sc1\n\t" \
               "global_load_dwordx4 %6, %9, off offset:2048 sc0 sc1\n\tglobal_load_dwordx4 %7, %9, off offset:3072 sc0 sc1\n\ts_waitcnt vmcnt(0)" \
               : "=&v"(E.v[o]), "=&v"(E.v[o + 1]), "=&v"(E.v[o + 2]), "=&v"(E.v[o + 3]), "=&v"(E.v[o + 4]), "=&v"(E.v[o + 5]), "=&v"(E.v[o + 6]), "=&v"(E.v[o + 7]) \
               : "v"(p), "v"((p) + 4096) : "memory")
DI void early_fetch(const Ctx& C, Early& E, const int f, const int w, const int lane, const unsigned call) {
  const unsigned* fl = (const unsigned*)(C.ws + WS_EF + (size_t)(f * 4 + w) * 128);
  unsigned spins = 0;
  const u64 t0 = __builtin_amdgcn_s_memrealtime();
  while (ld32<SYS>(fl) < call) {
    if (give_up((unsigned*)(C.ws + WS_TMO), call, t0, spins)) break;
    __builtin_amdgcn_s_sleep(2);
  }
  const u8* ep = C.ws + WS_ES + ((size_t)(call & 1) * (NR * 64) + f * 4 + w) * 16384 + lane * 16;
  LD8(0, ep);
  if ((FASTM(f & 15) ? NEF : NES) > 4) LD8(8, ep + 8192);
}
DI void early_store(const Ctx& C, const Early& E, const int f, const int w, const int lane, const float inv) {
  u8* op = C.out + (size_t)((f >> 4) * M + (w >> 1) * 128 + (lane & 15)) * NS + (f & 15) * BN + (w & 1) * 64 + (lane & 48);
  const int ne = FASTM(f & 15) ? NEF : NES;
#pragma unroll
  for (int m = 0; m < NEF; ++m)
    if (m < ne) {
      unsigned q[4];
#pragma unroll
      for (int b = 0; b < 4; ++b) {
        const unsigned lo = E.v[2 * m + (b >> 1)][(b & 1) * 2], hi = E.v[2 * m + (b >> 1)][(b & 1) * 2 + 1];
        q[b] = q4(__uint_as_float(lo << 16), __uint_as_float(lo & 0xffff0000u), __uint_as_float(hi << 16), __uint_as_float(hi & 0xffff0000u), inv);
      }
      store_row16(op + (size_t)m * 16 * NS, q[0], q[1], q[2], q[3], (lane >> 4) & 1);
    }
}

__global__ __launch_bounds__(512, 2) void gemm_a2a_q(const Params P) {
  extern __shared__ __attribute__((aligned(16))) char smem[];
  LAS char* lds = (LAS char*)smem;
  const int tid = threadIdx.x, lane = tid & 63;
  const int wid = __builtin_amdgcn_readfirstlane(tid >> 6);
  const int bid = blockIdx.x;
  const int xcd = bid & 7, idx = bid >> 3;
  const int pj = (idx < 2 ? wid + 1 : idx < 16 ? 1 + (idx - 2) % 7 : 0) & 7;
  Ctx C = {P.x, P.w, P.ws[0], P.out[0], P.ws[pj], P.out[pj], P.rank};
  asm volatile("" : "+s"(C.x), "+s"(C.w), "+s"(C.ws), "+s"(C.out), "+s"(C.pws), "+s"(C.pout), "+s"(C.rank));
  const int rank = C.rank;
  unsigned* const tmo = (unsigned*)(C.ws + WS_TMO);
  unsigned* const cnt = (unsigned*)(C.ws + WS_CNT + (size_t)bid * 128);
  LAS unsigned* xw = (LAS unsigned*)(lds + X_BASE);
  if (wid == 0) STAMPW(0);

  if (idx < 2) {
    const int cw = xcd * 2 + idx;
    if (wid < 7) {
      const char* sp = (const char*)(C.x + (size_t)(BM + 16 * cw + (lane >> 3)) * K);
      const char* sp0 = sp + (((lane & 7) ^ (lane >> 4)) << 4);
      const char* sp1 = sp + 8 * K * 2 + (((lane & 7) ^ (4 | (lane >> 4))) << 4);
      u8* dp = C.pws + WS_XS + (size_t)rank * XS_STRIDE + cw * 2048 + lane * 16;
      unsigned* fp = (unsigned*)(C.pws + WS_XF + (rank * 16 + cw) * 128);
      u32x4 r0[8], r1[8];
#pragma unroll
      for (int s = 0; s < 8; ++s) { r0[s] = (u32x4){0u, 0u, 0u, 0u}; r1[s] = (u32x4){0u, 0u, 0u, 0u}; }
#define CLOAD(s) asm volatile("global_load_dwordx4 %0, %2, off offset:%4\n\tglobal_load_dwordx4 %1, %3, off offset:%4" \
                              : "+v"(r0[s]), "+v"(r1[s]) : "v"(sp0), "v"(sp1), "n"((s) * 128) : "memory")
#define CSTORE(s) do { asm volatile("global_store_dwordx4 %0, %1, off sc0 sc1\n\tglobal_store_dwordx4 %0, %2, off offset:1024 sc0 sc1\n\ts_nop 1" \
                                    :: "v"(dp), "v"(r0[s]), "v"(r1[s]) : "memory"); dp += XT_BYTES; } while (0)
#define CFLAG() do { ++fv; asm volatile("s_mov_b64 exec, 1\n\tglobal_store_dword %0, %1, off sc0 sc1\n\ts_mov_b64 exec, -1" :: "v"(fp), "v"(fv) : "memory"); } while (0)
      CLOAD(0); CLOAD(1); CLOAD(2);
      WAITV(4); CSTORE(0);
      WAITV(4); CSTORE(1);
#pragma unroll
      for (int s = 3; s < 8; ++s) CLOAD(s);
      sp0 += 1024; sp1 += 1024;
      CLOAD(0); CLOAD(1);
      BAR();
      unsigned fv = (xw[1] - 1u) * NT;
      WAITV(16); CFLAG();
      CSTORE(2); CLOAD(2); WAITV(19); CFLAG(); BAR();
      WAITV(10);
#pragma unroll
      for (int t = 3; t < 8; ++t) { CSTORE(t); CLOAD(t); WAITV(7); CFLAG();
        BAR(); }
      for (int k8 = 8; k8 < NT - 8; k8 += 8) {
        sp0 += 1024; sp1 += 1024;
#pragma unroll
        for (int i = 0; i < 8; ++i) { CSTORE(i); CLOAD(i); WAITV(7); CFLAG(); BAR(); }
      }
#pragma unroll
      for (int i = 0; i < 8; ++i) { CSTORE(i); WAITV(3); CFLAG(); BAR(); }
      WAITV(0);
      CFLAG();
      if (wid == 0) { STAMPW(1); STAMPW(2); }
    } else {
      unsigned ticket = 0;
      if (lane == 0) ticket = draw(cnt);
      const unsigned call = (unsigned)__builtin_amdgcn_readfirstlane((int)ticket) + 1u;
      if (lane == 0) xw[1] = call;
      WAITL0();
      BAR();
      for (int k = 2; k < NT; ++k) BAR();
      if (idx == 0) {
        const u8* gp = C.ws + WS_G1 + xcd * 1024 + lane * 16;
        u64* dstg = (u64*)(P.ws[lane & 7] + WS_G2) + (call & 1) * (NR * 8) + rank * 8 + xcd;
        asm volatile("" : "+v"(dstg));
        unsigned spins = 0;
        const u64 t0 = __builtin_amdgcn_s_memrealtime();
        for (;;) {
          u32x4 a;
          asm volatile("global_load_dwordx4 %0, %1, off sc0 sc1\n\ts_waitcnt vmcnt(0)" : "=&v"(a) : "v"(gp) : "memory");
          if (__all(lane < 4 || (a[1] == call && a[3] == call))) {
            const unsigned mx = wave_max_u32(lane < 4 ? 0u : max(a[0], a[2]));
            if (lane < NR) st64(dstg, call, mx);
            break;
          }
          if (give_up(tmo, call, t0, spins)) break;
          __builtin_amdgcn_s_sleep(1);
        }
      }
    }
    return;
  }

  const bool tall = idx >= 16;
  Tile T;
  if (!tall) { T.src = rank; T.dst = (rank + pj) & 7; T.ct = xcd * 2 + (idx - 2) / 7; T.grp = 0; }
  else { const int u = idx - 16; T.grp = (xcd >> 2) * 4 + (u >> 2); T.src = (rank + T.grp) & 7; T.dst = rank; T.ct = (xcd & 3) * 4 + (u & 3); }

  if (wid < 4) {
    if (tall) consumer<9>(C, T, lds, wid, lane); else consumer<8>(C, T, lds, wid, lane);
    return;
  }
  const int f = bid - 128;
  const bool incoming = tall && (f >> 4) != rank;
  const bool remote = T.grp != 0;
  Early E;
  if (wid < 6) {
    if (tall) loader<true>(C, T, lds, wid - 4, lane); else loader<false>(C, T, lds, wid - 4, lane);
  } else if (wid == 6) {
    const u64 tb = __builtin_amdgcn_s_memrealtime();
    unsigned warm = 0;
    if (DLY && remote) __builtin_amdgcn_s_sleep(DLY);
    if (tall)
      asm volatile("global_load_dword %0, %1, off\n\tglobal_load_dword %0, %2, off\n\tglobal_load_dword %0, %3, off\n\tglobal_load_dword %0, %4, off" : "+v"(warm)
                   : "v"((const char*)C.x + (size_t)f * 16384 + lane * 128), "v"((const char*)C.x + (size_t)f * 16384 + 8192 + lane * 128),
                     "v"((const char*)C.x + (size_t)bid * 16384 + lane * 128), "v"((const char*)C.x + (size_t)bid * 16384 + 8192 + lane * 128) : "memory");
    if (remote) { BAR(); BAR(); }
    BAR();
    const int hold = tall ? (remote ? QB : QL) : FASTM(T.ct) ? QAF : QAS;
    int tl = (int)(__builtin_amdgcn_s_memrealtime() - tb);
    for (int k = 1; k <= NT; ++k) {
      if (tl >= QT1 && tl < QT2) {
        tl += hold;
        while ((int)(__builtin_amdgcn_s_memrealtime() - tb) < tl) __builtin_amdgcn_s_sleep(1);
      }
      BAR();
      tl = max(tl, (int)(__builtin_amdgcn_s_memrealtime() - tb) - 40);
    }
    asm volatile("s_waitcnt vmcnt(0)" :: "v"(warm) : "memory");
  }
  if (wid < 7) {
    if (incoming) {
      const unsigned call = xw[1];
      early_fetch(C, E, f, wid - 4, lane, call);
      const bool dead = ld32<AGT>(tmo) == call;
      BAR();
      early_store(C, E, f, wid - 4, lane, dead ? __uint_as_float(0x7fc00000u) : 448.0f / __uint_as_float(xw[0]));
    }
    return;
  }
  unsigned ticket = 0, spins = 0;
  if (lane == 0) ticket = draw(cnt);
  const unsigned call = (unsigned)__builtin_amdgcn_readfirstlane((int)ticket) + 1u;
  if (lane == 0) { xw[1] = call; xw[2] = 0u; }
  WAITL0();
  if (remote) {
    const unsigned* fp = (const unsigned*)(C.ws + WS_XF + (T.src * 16 + (lane & 15)) * 128);
    const unsigned base = (call - 1u) * NT;
    unsigned ready = 0;
    for (int k = -3; k < NT; ++k) {
      const unsigned need = (unsigned)min(k + 4, NT);
      if (ready < (k < 0 ? need : min(need + 2u, (unsigned)NT))) {
        const u64 t0 = __builtin_amdgcn_s_memrealtime();
        for (;;) {
          const unsigned d = ld32<SYS>(fp) - base;
          ready = NT - wave_max_u32(d > (unsigned)NT ? (unsigned)NT : NT - d);
          if (ready >= need) break;
          if (give_up(tmo, call, t0, spins)) { ready = NT; break; }
          __builtin_amdgcn_s_sleep(1);
        }
      }
      BAR();
    }
  } else {
    const u8* cp = C.ws + WS_XC + (lane & 7) * 128;
    unsigned cv = 0;
    for (int k = -1; k < 5; ++k) BAR();
    if (RETILE && !tall) asm volatile("global_load_dword %0, %1, off sc0 sc1" : "+v"(cv) : "v"(cp) : "memory");
    BAR(); BAR();
    if (RETILE && !tall) {
      asm volatile("s_waitcnt vmcnt(0)" : "+v"(cv) :: "memory");
      const u64 t0 = __builtin_amdgcn_s_memrealtime();
      while (!__all(cv - call * 32u < 0x80000000u)) {
        if (give_up(tmo, call, t0, spins)) break;
        __builtin_amdgcn_s_sleep(1);
        cv = ld32<SYS>(cp);
      }
    }
    for (int k = 7; k < NT; ++k) BAR();
  }
  if (incoming) early_fetch(C, E, f, 3, lane, call);
  unsigned mx;
  {
    const u64* gp = (const u64*)(C.ws + WS_G2) + (call & 1) * (NR * 8) + lane;
    const u64 t0 = __builtin_amdgcn_s_memrealtime();
    for (;;) {
      u64 a; unsigned tm;
      asm volatile("global_load_dwordx2 %0, %2, off sc0 sc1\n\tglobal_load_dword %1, %3, off sc1\n\ts_waitcnt vmcnt(0)"
                   : "=&v"(a), "=&v"(tm) : "v"(gp), "v"(tmo) : "memory");
      if (tm == call) { mx = 0x7fc00000u; break; }
      if (__all((unsigned)(a >> 32) == call)) { mx = wave_max_u32((unsigned)a); break; }
      if (__builtin_amdgcn_s_memrealtime() - t0 > DEADLINE_TICKS) st32<AGT>(tmo, call);
      __builtin_amdgcn_s_sleep(2);
    }
    if (lane == 0) { xw[0] = mx; __hip_atomic_store(&xw[2], 1u, __ATOMIC_RELAXED, __HIP_MEMORY_SCOPE_WORKGROUP); }
    WAITL0();
  }
  BAR();
  if (!tall) {
    BAR();
    if (lane == 0) st32<SYS>(C.pws + WS_DONE + (size_t)(rank * 16 + T.ct) * 128, call);
  } else if (incoming) {
    early_store(C, E, f, 3, lane, 448.0f / __uint_as_float(mx));
    const unsigned* fl = (const unsigned*)(C.ws + WS_DONE + (size_t)f * 128);
    const u64 t1 = __builtin_amdgcn_s_memrealtime();
    while (ld32<SYS>(fl) < call) {
      if (give_up(tmo, call, t1, spins)) { if (lane == 0) *(GAS u8*)(size_t)(C.out + (size_t)rank * M * NS) = 0x7f; break; }
      __builtin_amdgcn_s_sleep(1);
    }
  }
  STAMPW(7);
}

extern "C" void kernel_launch(int rank, int nranks, ncclComm_t comm, hipStream_t stream,
                              void* const* d_in, const int* in_sizes, int n_in,
                              void* d_out, int out_size, const int* in_dtypes, int out_dtype,
                              void* const* d_out_all, void* const* d_ws_all, size_t ws_bytes) {
  (void)comm; (void)in_sizes; (void)n_in; (void)d_out; (void)out_size; (void)in_dtypes; (void)out_dtype;
  if (nranks != NR || ws_bytes < WS_END || (unsigned)rank >= (unsigned)NR) { fprintf(stderr, "unsupported configuration\n"); return; }
  static bool init = false;
  if (!init) {
    HIP_CHECK(hipFuncSetAttribute((const void*)gemm_a2a_q, hipFuncAttributeMaxDynamicSharedMemorySize, LDS_BYTES));
    init = true;
  }
  Params P;
  P.x = (const u16*)d_in[0]; P.w = (const u16*)d_in[1];
  for (int j = 0; j < NR; ++j) { P.out[j] = (u8*)d_out_all[(rank + j) % NR]; P.ws[j] = (u8*)d_ws_all[(rank + j) % NR]; }
  P.rank = rank; P.pad = 0;
  hipLaunchKernelGGL(gemm_a2a_q, dim3(NWG), dim3(512), LDS_BYTES, stream, P);
  HIP_CHECK(hipGetLastError());
}
```

```cpp
#include <hip/hip_runtime.h>
#include <rccl/rccl.h>
#include <cstdio>

#define HIP_CHECK(expr) do { hipError_t _e = (expr); if (_e != hipSuccess) { \
    fprintf(stderr, "HIP error %s at %s:%d\n", hipGetErrorString(_e), __FILE__, __LINE__); \
    return; } } while (0)

#define LAS __attribute__((address_space(3)))
#define GAS __attribute__((address_space(1)))
typedef short s16x4 __attribute__((ext_vector_type(4)));
typedef short s16x8 __attribute__((ext_vector_type(8)));
typedef __bf16 bf16x8 __attribute__((ext_vector_type(8)));
typedef float f32x4 __attribute__((ext_vector_type(4)));
typedef unsigned u32x4 __attribute__((ext_vector_type(4)));
typedef unsigned short u16;
typedef unsigned long long u64;
typedef unsigned char u8;

constexpr int NR = 8;
constexpr int M = 512, N = 16384, K = 4096;
constexpr int NS = N / NR;
constexpr int BM = 256, BN = 128, BK = 64, NT = K / BK;
constexpr int NWG = 256;
constexpr int XT_BYTES = BM * BK * 2;
constexpr int A_BYTES = 288 * BK * 2, B_BYTES = BK * BN * 2;
constexpr int B_BASE = 3 * A_BYTES;
constexpr int X_BASE = B_BASE + 3 * B_BYTES;
constexpr int LDS_BYTES = X_BASE + 256;
#define AUXW 2
#define QT1 1600
#define QT2 3620
#define QA0 71
#define QA1 90
#define QA3 122
#define QB 126
#define ECUT 6050
#define DLY 10
#define GS 2
#define EWIN 1
#define ESLEEP 8
#define RETILE 1

constexpr size_t WS_CNT = 0;
constexpr size_t WS_TMO = WS_CNT + (size_t)NWG * 128;
constexpr size_t WS_G1 = WS_TMO + 256;
constexpr size_t WS_G2 = WS_G1 + 8 * 128 * 8;
constexpr size_t WS_DONE = WS_G2 + 2 * NR * 8 * 8;
constexpr size_t WS_XF = WS_DONE + (size_t)NR * 16 * 128;
constexpr size_t WS_EF = WS_XF + (size_t)NR * 16 * 128;
constexpr size_t WS_XC = WS_EF + (size_t)2 * NR * 64 * 128;
constexpr size_t WS_XS = 1u << 20;
constexpr size_t XS_STRIDE = (size_t)NT * XT_BYTES;
constexpr size_t WS_ES = WS_XS + (size_t)NR * XS_STRIDE;
constexpr size_t WS_END = WS_ES + (size_t)2 * NR * 64 * 16384;
static_assert(WS_XC + 8 * 128 <= WS_XS, "layout");
constexpr u64 DEADLINE_TICKS = 40000000ull;

struct Params {
  const u16* x; const u16* w;
  int rank; int pad;
  u8* ws[NR];
  u8* out[NR];
};
struct Ctx { const u16* x; const u16* w; u8* ws; u8* out; u8* pws; u8* pout; int rank; };
struct Tile { int src, dst, ct, grp; };

#define STAMPW(k) do {} while (0)
#define SB() __builtin_amdgcn_sched_barrier(0)
#define BAR() do { asm volatile("" ::: "memory"); __builtin_amdgcn_s_barrier(); asm volatile("" ::: "memory"); } while (0)
#define WAITV(n) asm volatile("s_waitcnt vmcnt(%0)" :: "n"(n) : "memory")
#define WAITL0() asm volatile("s_waitcnt lgkmcnt(0)" ::: "memory")
#define DI __device__ __forceinline__

#define SYS __HIP_MEMORY_SCOPE_SYSTEM
#define AGT __HIP_MEMORY_SCOPE_AGENT
template <int SC> DI unsigned ld32(const void* p) { return __hip_atomic_load((const GAS unsigned*)(size_t)p, __ATOMIC_RELAXED, SC); }
template <int SC> DI void st32(void* p, unsigned v) { __hip_atomic_store((GAS unsigned*)(size_t)p, v, __ATOMIC_RELAXED, SC); }
DI void st64(void* p, unsigned tag, unsigned v) { __hip_atomic_store((GAS u64*)(size_t)p, ((u64)tag << 32) | v, __ATOMIC_RELAXED, SYS); }
DI unsigned draw(void* p) { return __hip_atomic_fetch_add((GAS unsigned*)(size_t)p, 1u, __ATOMIC_RELAXED, AGT); }
DI bf16x8 ldA(unsigned addr, int off) { return *(LAS bf16x8*)(size_t)(addr + off); }
DI s16x4 ldB(unsigned addr, int off) { return __builtin_amdgcn_ds_read_tr16_b64_v4i16((LAS s16x4*)(size_t)(addr + off)); }
DI unsigned wave_max_u32(unsigned v) {
  v = max(v, (unsigned)__builtin_amdgcn_update_dpp(0, (int)v, 0x111, 0xf, 0xf, false));
  v = max(v, (unsigned)__builtin_amdgcn_update_dpp(0, (int)v, 0x112, 0xf, 0xf, false));
  v = max(v, (unsigned)__builtin_amdgcn_update_dpp(0, (int)v, 0x114, 0xf, 0xf, false));
  v = max(v, (unsigned)__builtin_amdgcn_update_dpp(0, (int)v, 0x118, 0xf, 0xf, false));
  const unsigned a = __builtin_amdgcn_readlane(v, 15), b = __builtin_amdgcn_readlane(v, 31);
  const unsigned c = __builtin_amdgcn_readlane(v, 47), d = __builtin_amdgcn_readlane(v, 63);
  return max(max(a, b), max(c, d));
}
DI bool give_up(unsigned* tmo, unsigned call, u64 t0, unsigned& spins) {
  if ((++spins & 7u) != 0u) return false;
  if (ld32<AGT>(tmo) == call || __builtin_amdgcn_s_memrealtime() - t0 > DEADLINE_TICKS) {
    st32<AGT>(tmo, call);
    return true;
  }
  return false;
}
DI unsigned pkbf(float a, float b) { unsigned r; asm("v_cvt_pk_bf16_f32 %0, %1, %2" : "=v"(r) : "v"(a), "v"(b)); return r; }
DI unsigned q4(float a, float b, float c, float d, float inv) {
  const int t = __builtin_amdgcn_cvt_pk_fp8_f32(a * inv, b * inv, 0, false);
  return (unsigned)__builtin_amdgcn_cvt_pk_fp8_f32(c * inv, d * inv, t, true);
}
DI void store16(u8* p, const u32x4 v) { asm volatile("global_store_dwordx4 %0, %1, off sc0 sc1\n\ts_nop 1" :: "v"(p), "v"(v) : "memory"); }
DI void store_row16(u8* p, unsigned w0, unsigned w1, unsigned w2, unsigned w3, bool odd) {
  store16(p, (u32x4){odd ? w1 : w0, odd ? w0 : w1, odd ? w3 : w2, odd ? w2 : w3});
}

DI unsigned q4b(unsigned lo, unsigned hi, float inv) {
  return q4(__uint_as_float(lo << 16), __uint_as_float(lo & 0xffff0000u), __uint_as_float(hi << 16), __uint_as_float(hi & 0xffff0000u), inv);
}
DI void store_bf16_row(u8* p, const u32x4 a, const u32x4 b, float inv, bool odd) {
  store_row16(p, q4b(a[0], a[1], inv), q4b(a[2], a[3], inv), q4b(b[0], b[1], inv), q4b(b[2], b[3], inv), odd);
}

template <bool TALL>
DI void loader(const Ctx& C, const Tile T, LAS char* lds, const int p, const int lane) {
  const bool remote = TALL && T.grp != 0;
  const void* base = remote ? (const void*)(C.ws + WS_XS + (size_t)T.src * XS_STRIDE) : (const void*)(C.x + (size_t)(TALL ? BM : 0) * K);
  __amdgpu_buffer_rsrc_t rA = __builtin_amdgcn_make_buffer_rsrc((void*)base, 0, BM * K * 2, 0x00020000);
  __amdgpu_buffer_rsrc_t rE = __builtin_amdgcn_make_buffer_rsrc((void*)(C.x + (size_t)T.grp * 32 * K), 0, 32 * K * 2, 0x00020000);
  __amdgpu_buffer_rsrc_t rB = __builtin_amdgcn_make_buffer_rsrc((void*)(C.w + (size_t)T.dst * NS + T.ct * BN), 0, (K - 1) * N * 2 + BN * 2, 0x00020000);
  const unsigned pstr = remote ? 1024u : 8u * K * 2, kstr = remote ? (unsigned)XT_BYTES : BK * 2u;
  const unsigned voL = (unsigned)(((lane >> 3) + 8 * p) * (K * 2) + (((lane & 7) ^ ((p << 2) | (lane >> 4))) << 4));
  const unsigned voA = remote ? (unsigned)lane * 16u + p * 1024u : voL;
  const unsigned voB = (unsigned)(((lane >> 4) + 8 * p) * (N * 2) + (((lane & 15) ^ ((lane >> 4) | (p << 3))) << 4));
  __amdgpu_buffer_rsrc_t rX = __builtin_amdgcn_make_buffer_rsrc((void*)(C.ws + WS_XS + (size_t)C.rank * XS_STRIDE), 0, BM * K * 2, 0x00020000);
  auto issueA = [&](int kt, int buf) {
    if (RETILE && !TALL && kt > 8 && ((kt & 7) == 3 || kt == NT - 1)) {
#pragma unroll
      for (int j = 0; j < 16; ++j)
        __builtin_amdgcn_raw_ptr_buffer_load_lds(rX, (LAS void*)(lds + buf * A_BYTES + (p + 2 * j + 2 * (j >> 3)) * 1024), 16, (unsigned)lane * 16u + p * 1024u, kt * XT_BYTES + 2 * j * 1024, 0, 17);
      return;
    }
#pragma unroll
    for (int j = 0; j < 16; ++j)
      __builtin_amdgcn_raw_ptr_buffer_load_lds(rA, (LAS void*)(lds + buf * A_BYTES + (p + 2 * j + 2 * (j >> 3)) * 1024), 16, voA, kt * kstr + 2 * j * pstr, 0, TALL ? 17 : 0);
    if (TALL) {
#pragma unroll
      for (int j = 0; j < 2; ++j)
        __builtin_amdgcn_raw_ptr_buffer_load_lds(rE, (LAS void*)(lds + buf * A_BYTES + (16 + p + 18 * j) * 1024), 16, voL, kt * (BK * 2) + j * 16 * K * 2, 0, 0);
    }
  };
  auto issueB = [&](int kt, int buf) {
#pragma unroll
    for (int j = 0; j < 8; ++j) {
      const int pc = (j & 1) + 4 * (j >> 1);
      __builtin_amdgcn_raw_ptr_buffer_load_lds(rB, (LAS void*)(lds + B_BASE + buf * B_BYTES + (pc + 2 * p) * 1024), 16, voB, (kt * BK + pc * 4) * (N * 2), 0, TALL ? 0 : AUXW);
    }
  };
  constexpr int NA = TALL ? 18 : 16;
  if (remote) {
    if (DLY) __builtin_amdgcn_s_sleep(DLY);
    issueB(0, 0); issueB(1, 1);
    BAR();
    issueA(0, 0);
    BAR();
    issueA(1, 1);
    WAITV(NA);
  } else {
    issueB(0, 0); issueA(0, 0); issueB(1, 1); issueA(1, 1);
    WAITV(NA + 8);
  }
  BAR();
  int buf = 2;
  for (int kt = 0; kt < NT - 2; ++kt) {
    issueB(kt + 2, buf); issueA(kt + 2, buf);
    WAITV(NA + 8);
    buf = (buf == 2) ? 0 : buf + 1;
    BAR();
  }
  WAITV(0);
  BAR(); BAR();
}

template <int MB>
DI void consumer(const Ctx& C, const Tile T, LAS char* lds, const int wid, const int lane) {
  const u64 tb = __builtin_amdgcn_s_memrealtime();
  const unsigned lbase = (unsigned)(size_t)lds;
  LAS unsigned* xw = (LAS unsigned*)(lds + X_BASE);
  const int wr = wid >> 1, wc = wid & 1;
  const int g = lane >> 4, c = lane & 15, q = (lane & 15) >> 2, pp = lane & 3;
  unsigned aA[2], aB[4];
#pragma unroll
  for (int ks = 0; ks < 2; ++ks) aA[ks] = lbase + (wr * 144 + c) * 128 + (((4 * ks + g) ^ (c >> 1)) << 4);
#pragma unroll
  for (int b = 0; b < 4; ++b) {
    const int row = 8 * g + q, ch = 8 * wc + 2 * pp + (b >> 1);
    aB[b] = lbase + B_BASE + row * 256 + ((ch ^ ((row & 3) | (row & 8))) << 4) + (((b ^ pp) & 1) << 3);
  }
  f32x4 acc[MB][4];
  bf16x8 Af[MB][2];
  s16x4 Bq[2][4][2][2];
  auto rdA = [&](int m, int ks) { Af[m][ks] = ldA(aA[ks], m * 2048); SB(); };
  auto rdB = [&](int set, int i) { Bq[set][i >> 2][(i >> 1) & 1][i & 1] = ldB(aB[i >> 2], ((i >> 1) & 1) * 8192 + (i & 1) * 1024); SB(); };
  auto mma = [&](int set, int m, int i) {
    const int b = (m & 1) ? 3 - (i >> 1) : (i >> 1), ks = ((i >> 1) ^ i) & 1;
    s16x8 bv = __builtin_shufflevector(Bq[set][b][ks][0], Bq[set][b][ks][1], 0, 1, 2, 3, 4, 5, 6, 7);
    acc[m][b] = __builtin_amdgcn_mfma_f32_16x16x32_bf16(__builtin_bit_cast(bf16x8, bv), Af[m][ks], acc[m][b], 0, 0, 0);
    SB();
  };
  int buf = 0;
  auto ktile = [&](int set) {
    int da = 0, db = 0;
#pragma unroll
    for (int m = 0; m < MB - 4; ++m)
#pragma unroll
      for (int i = 0; i < 8; ++i) {
        mma(set, m, i);
        if (m == 0 && i < 6) rdA(2 + (i >> 1), i & 1);
        if (m > 0 && i < 2) rdA(m + 4, i);
        if (m == 0 && i == 6) { db = buf == 2 ? -2 * B_BYTES : B_BYTES; asm volatile("" : "+s"(db)); SB(); }
        if (m == 1 && i >= 3 && i < 7) { aB[i - 3] += db; SB(); }
        if (m == 2 && i == 3) { da = buf == 2 ? -2 * A_BYTES : A_BYTES; asm volatile("" : "+s"(da)); SB(); }
        if (m == 2 && i == 5) { buf = buf == 2 ? -1 : buf; asm volatile("" : "+s"(buf)); SB(); }
        if (m == 2 && i == 6) { ++buf; asm volatile("" : "+s"(buf)); SB(); }
        if (m == MB - 5 && (i == 3 || i == 4)) { aA[i - 3] += da; SB(); }
      }
    __builtin_amdgcn_s_waitcnt(0xc07f);
    SB();
    BAR();
    SB();
#pragma unroll
    for (int m = MB - 4; m < MB; ++m)
#pragma unroll
      for (int i = 0; i < 8; ++i) {
        const int j = (m - (MB - 4)) * 8 + i;
        mma(set, m, i);
        if (j < 16) rdB(set ^ 1, j);
        if (j >= 16 && j < 20) rdA((j - 16) >> 1, j & 1);
      }
  };
#pragma unroll
  for (int m = 0; m < MB; ++m)
#pragma unroll
    for (int b = 0; b < 4; ++b) { acc[m][b] = (f32x4){0.f, 0.f, 0.f, 0.f}; asm volatile("" : "+v"(acc[m][b])); }
  SB();
  const int id = ((T.grp - 1) * 16 + T.ct) * 3 + wid;
  if (RETILE && MB == 9 && T.grp != 0 && wid < 3 && id < 256) {
    const int bi = id >> 5, pc = id & 31, kt = bi < 7 ? 11 + 8 * bi : NT - 1;
    u32x4 v;
    if (DLY) __builtin_amdgcn_s_sleep(DLY);
    asm volatile("global_load_dwordx4 %0, %1, off\n\ts_waitcnt vmcnt(0)" : "=&v"(v)
                 : "v"((const char*)(C.x + (size_t)(8 * pc + (lane >> 3)) * K + kt * BK) + (((lane & 7) ^ (((pc & 1) << 2) | (lane >> 4))) << 4)) : "memory");
    store16(C.ws + WS_XS + (size_t)C.rank * XS_STRIDE + kt * XT_BYTES + pc * 1024 + lane * 16, v);
    WAITV(0);
    if (lane == 0) __hip_atomic_fetch_add((GAS unsigned*)(size_t)(C.ws + WS_XC + bi * 128), 1u, __ATOMIC_RELAXED, SYS);
  }
  if (MB == 9 && T.grp != 0) { BAR(); BAR(); }
  BAR();
  if (wid == 0) STAMPW(1);
#pragma unroll
  for (int i = 0; i < 16; ++i) rdB(0, i);
#pragma unroll
  for (int i = 0; i < 4; ++i) rdA(i >> 1, i & 1);
#pragma unroll 1
  for (int kt = 0; kt < NT; kt += 2) { ktile(0); ktile(1); }
  if (wid == 0) STAMPW(2);
  float am = 0.f;
#pragma unroll
  for (int m = 0; m < MB; ++m)
#pragma unroll
    for (int b = 0; b < 4; ++b) {
      am = __builtin_fmaxf(__builtin_fmaxf(am, __builtin_fabsf(acc[m][b][0])), __builtin_fabsf(acc[m][b][1]));
      am = __builtin_fmaxf(__builtin_fmaxf(am, __builtin_fabsf(acc[m][b][2])), __builtin_fabsf(acc[m][b][3]));
    }
  const unsigned mu = wave_max_u32(__float_as_uint(am));
  const unsigned call = xw[1];
  if (lane == 0)
    st64((u64*)(C.ws + WS_G1) + (blockIdx.x & 7) * 128 + (blockIdx.x >> 3) * 4 + wid, call, mu);
  if (wid == 0) STAMPW(3);
  u8* ob = C.pout + T.ct * BN + wc * 64 + 16 * g;
  u8* op = ob + (size_t)(T.src * M + (MB == 9 ? BM : 0) + wr * 128 + c) * NS;
  if (MB == 8) {
    u32x4 pk[8][2];
#pragma unroll
    for (int m = 0; m < 8; ++m)
#pragma unroll
      for (int h = 0; h < 2; ++h)
        pk[m][h] = (u32x4){pkbf(acc[m][2 * h][0], acc[m][2 * h][1]), pkbf(acc[m][2 * h][2], acc[m][2 * h][3]),
                           pkbf(acc[m][2 * h + 1][0], acc[m][2 * h + 1][1]), pkbf(acc[m][2 * h + 1][2], acc[m][2 * h + 1][3])};
    const int slot = (call & 1) * (NR * 64) + (C.rank * 16 + T.ct) * 4 + wid;
    u8* ep = C.pws + WS_ES + (size_t)slot * 16384 + lane * 16;
    int ne = 0;
#pragma unroll
    for (int m = 0; m < 8; ++m)
      if (ne == m && (int)(__builtin_amdgcn_s_memrealtime() - tb) < ECUT &&
          __builtin_amdgcn_readfirstlane(__hip_atomic_load(&xw[2], __ATOMIC_RELAXED, __HIP_MEMORY_SCOPE_WORKGROUP)) == 0) {
#pragma unroll
        for (int h = 0; h < 2; ++h) {
          store16(ep + m * 2048 + h * 1024, pk[m][h]);
          WAITV(EWIN);
          __builtin_amdgcn_s_sleep(ESLEEP);
        }
        ne = m + 1;
      }
    WAITV(0);
    if (lane == 0) st32<SYS>(C.pws + WS_EF + (size_t)slot * 128, call * 16u + ne);
    if (wid == 0) STAMPW(5);
    BAR();
    if (wid == 0) STAMPW(4);
    const float inv = 448.0f / __uint_as_float(xw[0]);
#pragma unroll
    for (int m = 7; m >= 0; --m)
      if (m >= ne) { store_bf16_row(op + (size_t)m * 16 * NS, pk[m][0], pk[m][1], inv, g & 1); SB(); }
  } else {
    u8* oe = ob + (size_t)(C.rank * M + T.grp * 32 + wr * 16 + c) * NS;
    BAR();
    if (wid == 0) STAMPW(4);
    const float inv = 448.0f / __uint_as_float(xw[0]);
#pragma unroll
    for (int m = 0; m < MB; ++m) {
      unsigned w[4];
#pragma unroll
      for (int b = 0; b < 4; ++b) w[b] = q4(acc[m][b][0], acc[m][b][1], acc[m][b][2], acc[m][b][3], inv);
      store_row16(m < 8 ? op + (size_t)m * 16 * NS : oe, w[0], w[1], w[2], w[3], g & 1);
      SB();
    }
  }
  if (MB == 8) {
    WAITV(0);
    if (wid == 0) STAMPW(6);
    BAR();
  }
}

struct Early { u32x4 v[16]; int ne; };
#define LD8(o, p) asm volatile("global_load_dwordx4 %0, %8, off sc0 sc1\n\tglobal_load_dwordx4 %1, %8, off offset:1024 sc0 sc1\n\t" \
               "global_load_dwordx4 %2, %8, off offset:2048 sc0 sc1\n\tglobal_load_dwordx4 %3, %8, off offset:3072 sc0 sc1\n\t" \
               "global_load_dwordx4 %4, %9, off sc0 sc1\n\tglobal_load_dwordx4 %5, %9, off offset:1024 sc0 sc1\n\t" \
               "global_load_dwordx4 %6, %9, off offset:2048 sc0 sc1\n\tglobal_load_dwordx4 %7, %9, off offset:3072 sc0 sc1\n\ts_waitcnt vmcnt(0)" \
               : "=&v"(E.v[o]), "=&v"(E.v[o + 1]), "=&v"(E.v[o + 2]), "=&v"(E.v[o + 3]), "=&v"(E.v[o + 4]), "=&v"(E.v[o + 5]), "=&v"(E.v[o + 6]), "=&v"(E.v[o + 7]) \
               : "v"(p), "v"((p) + 4096) : "memory")
DI void early_fetch(const Ctx& C, Early& E, const int f, const int w, const int lane, const unsigned call) {
  const int slot = (call & 1) * (NR * 64) + f * 4 + w;
  const unsigned* fl = (const unsigned*)(C.ws + WS_EF + (size_t)slot * 128);
  unsigned spins = 0;
  const u64 t0 = __builtin_amdgcn_s_memrealtime();
  unsigned v;
  while ((v = (unsigned)__builtin_amdgcn_readfirstlane((int)ld32<SYS>(fl))) < call * 16u) {
    if (give_up((unsigned*)(C.ws + WS_TMO), call, t0, spins)) { v = call * 16u + 8u; break; }
    __builtin_amdgcn_s_sleep(2);
  }
  E.ne = (int)(v - call * 16u);
  const u8* ep = C.ws + WS_ES + (size_t)slot * 16384 + lane * 16;
  if (E.ne > 0) LD8(0, ep);
  if (E.ne > 4) LD8(8, ep + 8192);
}
DI void early_store(const Ctx& C, const Early& E, const int f, const int w, const int lane, const float inv) {
  u8* op = C.out + (size_t)((f >> 4) * M + (w >> 1) * 128 + (lane & 15)) * NS + (f & 15) * BN + (w & 1) * 64 + (lane & 48);
#pragma unroll
  for (int m = 0; m < 8; ++m)
    if (m < E.ne) store_bf16_row(op + (size_t)m * 16 * NS, E.v[2 * m], E.v[2 * m + 1], inv, (lane >> 4) & 1);
}

__global__ __launch_bounds__(512, 2) void gemm_a2a_q(const Params P) {
  extern __shared__ __attribute__((aligned(16))) char smem[];
  LAS char* lds = (LAS char*)smem;
  const int tid = threadIdx.x, lane = tid & 63;
  const int wid = __builtin_amdgcn_readfirstlane(tid >> 6);
  const int bid = blockIdx.x;
  const int xcd = bid & 7, idx = bid >> 3;
  const int pj = (idx < 2 ? wid + 1 : idx < 16 ? 1 + (idx - 2) % 7 : 0) & 7;
  Ctx C = {P.x, P.w, P.ws[0], P.out[0], P.ws[pj], P.out[pj], P.rank};
  asm volatile("" : "+s"(C.x), "+s"(C.w), "+s"(C.ws), "+s"(C.out), "+s"(C.pws), "+s"(C.pout), "+s"(C.rank));
  const int rank = C.rank;
  unsigned* const tmo = (unsigned*)(C.ws + WS_TMO);
  unsigned* const cnt = (unsigned*)(C.ws + WS_CNT + (size_t)bid * 128);
  LAS unsigned* xw = (LAS unsigned*)(lds + X_BASE);
  if (wid == 0) STAMPW(0);

  if (idx < 2) {
    const int cw = xcd * 2 + idx;
    if (wid < 7) {
      const char* sp = (const char*)(C.x + (size_t)(BM + 16 * cw + (lane >> 3)) * K);
      const char* sp0 = sp + (((lane & 7) ^ (lane >> 4)) << 4);
      const char* sp1 = sp + 8 * K * 2 + (((lane & 7) ^ (4 | (lane >> 4))) << 4);
      u8* dp = C.pws + WS_XS + (size_t)rank * XS_STRIDE + cw * 2048 + lane * 16;
      unsigned* fp = (unsigned*)(C.pws + WS_XF + (rank * 16 + cw) * 128);
      u32x4 r0[8], r1[8];
#pragma unroll
      for (int s = 0; s < 8; ++s) { r0[s] = (u32x4){0u, 0u, 0u, 0u}; r1[s] = (u32x4){0u, 0u, 0u, 0u}; }
#define CLOAD(s) asm volatile("global_load_dwordx4 %0, %2, off offset:%4\n\tglobal_load_dwordx4 %1, %3, off offset:%4" \
                              : "+v"(r0[s]), "+v"(r1[s]) : "v"(sp0), "v"(sp1), "n"((s) * 128) : "memory")
#define CSTORE(s) do { asm volatile("global_store_dwordx4 %0, %1, off sc0 sc1\n\tglobal_store_dwordx4 %0, %2, off offset:1024 sc0 sc1\n\ts_nop 1" \
                                    :: "v"(dp), "v"(r0[s]), "v"(r1[s]) : "memory"); dp += XT_BYTES; } while (0)
#define CFLAG() do { ++fv; asm volatile("s_mov_b64 exec, 1\n\tglobal_store_dword %0, %1, off sc0 sc1\n\ts_mov_b64 exec, -1" :: "v"(fp), "v"(fv) : "memory"); } while (0)
      CLOAD(0); CLOAD(1); CLOAD(2);
      WAITV(4); CSTORE(0);
      WAITV(4); CSTORE(1);
#pragma unroll
      for (int s = 3; s < 8; ++s) CLOAD(s);
      sp0 += 1024; sp1 += 1024;
      CLOAD(0); CLOAD(1);
      BAR();
      unsigned fv = (xw[1] - 1u) * NT;
      WAITV(16); CFLAG();
      CSTORE(2); CLOAD(2); WAITV(19); CFLAG(); BAR();
      WAITV(10);
#pragma unroll
      for (int t = 3; t < 8; ++t) { CSTORE(t); CLOAD(t); WAITV(7); CFLAG();
        BAR(); }
      for (int k8 = 8; k8 < NT - 8; k8 += 8) {
        sp0 += 1024; sp1 += 1024;
#pragma unroll
        for (int i = 0; i < 8; ++i) { CSTORE(i); CLOAD(i); WAITV(7); CFLAG(); BAR(); }
      }
#pragma unroll
      for (int i = 0; i < 8; ++i) { CSTORE(i); WAITV(3); CFLAG(); BAR(); }
      WAITV(0);
      CFLAG();
      if (wid == 0) { STAMPW(1); STAMPW(2); }
    } else {
      unsigned ticket = 0;
      if (lane == 0) ticket = draw(cnt);
      const unsigned call = (unsigned)__builtin_amdgcn_readfirstlane((int)ticket) + 1u;
      if (lane == 0) xw[1] = call;
      WAITL0();
      BAR();
      for (int k = 2; k < NT; ++k) BAR();
      if (idx == 0) {
        const u8* gp = C.ws + WS_G1 + xcd * 1024 + lane * 16;
        u64* dstg = (u64*)(P.ws[lane & 7] + WS_G2) + (call & 1) * (NR * 8) + rank * 8 + xcd;
        asm volatile("" : "+v"(dstg));
        unsigned spins = 0;
        const u64 t0 = __builtin_amdgcn_s_memrealtime();
        for (;;) {
          u32x4 a;
          asm volatile("global_load_dwordx4 %0, %1, off sc0 sc1\n\ts_waitcnt vmcnt(0)" : "=&v"(a) : "v"(gp) : "memory");
          if (__all(lane < 4 || (a[1] == call && a[3] == call))) {
            const unsigned mx = wave_max_u32(lane < 4 ? 0u : max(a[0], a[2]));
            if (lane < NR) st64(dstg, call, mx);
            break;
          }
          if (give_up(tmo, call, t0, spins)) break;
          __builtin_amdgcn_s_sleep(1);
        }
      }
    }
    return;
  }

  const bool tall = idx >= 16;
  Tile T;
  if (!tall) { T.src = rank; T.dst = (rank + pj) & 7; T.ct = xcd * 2 + (idx - 2) / 7; T.grp = 0; }
  else { const int u = idx - 16; T.grp = (xcd >> 2) * 4 + (u >> 2); T.src = (rank + T.grp) & 7; T.dst = rank; T.ct = (xcd & 3) * 4 + (u & 3); }

  if (wid < 4) {
    if (tall) consumer<9>(C, T, lds, wid, lane); else consumer<8>(C, T, lds, wid, lane);
    return;
  }
  const int f = bid - 128;
  const bool incoming = tall && (f >> 4) != rank;
  const bool remote = T.grp != 0;
  Early E;
  if (wid < 6) {
    if (tall) loader<true>(C, T, lds, wid - 4, lane); else loader<false>(C, T, lds, wid - 4, lane);
  } else if (wid == 6) {
    const u64 tb = __builtin_amdgcn_s_memrealtime();
    unsigned warm = 0;
    if (remote) {
      if (DLY) __builtin_amdgcn_s_sleep(DLY);
#pragma unroll
      for (int i = 0; i < 5; ++i)
        asm volatile("global_load_dword %0, %1, off" : "+v"(warm) : "v"((const char*)C.x + (size_t)min((T.grp - 1) * 16 + T.ct + 112 * i, M - 1) * (K * 2) + lane * 128) : "memory");
    }
    if (remote) { BAR(); BAR(); }
    BAR();
    const int hold = tall ? (remote ? QB : 0) : !(T.ct & 1) ? QA0 : ((T.ct & 2) && T.ct != 15) ? QA3 : QA1;
    int tl = (int)(__builtin_amdgcn_s_memrealtime() - tb);
    for (int k = 1; k <= NT; ++k) {
      if (tl >= QT1 && tl < QT2) {
        tl += hold;
        while ((int)(__builtin_amdgcn_s_memrealtime() - tb) < tl) __builtin_amdgcn_s_sleep(1);
      }
      BAR();
      tl = max(tl, (int)(__builtin_amdgcn_s_memrealtime() - tb) - 40);
    }
    asm volatile("s_waitcnt vmcnt(0)" :: "v"(warm) : "memory");
  }
  if (wid < 7) {
    if (incoming) {
      const unsigned call = xw[1];
      early_fetch(C, E, f, wid - 4, lane, call);
      const bool dead = ld32<AGT>(tmo) == call;
      BAR();
      early_store(C, E, f, wid - 4, lane, dead ? __uint_as_float(0x7fc00000u) : 448.0f / __uint_as_float(xw[0]));
    }
    return;
  }
  unsigned ticket = 0, spins = 0;
  if (lane == 0) ticket = draw(cnt);
  const unsigned call = (unsigned)__builtin_amdgcn_readfirstlane((int)ticket) + 1u;
  if (lane == 0) { xw[1] = call; xw[2] = 0u; }
  WAITL0();
  if (remote) {
    const unsigned* fp = (const unsigned*)(C.ws + WS_XF + (T.src * 16 + (lane & 15)) * 128);
    const unsigned base = (call - 1u) * NT;
    unsigned ready = 0;
    for (int k = -3; k < NT; ++k) {
      const unsigned need = (unsigned)min(k + 4, NT);
      if (ready < (k < 0 ? need : min(need + 2u, (unsigned)NT))) {
        const u64 t0 = __builtin_amdgcn_s_memrealtime();
        for (;;) {
          const unsigned d = ld32<SYS>(fp) - base;
          ready = NT - wave_max_u32(d > (unsigned)NT ? (unsigned)NT : NT - d);
          if (ready >= need) break;
          if (give_up(tmo, call, t0, spins)) { ready = NT; break; }
          __builtin_amdgcn_s_sleep(1);
        }
      }
      BAR();
    }
  } else {
    const u8* cp = C.ws + WS_XC + (lane & 7) * 128;
    unsigned cv = 0;
    for (int k = -1; k < 5; ++k) BAR();
    if (RETILE && !tall) asm volatile("global_load_dword %0, %1, off sc0 sc1" : "+v"(cv) : "v"(cp) : "memory");
    BAR(); BAR();
    if (RETILE && !tall) {
      asm volatile("s_waitcnt vmcnt(0)" : "+v"(cv) :: "memory");
      const u64 t0 = __builtin_amdgcn_s_memrealtime();
      while (!__all(cv - call * 32u < 0x80000000u)) {
        if (give_up(tmo, call, t0, spins)) break;
        __builtin_amdgcn_s_sleep(1);
        cv = ld32<SYS>(cp);
      }
    }
    for (int k = 7; k < NT; ++k) BAR();
  }
  if (incoming) early_fetch(C, E, f, 3, lane, call);
  unsigned mx;
  {
    const u64* gp = (const u64*)(C.ws + WS_G2) + (call & 1) * (NR * 8) + lane;
    const u64 t0 = __builtin_amdgcn_s_memrealtime();
    for (;;) {
      u64 a; unsigned tm;
      asm volatile("global_load_dwordx2 %0, %2, off sc0 sc1\n\tglobal_load_dword %1, %3, off sc1\n\ts_waitcnt vmcnt(0)"
                   : "=&v"(a), "=&v"(tm) : "v"(gp), "v"(tmo) : "memory");
      if (tm == call) { mx = 0x7fc00000u; break; }
      if (__all((unsigned)(a >> 32) == call)) { mx = wave_max_u32((unsigned)a); break; }
      if (__builtin_amdgcn_s_memrealtime() - t0 > DEADLINE_TICKS) st32<AGT>(tmo, call);
      __builtin_amdgcn_s_sleep(GS);
    }
    if (lane == 0) { xw[0] = mx; __hip_atomic_store(&xw[2], 1u, __ATOMIC_RELAXED, __HIP_MEMORY_SCOPE_WORKGROUP); }
    WAITL0();
  }
  BAR();
  if (!tall) {
    BAR();
    if (lane == 0) st32<SYS>(C.pws + WS_DONE + (size_t)(rank * 16 + T.ct) * 128, call);
  } else if (incoming) {
    early_store(C, E, f, 3, lane, 448.0f / __uint_as_float(mx));
    const unsigned* fl = (const unsigned*)(C.ws + WS_DONE + (size_t)f * 128);
    const u64 t1 = __builtin_amdgcn_s_memrealtime();
    while (ld32<SYS>(fl) < call) {
      if (give_up(tmo, call, t1, spins)) { if (lane == 0) *(GAS u8*)(size_t)(C.out + (size_t)rank * M * NS) = 0x7f; break; }
      __builtin_amdgcn_s_sleep(1);
    }
  }
  STAMPW(7);
}

extern "C" void kernel_launch(int rank, int nranks, ncclComm_t comm, hipStream_t stream,
                              void* const* d_in, const int* in_sizes, int n_in,
                              void* d_out, int out_size, const int* in_dtypes, int out_dtype,
                              void* const* d_out_all, void* const* d_ws_all, size_t ws_bytes) {
  (void)comm; (void)in_sizes; (void)n_in; (void)d_out; (void)out_size; (void)in_dtypes; (void)out_dtype;
  if (nranks != NR || ws_bytes < WS_END || (unsigned)rank >= (unsigned)NR) { fprintf(stderr, "unsupported configuration\n"); return; }
  static bool init = false;
  if (!init) {
    HIP_CHECK(hipFuncSetAttribute((const void*)gemm_a2a_q, hipFuncAttributeMaxDynamicSharedMemorySize, LDS_BYTES));
    init = true;
  }
  Params P;
  P.x = (const u16*)d_in[0]; P.w = (const u16*)d_in[1];
  for (int j = 0; j < NR; ++j) { P.out[j] = (u8*)d_out_all[(rank + j) % NR]; P.ws[j] = (u8*)d_ws_all[(rank + j) % NR]; }
  P.rank = rank; P.pad = 0;
  hipLaunchKernelGGL(gemm_a2a_q, dim3(NWG), dim3(512), LDS_BYTES, stream, P);
  HIP_CHECK(hipGetLastError());
}
```

```cpp
#include <hip/hip_runtime.h>
#include <rccl/rccl.h>
#include <cstdio>

#define HIP_CHECK(expr) do { hipError_t _e = (expr); if (_e != hipSuccess) { \
    fprintf(stderr, "HIP error %s at %s:%d\n", hipGetErrorString(_e), __FILE__, __LINE__); \
    return; } } while (0)

#define LAS __attribute__((address_space(3)))
#define GAS __attribute__((address_space(1)))
typedef short s16x4 __attribute__((ext_vector_type(4)));
typedef short s16x8 __attribute__((ext_vector_type(8)));
typedef __bf16 bf16x8 __attribute__((ext_vector_type(8)));
typedef float f32x4 __attribute__((ext_vector_type(4)));
typedef unsigned u32x4 __attribute__((ext_vector_type(4)));
typedef unsigned short u16;
typedef unsigned long long u64;
typedef unsigned char u8;

constexpr int NR = 8;
constexpr int M = 512, N = 16384, K = 4096;
constexpr int NS = N / NR;
constexpr int BM = 256, BN = 128, BK = 64, NT = K / BK;
constexpr int NWG = 256;
constexpr int XT_BYTES = BM * BK * 2;
constexpr int A_BYTES = 288 * BK * 2, B_BYTES = BK * BN * 2;
constexpr int B_BASE = 3 * A_BYTES;
constexpr int X_BASE = B_BASE + 3 * B_BYTES;
constexpr int LDS_BYTES = X_BASE + 256;
constexpr int AUXW = 2;
constexpr int QT1 = 1600, QT2 = 3620, QA0 = 68, QA1 = 90, QA3 = 124, QB = 127, ECUT = 6050;

constexpr size_t WS_CNT = 0;
constexpr size_t WS_TMO = WS_CNT + (size_t)NWG * 128;
constexpr size_t WS_G1 = WS_TMO + 256;
constexpr size_t WS_G2 = WS_G1 + 8 * 128 * 8;
constexpr size_t WS_DONE = WS_G2 + 2 * NR * 8 * 8;
constexpr size_t WS_XF = WS_DONE + (size_t)NR * 16 * 128;
constexpr size_t WS_EF = WS_XF + (size_t)NR * 16 * 128;
constexpr size_t WS_XC = WS_EF + (size_t)2 * NR * 64 * 128;
constexpr size_t WS_XS = 1u << 20;
constexpr size_t XS_STRIDE = (size_t)NT * XT_BYTES;
constexpr size_t WS_ES = WS_XS + (size_t)NR * XS_STRIDE;
constexpr size_t WS_END = WS_ES + (size_t)2 * NR * 64 * 16384;
static_assert(WS_XC + 8 * 128 <= WS_XS, "layout");
constexpr u64 DEADLINE_TICKS = 40000000ull;

struct Params {
  const u16* x; const u16* w;
  int rank; int pad;
  u8* ws[NR];
  u8* out[NR];
};
struct Ctx { const u16* x; const u16* w; u8* ws; u8* out; u8* pws; u8* pout; int rank; };
struct Tile { int src, dst, ct, grp; };

#define SB() __builtin_amdgcn_sched_barrier(0)
#define BAR() do { asm volatile("" ::: "memory"); __builtin_amdgcn_s_barrier(); asm volatile("" ::: "memory"); } while (0)
#define WAITV(n) asm volatile("s_waitcnt vmcnt(%0)" :: "n"(n) : "memory")
#define WAITL0() asm volatile("s_waitcnt lgkmcnt(0)" ::: "memory")
#define DI __device__ __forceinline__

#define SYS __HIP_MEMORY_SCOPE_SYSTEM
#define AGT __HIP_MEMORY_SCOPE_AGENT
template <int SC> DI unsigned ld32(const void* p) { return __hip_atomic_load((const GAS unsigned*)(size_t)p, __ATOMIC_RELAXED, SC); }
template <int SC> DI void st32(void* p, unsigned v) { __hip_atomic_store((GAS unsigned*)(size_t)p, v, __ATOMIC_RELAXED, SC); }
DI void st64(void* p, unsigned tag, unsigned v) { __hip_atomic_store((GAS u64*)(size_t)p, ((u64)tag << 32) | v, __ATOMIC_RELAXED, SYS); }
DI unsigned draw(void* p) { return __hip_atomic_fetch_add((GAS unsigned*)(size_t)p, 1u, __ATOMIC_RELAXED, AGT); }
DI bf16x8 ldA(unsigned addr, int off) { return *(LAS bf16x8*)(size_t)(addr + off); }
DI s16x4 ldB(unsigned addr, int off) { return __builtin_amdgcn_ds_read_tr16_b64_v4i16((LAS s16x4*)(size_t)(addr + off)); }
DI unsigned wave_max_u32(unsigned v) {
  v = max(v, (unsigned)__builtin_amdgcn_update_dpp(0, (int)v, 0x111, 0xf, 0xf, false));
  v = max(v, (unsigned)__builtin_amdgcn_update_dpp(0, (int)v, 0x112, 0xf, 0xf, false));
  v = max(v, (unsigned)__builtin_amdgcn_update_dpp(0, (int)v, 0x114, 0xf, 0xf, false));
  v = max(v, (unsigned)__builtin_amdgcn_update_dpp(0, (int)v, 0x118, 0xf, 0xf, false));
  const unsigned a = __builtin_amdgcn_readlane(v, 15), b = __builtin_amdgcn_readlane(v, 31);
  const unsigned c = __builtin_amdgcn_readlane(v, 47), d = __builtin_amdgcn_readlane(v, 63);
  return max(max(a, b), max(c, d));
}
DI bool give_up(unsigned* tmo, unsigned call, u64 t0, unsigned& spins) {
  if ((++spins & 7u) != 0u) return false;
  if (ld32<AGT>(tmo) == call || __builtin_amdgcn_s_memrealtime() - t0 > DEADLINE_TICKS) {
    st32<AGT>(tmo, call);
    return true;
  }
  return false;
}
DI unsigned pkbf(float a, float b) {
  typedef float f32x2 __attribute__((ext_vector_type(2)));
  typedef __bf16 bf16x2 __attribute__((ext_vector_type(2)));
  return __builtin_bit_cast(unsigned, __builtin_convertvector((f32x2){a, b}, bf16x2));
}
DI unsigned q4(float a, float b, float c, float d, float inv) {
  const int t = __builtin_amdgcn_cvt_pk_fp8_f32(a * inv, b * inv, 0, false);
  return (unsigned)__builtin_amdgcn_cvt_pk_fp8_f32(c * inv, d * inv, t, true);
}
DI void store16(u8* p, const u32x4 v) { asm volatile("global_store_dwordx4 %0, %1, off sc0 sc1\n\ts_nop 1" :: "v"(p), "v"(v) : "memory"); }
DI void store_row16(u8* p, unsigned w0, unsigned w1, unsigned w2, unsigned w3, bool odd) {
  store16(p, (u32x4){odd ? w1 : w0, odd ? w0 : w1, odd ? w3 : w2, odd ? w2 : w3});
}

DI unsigned q4b(unsigned lo, unsigned hi, float inv) {
  return q4(__uint_as_float(lo << 16), __uint_as_float(lo & 0xffff0000u), __uint_as_float(hi << 16), __uint_as_float(hi & 0xffff0000u), inv);
}
DI void store_bf16_row(u8* p, const u32x4 a, const u32x4 b, float inv, bool odd) {
  store_row16(p, q4b(a[0], a[1], inv), q4b(a[2], a[3], inv), q4b(b[0], b[1], inv), q4b(b[2], b[3], inv), odd);
}

template <bool TALL>
DI void loader(const Ctx& C, const Tile T, LAS char* lds, const int p, const int lane) {
  const bool remote = TALL && T.grp != 0;
  const void* base = remote ? (const void*)(C.ws + WS_XS + (size_t)T.src * XS_STRIDE) : (const void*)(C.x + (size_t)(TALL ? BM : 0) * K);
  __amdgpu_buffer_rsrc_t rA = __builtin_amdgcn_make_buffer_rsrc((void*)base, 0, BM * K * 2, 0x00020000);
  __amdgpu_buffer_rsrc_t rE = __builtin_amdgcn_make_buffer_rsrc((void*)(C.x + (size_t)T.grp * 32 * K), 0, 32 * K * 2, 0x00020000);
  __amdgpu_buffer_rsrc_t rB = __builtin_amdgcn_make_buffer_rsrc((void*)(C.w + (size_t)T.dst * NS + T.ct * BN), 0, (K - 1) * N * 2 + BN * 2, 0x00020000);
  const unsigned pstr = remote ? 1024u : 8u * K * 2, kstr = remote ? (unsigned)XT_BYTES : BK * 2u;
  const unsigned voL = (unsigned)(((lane >> 3) + 8 * p) * (K * 2) + (((lane & 7) ^ ((p << 2) | (lane >> 4))) << 4));
  const unsigned voA = remote ? (unsigned)lane * 16u + p * 1024u : voL;
  const unsigned voB = (unsigned)(((lane >> 4) + 8 * p) * (N * 2) + (((lane & 15) ^ ((lane >> 4) | (p << 3))) << 4));
  __amdgpu_buffer_rsrc_t rX = __builtin_amdgcn_make_buffer_rsrc((void*)(C.ws + WS_XS + (size_t)C.rank * XS_STRIDE), 0, BM * K * 2, 0x00020000);
  auto issueA = [&](int kt, int buf) {
    if (!TALL && kt > 8 && ((kt & 7) == 3 || kt == NT - 1)) {
#pragma unroll
      for (int j = 0; j < 16; ++j)
        __builtin_amdgcn_raw_ptr_buffer_load_lds(rX, (LAS void*)(lds + buf * A_BYTES + (p + 2 * j + 2 * (j >> 3)) * 1024), 16, (unsigned)lane * 16u + p * 1024u, kt * XT_BYTES + 2 * j * 1024, 0, 17);
      return;
    }
#pragma unroll
    for (int j = 0; j < 16; ++j)
      __builtin_amdgcn_raw_ptr_buffer_load_lds(rA, (LAS void*)(lds + buf * A_BYTES + (p + 2 * j + 2 * (j >> 3)) * 1024), 16, voA, kt * kstr + 2 * j * pstr, 0, TALL ? 17 : 0);
    if (TALL) {
#pragma unroll
      for (int j = 0; j < 2; ++j)
        __builtin_amdgcn_raw_ptr_buffer_load_lds(rE, (LAS void*)(lds + buf * A_BYTES + (16 + p + 18 * j) * 1024), 16, voL, kt * (BK * 2) + j * 16 * K * 2, 0, 0);
    }
  };
  auto issueB = [&](int kt, int buf) {
#pragma unroll
    for (int j = 0; j < 8; ++j) {
      const int pc = (j & 1) + 4 * (j >> 1);
      __builtin_amdgcn_raw_ptr_buffer_load_lds(rB, (LAS void*)(lds + B_BASE + buf * B_BYTES + (pc + 2 * p) * 1024), 16, voB, (kt * BK + pc * 4) * (N * 2), 0, TALL ? 0 : AUXW);
    }
  };
  constexpr int NA = TALL ? 18 : 16;
  if (remote) {
    __builtin_amdgcn_s_sleep(24);
    issueB(0, 0); issueB(1, 1);
    BAR();
    issueA(0, 0);
    BAR();
    issueA(1, 1);
    WAITV(NA);
  } else {
    issueB(0, 0); issueA(0, 0); issueB(1, 1); issueA(1, 1);
    WAITV(NA + 8);
  }
  BAR();
  int buf = 2;
  for (int kt = 0; kt < NT - 2; ++kt) {
    issueB(kt + 2, buf); issueA(kt + 2, buf);
    WAITV(NA + 8);
    buf = (buf == 2) ? 0 : buf + 1;
    BAR();
  }
  WAITV(0);
  BAR(); BAR();
}

template <int MB>
DI void consumer(const Ctx& C, const Tile T, LAS char* lds, const int wid, const int lane) {
  const u64 tb = __builtin_amdgcn_s_memrealtime();
  const unsigned lbase = (unsigned)(size_t)lds;
  LAS unsigned* xw = (LAS unsigned*)(lds + X_BASE);
  const int wr = wid >> 1, wc = wid & 1;
  const int g = lane >> 4, c = lane & 15, q = (lane & 15) >> 2, pp = lane & 3;
  unsigned aA[2], aB[4];
#pragma unroll
  for (int ks = 0; ks < 2; ++ks) aA[ks] = lbase + (wr * 144 + c) * 128 + (((4 * ks + g) ^ (c >> 1)) << 4);
#pragma unroll
  for (int b = 0; b < 4; ++b) {
    const int row = 8 * g + q, ch = 8 * wc + 2 * pp + (b >> 1);
    aB[b] = lbase + B_BASE + row * 256 + ((ch ^ ((row & 3) | (row & 8))) << 4) + (((b ^ pp) & 1) << 3);
  }
  f32x4 acc[MB][4];
  bf16x8 Af[MB][2];
  s16x4 Bq[2][4][2][2];
  auto rdA = [&](int m, int ks) { Af[m][ks] = ldA(aA[ks], m * 2048); SB(); };
  auto rdB = [&](int set, int i) { Bq[set][i >> 2][(i >> 1) & 1][i & 1] = ldB(aB[i >> 2], ((i >> 1) & 1) * 8192 + (i & 1) * 1024); SB(); };
  auto mma = [&](int set, int m, int i) {
    const int b = (m & 1) ? 3 - (i >> 1) : (i >> 1), ks = ((i >> 1) ^ i) & 1;
    s16x8 bv = __builtin_shufflevector(Bq[set][b][ks][0], Bq[set][b][ks][1], 0, 1, 2, 3, 4, 5, 6, 7);
    acc[m][b] = __builtin_amdgcn_mfma_f32_16x16x32_bf16(__builtin_bit_cast(bf16x8, bv), Af[m][ks], acc[m][b], 0, 0, 0);
    SB();
  };
  int buf = 0;
  auto ktile = [&](int set) {
    int da = 0, db = 0;
#pragma unroll
    for (int m = 0; m < MB - 4; ++m)
#pragma unroll
      for (int i = 0; i < 8; ++i) {
        mma(set, m, i);
        if (m == 0 && i < 6) rdA(2 + (i >> 1), i & 1);
        if (m > 0 && i < 2) rdA(m + 4, i);
        if (m == 0 && i == 6) { db = buf == 2 ? -2 * B_BYTES : B_BYTES; asm volatile("" : "+s"(db)); SB(); }
        if (m == 1 && i >= 3 && i < 7) { aB[i - 3] += db; SB(); }
        if (m == 2 && i == 3) { da = buf == 2 ? -2 * A_BYTES : A_BYTES; asm volatile("" : "+s"(da)); SB(); }
        if (m == 2 && i == 5) { buf = buf == 2 ? -1 : buf; asm volatile("" : "+s"(buf)); SB(); }
        if (m == 2 && i == 6) { ++buf; asm volatile("" : "+s"(buf)); SB(); }
        if (m == MB - 5 && (i == 3 || i == 4)) { aA[i - 3] += da; SB(); }
      }
    __builtin_amdgcn_s_waitcnt(0xc07f);
    SB();
    BAR();
    SB();
#pragma unroll
    for (int m = MB - 4; m < MB; ++m)
#pragma unroll
      for (int i = 0; i < 8; ++i) {
        const int j = (m - (MB - 4)) * 8 + i;
        mma(set, m, i);
        if (j < 16) rdB(set ^ 1, j);
        if (j >= 16 && j < 20) rdA((j - 16) >> 1, j & 1);
      }
  };
#pragma unroll
  for (int m = 0; m < MB; ++m)
#pragma unroll
    for (int b = 0; b < 4; ++b) { acc[m][b] = (f32x4){0.f, 0.f, 0.f, 0.f}; asm volatile("" : "+v"(acc[m][b])); }
  SB();
  const int id = ((T.grp - 1) * 16 + T.ct) * 3 + wid;
  if (MB == 9 && T.grp != 0 && wid < 3 && id < 256) {
    const int bi = id >> 5, pc = id & 31, kt = bi < 7 ? 11 + 8 * bi : NT - 1;
    u32x4 v;
    __builtin_amdgcn_s_sleep(24);
    asm volatile("global_load_dwordx4 %0, %1, off\n\ts_waitcnt vmcnt(0)" : "=&v"(v)
                 : "v"((const char*)(C.x + (size_t)(8 * pc + (lane >> 3)) * K + kt * BK) + (((lane & 7) ^ (((pc & 1) << 2) | (lane >> 4))) << 4)) : "memory");
    store16(C.ws + WS_XS + (size_t)C.rank * XS_STRIDE + kt * XT_BYTES + pc * 1024 + lane * 16, v);
    WAITV(0);
    if (lane == 0) __hip_atomic_fetch_add((GAS unsigned*)(size_t)(C.ws + WS_XC + bi * 128), 1u, __ATOMIC_RELAXED, SYS);
  }
  if (MB == 9 && T.grp != 0) { BAR(); BAR(); }
  BAR();
#pragma unroll
  for (int i = 0; i < 16; ++i) rdB(0, i);
#pragma unroll
  for (int i = 0; i < 4; ++i) rdA(i >> 1, i & 1);
#pragma unroll 1
  for (int kt = 0; kt < NT; kt += 2) { ktile(0); ktile(1); }
  float am = 0.f;
#pragma unroll
  for (int m = 0; m < MB; ++m)
#pragma unroll
    for (int b = 0; b < 4; ++b) {
      am = __builtin_fmaxf(__builtin_fmaxf(am, __builtin_fabsf(acc[m][b][0])), __builtin_fabsf(acc[m][b][1]));
      am = __builtin_fmaxf(__builtin_fmaxf(am, __builtin_fabsf(acc[m][b][2])), __builtin_fabsf(acc[m][b][3]));
    }
  const unsigned mu = wave_max_u32(__float_as_uint(am));
  const unsigned call = xw[1];
  if (lane == 0)
    st64((u64*)(C.ws + WS_G1) + (blockIdx.x & 7) * 128 + (blockIdx.x >> 3) * 4 + wid, call, mu);
  u8* ob = C.pout + T.ct * BN + wc * 64 + 16 * g;
  u8* op = ob + (size_t)(T.src * M + (MB == 9 ? BM : 0) + wr * 128 + c) * NS;
  if (MB == 8) {
    u32x4 pk[8][2];
#pragma unroll
    for (int m = 0; m < 8; ++m)
#pragma unroll
      for (int h = 0; h < 2; ++h)
        pk[m][h] = (u32x4){pkbf(acc[m][2 * h][0], acc[m][2 * h][1]), pkbf(acc[m][2 * h][2], acc[m][2 * h][3]),
                           pkbf(acc[m][2 * h + 1][0], acc[m][2 * h + 1][1]), pkbf(acc[m][2 * h + 1][2], acc[m][2 * h + 1][3])};
    const int slot = (call & 1) * (NR * 64) + (C.rank * 16 + T.ct) * 4 + wid;
    u8* ep = C.pws + WS_ES + (size_t)slot * 16384 + lane * 16;
    int ne = 0;
#pragma unroll
    for (int m = 0; m < 8; ++m)
      if (ne == m && (int)(__builtin_amdgcn_s_memrealtime() - tb) < ECUT &&
          __builtin_amdgcn_readfirstlane(__hip_atomic_load(&xw[2], __ATOMIC_RELAXED, __HIP_MEMORY_SCOPE_WORKGROUP)) == 0) {
#pragma unroll
        for (int h = 0; h < 2; ++h) {
          store16(ep + m * 2048 + h * 1024, pk[m][h]);
          WAITV(1);
          __builtin_amdgcn_s_sleep(8);
        }
        ne = m + 1;
      }
    WAITV(0);
    if (lane == 0) st32<SYS>(C.pws + WS_EF + (size_t)slot * 128, call * 16u + ne);
    BAR();
    const float inv = 448.0f / __uint_as_float(xw[0]);
#pragma unroll
    for (int m = 7; m >= 0; --m)
      if (m >= ne) { store_bf16_row(op + (size_t)m * 16 * NS, pk[m][0], pk[m][1], inv, g & 1); SB(); }
  } else {
    u8* oe = ob + (size_t)(C.rank * M + T.grp * 32 + wr * 16 + c) * NS;
    BAR();
    const float inv = 448.0f / __uint_as_float(xw[0]);
#pragma unroll
    for (int m = 0; m < MB; ++m) {
      unsigned w[4];
#pragma unroll
      for (int b = 0; b < 4; ++b) w[b] = q4(acc[m][b][0], acc[m][b][1], acc[m][b][2], acc[m][b][3], inv);
      store_row16(m < 8 ? op + (size_t)m * 16 * NS : oe, w[0], w[1], w[2], w[3], g & 1);
      SB();
    }
  }
  if (MB == 8) {
    WAITV(0);
    BAR();
  }
}

struct Early { u32x4 v[16]; int ne; };
#define LD8(o, p) asm volatile("global_load_dwordx4 %0, %8, off sc0 sc1\n\tglobal_load_dwordx4 %1, %8, off offset:1024 sc0 sc1\n\t" \
               "global_load_dwordx4 %2, %8, off offset:2048 sc0 sc1\n\tglobal_load_dwordx4 %3, %8, off offset:3072 sc0 sc1\n\t" \
               "global_load_dwordx4 %4, %9, off sc0 sc1\n\tglobal_load_dwordx4 %5, %9, off offset:1024 sc0 sc1\n\t" \
               "global_load_dwordx4 %6, %9, off offset:2048 sc0 sc1\n\tglobal_load_dwordx4 %7, %9, off offset:3072 sc0 sc1\n\ts_waitcnt vmcnt(0)" \
               : "=&v"(E.v[o]), "=&v"(E.v[o + 1]), "=&v"(E.v[o + 2]), "=&v"(E.v[o + 3]), "=&v"(E.v[o + 4]), "=&v"(E.v[o + 5]), "=&v"(E.v[o + 6]), "=&v"(E.v[o + 7]) \
               : "v"(p), "v"((p) + 4096) : "memory")
DI void early_fetch(const Ctx& C, Early& E, const int f, const int w, const int lane, const unsigned call) {
  const int slot = (call & 1) * (NR * 64) + f * 4 + w;
  const unsigned* fl = (const unsigned*)(C.ws + WS_EF + (size_t)slot * 128);
  unsigned spins = 0;
  const u64 t0 = __builtin_amdgcn_s_memrealtime();
  unsigned v;
  while ((v = (unsigned)__builtin_amdgcn_readfirstlane((int)ld32<SYS>(fl))) < call * 16u) {
    if (give_up((unsigned*)(C.ws + WS_TMO), call, t0, spins)) { v = call * 16u + 8u; break; }
    __builtin_amdgcn_s_sleep(2);
  }
  E.ne = (int)(v - call * 16u);
  const u8* ep = C.ws + WS_ES + (size_t)slot * 16384 + lane * 16;
  if (E.ne > 0) LD8(0, ep);
  if (E.ne > 4) LD8(8, ep + 8192);
}
DI void early_store(const Ctx& C, const Early& E, const int f, const int w, const int lane, const float inv) {
  u8* op = C.out + (size_t)((f >> 4) * M + (w >> 1) * 128 + (lane & 15)) * NS + (f & 15) * BN + (w & 1) * 64 + (lane & 48);
#pragma unroll
  for (int m = 0; m < 8; ++m)
    if (m < E.ne) store_bf16_row(op + (size_t)m * 16 * NS, E.v[2 * m], E.v[2 * m + 1], inv, (lane >> 4) & 1);
}

__global__ __launch_bounds__(512, 2) void gemm_a2a_q(const Params P) {
  extern __shared__ __attribute__((aligned(16))) char smem[];
  LAS char* lds = (LAS char*)smem;
  const int tid = threadIdx.x, lane = tid & 63;
  const int wid = __builtin_amdgcn_readfirstlane(tid >> 6);
  const int bid = blockIdx.x;
  const int xcd = bid & 7, idx = bid >> 3;
  const int pj = (idx < 2 ? wid + 1 : idx < 16 ? 1 + (idx - 2) % 7 : 0) & 7;
  Ctx C = {P.x, P.w, P.ws[0], P.out[0], P.ws[pj], P.out[pj], P.rank};
  asm volatile("" : "+s"(C.x), "+s"(C.w), "+s"(C.ws), "+s"(C.out), "+s"(C.pws), "+s"(C.pout), "+s"(C.rank));
  const int rank = C.rank;
  unsigned* const tmo = (unsigned*)(C.ws + WS_TMO);
  unsigned* const cnt = (unsigned*)(C.ws + WS_CNT + (size_t)bid * 128);
  LAS unsigned* xw = (LAS unsigned*)(lds + X_BASE);

  if (idx < 2) {
    const int cw = xcd * 2 + idx;
    if (wid < 7) {
      const char* sp = (const char*)(C.x + (size_t)(BM + 16 * cw + (lane >> 3)) * K);
      const char* sp0 = sp + (((lane & 7) ^ (lane >> 4)) << 4);
      const char* sp1 = sp + 8 * K * 2 + (((lane & 7) ^ (4 | (lane >> 4))) << 4);
      u8* dp = C.pws + WS_XS + (size_t)rank * XS_STRIDE + cw * 2048 + lane * 16;
      unsigned* fp = (unsigned*)(C.pws + WS_XF + (rank * 16 + cw) * 128);
      u32x4 r0[8], r1[8];
#pragma unroll
      for (int s = 0; s < 8; ++s) { r0[s] = (u32x4){0u, 0u, 0u, 0u}; r1[s] = (u32x4){0u, 0u, 0u, 0u}; }
#define CLOAD(s) asm volatile("global_load_dwordx4 %0, %2, off offset:%4\n\tglobal_load_dwordx4 %1, %3, off offset:%4" \
                              : "+v"(r0[s]), "+v"(r1[s]) : "v"(sp0), "v"(sp1), "n"((s) * 128) : "memory")
#define CSTORE(s) do { asm volatile("global_store_dwordx4 %0, %1, off sc0 sc1\n\tglobal_store_dwordx4 %0, %2, off offset:1024 sc0 sc1\n\ts_nop 1" \
                                    :: "v"(dp), "v"(r0[s]), "v"(r1[s]) : "memory"); dp += XT_BYTES; } while (0)
#define CFLAG() do { ++fv; asm volatile("s_mov_b64 exec, 1\n\tglobal_store_dword %0, %1, off sc0 sc1\n\ts_mov_b64 exec, -1" :: "v"(fp), "v"(fv) : "memory"); } while (0)
      CLOAD(0); CLOAD(1); CLOAD(2);
      WAITV(4); CSTORE(0);
      WAITV(4); CSTORE(1);
#pragma unroll
      for (int s = 3; s < 8; ++s) CLOAD(s);
      sp0 += 1024; sp1 += 1024;
      CLOAD(0); CLOAD(1);
      BAR();
      unsigned fv = (xw[1] - 1u) * NT;
      WAITV(16); CFLAG();
      CSTORE(2); CLOAD(2); WAITV(19); CFLAG(); BAR();
      WAITV(10);
#pragma unroll
      for (int t = 3; t < 8; ++t) { CSTORE(t); CLOAD(t); WAITV(7); CFLAG(); BAR(); }
      for (int k8 = 8; k8 < NT - 8; k8 += 8) {
        sp0 += 1024; sp1 += 1024;
#pragma unroll
        for (int i = 0; i < 8; ++i) { CSTORE(i); CLOAD(i); WAITV(7); CFLAG(); BAR(); }
      }
#pragma unroll
      for (int i = 0; i < 8; ++i) { CSTORE(i); WAITV(3); CFLAG(); BAR(); }
      WAITV(0);
      CFLAG();
    } else {
      unsigned ticket = 0;
      if (lane == 0) ticket = draw(cnt);
      const unsigned call = (unsigned)__builtin_amdgcn_readfirstlane((int)ticket) + 1u;
      if (lane == 0) xw[1] = call;
      WAITL0();
      BAR();
      for (int k = 2; k < NT; ++k) BAR();
      if (idx == 0) {
        const u8* gp = C.ws + WS_G1 + xcd * 1024 + lane * 16;
        u64* dstg = (u64*)(P.ws[lane & 7] + WS_G2) + (call & 1) * (NR * 8) + rank * 8 + xcd;
        asm volatile("" : "+v"(dstg));
        unsigned spins = 0;
        const u64 t0 = __builtin_amdgcn_s_memrealtime();
        for (;;) {
          u32x4 a;
          asm volatile("global_load_dwordx4 %0, %1, off sc0 sc1\n\ts_waitcnt vmcnt(0)" : "=&v"(a) : "v"(gp) : "memory");
          if (__all(lane < 4 || (a[1] == call && a[3] == call))) {
            const unsigned mx = wave_max_u32(lane < 4 ? 0u : max(a[0], a[2]));
            if (lane < NR) st64(dstg, call, mx);
            break;
          }
          if (give_up(tmo, call, t0, spins)) break;
          __builtin_amdgcn_s_sleep(1);
        }
      }
    }
    return;
  }

  const bool tall = idx >= 16;
  Tile T;
  if (!tall) { T.src = rank; T.dst = (rank + pj) & 7; T.ct = xcd * 2 + (idx - 2) / 7; T.grp = 0; }
  else { const int u = idx - 16; T.grp = (xcd >> 2) * 4 + (u >> 2); T.src = (rank + T.grp) & 7; T.dst = rank; T.ct = (xcd & 3) * 4 + (u & 3); }

  if (wid < 4) {
    if (tall) consumer<9>(C, T, lds, wid, lane); else consumer<8>(C, T, lds, wid, lane);
    return;
  }
  const int f = bid - 128;
  const bool incoming = tall && (f >> 4) != rank;
  const bool remote = T.grp != 0;
  Early E;
  if (wid < 6) {
    if (tall) loader<true>(C, T, lds, wid - 4, lane); else loader<false>(C, T, lds, wid - 4, lane);
  } else if (wid == 6) {
    const u64 tb = __builtin_amdgcn_s_memrealtime();
    unsigned warm = 0;
    if (remote) {
      __builtin_amdgcn_s_sleep(24);
#pragma unroll
      for (int i = 0; i < 5; ++i)
        asm volatile("global_load_dword %0, %1, off" : "+v"(warm) : "v"((const char*)C.x + (size_t)min((T.grp - 1) * 16 + T.ct + 112 * i, M - 1) * (K * 2) + lane * 128) : "memory");
    }
    if (remote) { BAR(); BAR(); }
    BAR();
    const int hold = tall ? (remote ? QB : 0) : !(T.ct & 1) ? QA0 : ((T.ct & 2) && T.ct != 15) ? QA3 : QA1;
    int tl = (int)(__builtin_amdgcn_s_memrealtime() - tb);
    for (int k = 1; k <= NT; ++k) {
      if (tl >= QT1 && tl < QT2) {
        tl += hold;
        while ((int)(__builtin_amdgcn_s_memrealtime() - tb) < tl) __builtin_amdgcn_s_sleep(1);
      }
      BAR();
      tl = max(tl, (int)(__builtin_amdgcn_s_memrealtime() - tb) - 40);
    }
    asm volatile("s_waitcnt vmcnt(0)" :: "v"(warm) : "memory");
  }
  if (wid < 7) {
    if (incoming) {
      const unsigned call = xw[1];
      early_fetch(C, E, f, wid - 4, lane, call);
      const bool dead = ld32<AGT>(tmo) == call;
      BAR();
      early_store(C, E, f, wid - 4, lane, dead ? __uint_as_float(0x7fc00000u) : 448.0f / __uint_as_float(xw[0]));
    }
    return;
  }
  unsigned ticket = 0, spins = 0;
  if (lane == 0) ticket = draw(cnt);
  const unsigned call = (unsigned)__builtin_amdgcn_readfirstlane((int)ticket) + 1u;
  if (lane == 0) { xw[1] = call; xw[2] = 0u; }
  WAITL0();
  if (remote) {
    const unsigned* fp = (const unsigned*)(C.ws + WS_XF + (T.src * 16 + (lane & 15)) * 128);
    const unsigned base = (call - 1u) * NT;
    unsigned ready = 0;
    for (int k = -3; k < NT; ++k) {
      const unsigned need = (unsigned)min(k + 4, NT);
      if (ready < (k < 0 ? need : min(need + 2u, (unsigned)NT))) {
        const u64 t0 = __builtin_amdgcn_s_memrealtime();
        for (;;) {
          const unsigned d = ld32<SYS>(fp) - base;
          ready = NT - wave_max_u32(d > (unsigned)NT ? (unsigned)NT : NT - d);
          if (ready >= need) break;
          if (give_up(tmo, call, t0, spins)) { ready = NT; break; }
          __builtin_amdgcn_s_sleep(1);
        }
      }
      BAR();
    }
  } else {
    const u8* cp = C.ws + WS_XC + (lane & 7) * 128;
    unsigned cv = 0;
    for (int k = -1; k < 5; ++k) BAR();
    if (!tall) asm volatile("global_load_dword %0, %1, off sc0 sc1" : "+v"(cv) : "v"(cp) : "memory");
    BAR(); BAR();
    if (!tall) {
      asm volatile("s_waitcnt vmcnt(0)" : "+v"(cv) :: "memory");
      const u64 t0 = __builtin_amdgcn_s_memrealtime();
      while (!__all(cv - call * 32u < 0x80000000u)) {
        if (give_up(tmo, call, t0, spins)) break;
        __builtin_amdgcn_s_sleep(1);
        cv = ld32<SYS>(cp);
      }
    }
    for (int k = 7; k < NT; ++k) BAR();
  }
  if (incoming) early_fetch(C, E, f, 3, lane, call);
  unsigned mx;
  {
    const u64* gp = (const u64*)(C.ws + WS_G2) + (call & 1) * (NR * 8) + lane;
    const u64 t0 = __builtin_amdgcn_s_memrealtime();
    for (;;) {
      u64 a; unsigned tm;
      asm volatile("global_load_dwordx2 %0, %2, off sc0 sc1\n\tglobal_load_dword %1, %3, off sc1\n\ts_waitcnt vmcnt(0)"
                   : "=&v"(a), "=&v"(tm) : "v"(gp), "v"(tmo) : "memory");
      if (tm == call) { mx = 0x7fc00000u; break; }
      if (__all((unsigned)(a >> 32) == call)) { mx = wave_max_u32((unsigned)a); break; }
      if (__builtin_amdgcn_s_memrealtime() - t0 > DEADLINE_TICKS) st32<AGT>(tmo, call);
      __builtin_amdgcn_s_sleep(2);
    }
    if (lane == 0) { xw[0] = mx; __hip_atomic_store(&xw[2], 1u, __ATOMIC_RELAXED, __HIP_MEMORY_SCOPE_WORKGROUP); }
    WAITL0();
  }
  BAR();
  if (!tall) {
    BAR();
    if (lane == 0) st32<SYS>(C.pws + WS_DONE + (size_t)(rank * 16 + T.ct) * 128, call);
  } else if (incoming) {
    early_store(C, E, f, 3, lane, 448.0f / __uint_as_float(mx));
    const unsigned* fl = (const unsigned*)(C.ws + WS_DONE + (size_t)f * 128);
    const u64 t1 = __builtin_amdgcn_s_memrealtime();
    while (ld32<SYS>(fl) < call) {
      if (give_up(tmo, call, t1, spins)) { if (lane == 0) *(GAS u8*)(size_t)(C.out + (size_t)rank * M * NS) = 0x7f; break; }
      __builtin_amdgcn_s_sleep(1);
    }
  }
}

extern "C" void kernel_launch(int rank, int nranks, ncclComm_t comm, hipStream_t stream,
                              void* const* d_in, const int* in_sizes, int n_in,
                              void* d_out, int out_size, const int* in_dtypes, int out_dtype,
                              void* const* d_out_all, void* const* d_ws_all, size_t ws_bytes) {
  (void)comm; (void)in_sizes; (void)n_in; (void)d_out; (void)out_size; (void)in_dtypes; (void)out_dtype;
  if (nranks != NR || ws_bytes < WS_END || (unsigned)rank >= (unsigned)NR) { fprintf(stderr, "unsupported configuration\n"); return; }
  static bool init = false;
  if (!init) {
    HIP_CHECK(hipFuncSetAttribute((const void*)gemm_a2a_q, hipFuncAttributeMaxDynamicSharedMemorySize, LDS_BYTES));
    init = true;
  }
  Params P;
  P.x = (const u16*)d_in[0]; P.w = (const u16*)d_in[1];
  for (int j = 0; j < NR; ++j) { P.out[j] = (u8*)d_out_all[(rank + j) % NR]; P.ws[j] = (u8*)d_ws_all[(rank + j) % NR]; }
  P.rank = rank; P.pad = 0;
  hipLaunchKernelGGL(gemm_a2a_q, dim3(NWG), dim3(512), LDS_BYTES, stream, P);
  HIP_CHECK(hipGetLastError());
}
```

```cpp
#include <hip/hip_runtime.h>
#include <rccl/rccl.h>
#include <cstdio>

#define HIP_CHECK(expr) do { hipError_t _e = (expr); if (_e != hipSuccess) { \
    fprintf(stderr, "HIP error %s at %s:%d\n", hipGetErrorString(_e), __FILE__, __LINE__); \
    return; } } while (0)

#define LAS __attribute__((address_space(3)))
#define GAS __attribute__((address_space(1)))
typedef short s16x4 __attribute__((ext_vector_type(4)));
typedef short s16x8 __attribute__((ext_vector_type(8)));
typedef __bf16 bf16x8 __attribute__((ext_vector_type(8)));
typedef float f32x4 __attribute__((ext_vector_type(4)));
typedef unsigned u32x4 __attribute__((ext_vector_type(4)));
typedef unsigned short u16;
typedef unsigned long long u64;
typedef unsigned char u8;

constexpr int NR = 8;
constexpr int M = 512, N = 16384, K = 4096;
constexpr int NS = N / NR;
constexpr int BM = 256, BN = 128, BK = 64, NT = K / BK;
constexpr int NWG = 256;
constexpr int XT_BYTES = BM * BK * 2;
constexpr int A_BYTES = 288 * BK * 2, B_BYTES = BK * BN * 2;
constexpr int B_BASE = 3 * A_BYTES;
constexpr int X_BASE = B_BASE + 3 * B_BYTES;
constexpr int LDS_BYTES = X_BASE + 256;
#define QT1 1550
#define QT2 3700
#define QA0 72
#define QA1 94
#define QA3 128
#define QB 134
#define ECUT 6150

constexpr size_t WS_CNT = 0;
constexpr size_t WS_TMO = WS_CNT + (size_t)NWG * 128;
constexpr size_t WS_G1 = WS_TMO + 256;
constexpr size_t WS_G2 = WS_G1 + 8 * 128 * 8;
constexpr size_t WS_DONE = WS_G2 + 2 * NR * 8 * 8;
constexpr size_t WS_XF = WS_DONE + (size_t)NR * 16 * 128;
constexpr size_t WS_EF = WS_XF + (size_t)NR * 16 * 128;
constexpr size_t WS_XC = WS_EF + (size_t)2 * NR * 64 * 128;
constexpr size_t WS_XS = 1u << 20;
constexpr size_t XS_STRIDE = (size_t)NT * XT_BYTES;
constexpr size_t WS_ES = WS_XS + (size_t)NR * XS_STRIDE;
constexpr size_t WS_END = WS_ES + (size_t)2 * NR * 64 * 16384;
static_assert(WS_XC + 8 * 128 <= WS_XS, "layout");
constexpr u64 DEADLINE_TICKS = 40000000ull;

struct Params {
  const u16* x; const u16* w;
  int rank; int pad;
  u8* ws[NR];
  u8* out[NR];
};
struct Ctx { const u16* x; const u16* w; u8* ws; u8* out; u8* pws; u8* pout; int rank; };
struct Tile { int src, dst, ct, grp; };

#define SB() __builtin_amdgcn_sched_barrier(0)
#define BAR() do { asm volatile("" ::: "memory"); __builtin_amdgcn_s_barrier(); asm volatile("" ::: "memory"); } while (0)
#define WAITV(n) asm volatile("s_waitcnt vmcnt(%0)" :: "n"(n) : "memory")
#define WAITL0() asm volatile("s_waitcnt lgkmcnt(0)" ::: "memory")
#define DI __device__ __forceinline__

#define SYS __HIP_MEMORY_SCOPE_SYSTEM
#define AGT __HIP_MEMORY_SCOPE_AGENT
template <int SC> DI unsigned ld32(const void* p) { return __hip_atomic_load((const GAS unsigned*)(size_t)p, __ATOMIC_RELAXED, SC); }
template <int SC> DI void st32(void* p, unsigned v) { __hip_atomic_store((GAS unsigned*)(size_t)p, v, __ATOMIC_RELAXED, SC); }
DI void st64(void* p, unsigned tag, unsigned v) { __hip_atomic_store((GAS u64*)(size_t)p, ((u64)tag << 32) | v, __ATOMIC_RELAXED, SYS); }
DI unsigned draw(void* p) { return __hip_atomic_fetch_add((GAS unsigned*)(size_t)p, 1u, __ATOMIC_RELAXED, AGT); }
DI bf16x8 ldA(unsigned addr, int off) { return *(LAS bf16x8*)(size_t)(addr + off); }
DI s16x4 ldB(unsigned addr, int off) { return __builtin_amdgcn_ds_read_tr16_b64_v4i16((LAS s16x4*)(size_t)(addr + off)); }
DI unsigned wave_max_u32(unsigned v) {
  v = max(v, (unsigned)__builtin_amdgcn_update_dpp(0, (int)v, 0x111, 0xf, 0xf, false));
  v = max(v, (unsigned)__builtin_amdgcn_update_dpp(0, (int)v, 0x112, 0xf, 0xf, false));
  v = max(v, (unsigned)__builtin_amdgcn_update_dpp(0, (int)v, 0x114, 0xf, 0xf, false));
  v = max(v, (unsigned)__builtin_amdgcn_update_dpp(0, (int)v, 0x118, 0xf, 0xf, false));
  const unsigned a = __builtin_amdgcn_readlane(v, 15), b = __builtin_amdgcn_readlane(v, 31);
  const unsigned c = __builtin_amdgcn_readlane(v, 47), d = __builtin_amdgcn_readlane(v, 63);
  return max(max(a, b), max(c, d));
}
DI bool give_up(unsigned* tmo, unsigned call, u64 t0, unsigned& spins) {
  if ((++spins & 7u) != 0u) return false;
  if (ld32<AGT>(tmo) == call || __builtin_amdgcn_s_memrealtime() - t0 > DEADLINE_TICKS) {
    st32<AGT>(tmo, call);
    return true;
  }
  return false;
}
DI unsigned pkbf(float a, float b) {
  typedef float f32x2 __attribute__((ext_vector_type(2)));
  typedef __bf16 bf16x2 __attribute__((ext_vector_type(2)));
  return __builtin_bit_cast(unsigned, __builtin_convertvector((f32x2){a, b}, bf16x2));
}
DI unsigned q4(float a, float b, float c, float d, float inv) {
  const int t = __builtin_amdgcn_cvt_pk_fp8_f32(a * inv, b * inv, 0, false);
  return (unsigned)__builtin_amdgcn_cvt_pk_fp8_f32(c * inv, d * inv, t, true);
}
DI void store16(u8* p, const u32x4 v) { asm volatile("global_store_dwordx4 %0, %1, off sc0 sc1\n\ts_nop 1" :: "v"(p), "v"(v) : "memory"); }
DI void store_row16(u8* p, unsigned w0, unsigned w1, unsigned w2, unsigned w3, bool odd) {
  store16(p, (u32x4){odd ? w1 : w0, odd ? w0 : w1, odd ? w3 : w2, odd ? w2 : w3});
}

DI unsigned q4b(unsigned lo, unsigned hi, float inv) {
  return q4(__uint_as_float(lo << 16), __uint_as_float(lo & 0xffff0000u), __uint_as_float(hi << 16), __uint_as_float(hi & 0xffff0000u), inv);
}
DI void store_bf16_row(u8* p, const u32x4 a, const u32x4 b, float inv, bool odd) {
  store_row16(p, q4b(a[0], a[1], inv), q4b(a[2], a[3], inv), q4b(b[0], b[1], inv), q4b(b[2], b[3], inv), odd);
}

template <bool TALL>
DI void loader(const Ctx& C, const Tile T, LAS char* lds, const int p, const int lane) {
  const bool remote = TALL && T.grp != 0;
  const void* base = remote ? (const void*)(C.ws + WS_XS + (size_t)T.src * XS_STRIDE) : (const void*)(C.x + (size_t)(TALL ? BM : 0) * K);
  __amdgpu_buffer_rsrc_t rA = __builtin_amdgcn_make_buffer_rsrc((void*)base, 0, BM * K * 2, 0x00020000);
  __amdgpu_buffer_rsrc_t rE = __builtin_amdgcn_make_buffer_rsrc((void*)(C.x + (size_t)T.grp * 32 * K), 0, 32 * K * 2, 0x00020000);
  __amdgpu_buffer_rsrc_t rB = __builtin_amdgcn_make_buffer_rsrc((void*)(C.w + (size_t)T.dst * NS + T.ct * BN), 0, (K - 1) * N * 2 + BN * 2, 0x00020000);
  const unsigned pstr = remote ? 1024u : 8u * K * 2, kstr = remote ? (unsigned)XT_BYTES : BK * 2u;
  const unsigned voL = (unsigned)(((lane >> 3) + 8 * p) * (K * 2) + (((lane & 7) ^ ((p << 2) | (lane >> 4))) << 4));
  const unsigned voA = remote ? (unsigned)lane * 16u + p * 1024u : voL;
  const unsigned voB = (unsigned)(((lane >> 4) + 8 * p) * (N * 2) + (((lane & 15) ^ ((lane >> 4) | (p << 3))) << 4));
  __amdgpu_buffer_rsrc_t rX = __builtin_amdgcn_make_buffer_rsrc((void*)(C.ws + WS_XS + (size_t)C.rank * XS_STRIDE), 0, BM * K * 2, 0x00020000);
  auto issueA = [&](int kt, int buf) {
    if (!TALL && kt > 8 && ((kt & 7) == 3 || kt == NT - 1)) {
#pragma unroll
      for (int j = 0; j < 16; ++j)
        __builtin_amdgcn_raw_ptr_buffer_load_lds(rX, (LAS void*)(lds + buf * A_BYTES + (p + 2 * j + 2 * (j >> 3)) * 1024), 16, (unsigned)lane * 16u + p * 1024u, kt * XT_BYTES + 2 * j * 1024, 0, 17);
      return;
    }
#pragma unroll
    for (int j = 0; j < 16; ++j)
      __builtin_amdgcn_raw_ptr_buffer_load_lds(rA, (LAS void*)(lds + buf * A_BYTES + (p + 2 * j + 2 * (j >> 3)) * 1024), 16, voA, kt * kstr + 2 * j * pstr, 0, TALL ? 17 : 0);
    if (TALL) {
#pragma unroll
      for (int j = 0; j < 2; ++j)
        __builtin_amdgcn_raw_ptr_buffer_load_lds(rE, (LAS void*)(lds + buf * A_BYTES + (16 + p + 18 * j) * 1024), 16, voL, kt * (BK * 2) + j * 16 * K * 2, 0, 0);
    }
  };
  auto issueB = [&](int kt, int buf) {
#pragma unroll
    for (int j = 0; j < 8; ++j) {
      const int pc = (j & 1) + 4 * (j >> 1);
      __builtin_amdgcn_raw_ptr_buffer_load_lds(rB, (LAS void*)(lds + B_BASE + buf * B_BYTES + (pc + 2 * p) * 1024), 16, voB, (kt * BK + pc * 4) * (N * 2), 0, TALL ? 0 : 2)
      ;
    }
  };
  constexpr int NA = TALL ? 18 : 16;
  if (remote) {
    __builtin_amdgcn_s_sleep(24);
    issueB(0, 0); issueB(1, 1);
    BAR();
    issueA(0, 0);
    BAR();
    issueA(1, 1);
    WAITV(NA);
  } else {
    issueB(0, 0); issueA(0, 0); issueB(1, 1); issueA(1, 1);
    WAITV(NA + 8);
  }
  BAR();
  int buf = 2;
  for (int kt = 0; kt < NT - 2; ++kt) {
    issueB(kt + 2, buf); issueA(kt + 2, buf);
    WAITV(NA + 8);
    buf = (buf == 2) ? 0 : buf + 1;
    BAR();
  }
  WAITV(0);
  BAR(); BAR();
}

template <int MB>
DI void consumer(const Ctx& C, const Tile T, LAS char* lds, const int wid, const int lane) {
  const u64 tb = __builtin_amdgcn_s_memrealtime();
  const unsigned lbase = (unsigned)(size_t)lds;
  LAS unsigned* xw = (LAS unsigned*)(lds + X_BASE);
  const int wr = wid >> 1, wc = wid & 1;
  const int g = lane >> 4, c = lane & 15, q = (lane & 15) >> 2, pp = lane & 3;
  unsigned aA[2], aB[4];
#pragma unroll
  for (int ks = 0; ks < 2; ++ks) aA[ks] = lbase + (wr * 144 + c) * 128 + (((4 * ks + g) ^ (c >> 1)) << 4);
#pragma unroll
  for (int b = 0; b < 4; ++b) {
    const int row = 8 * g + q, ch = 8 * wc + 2 * pp + (b >> 1);
    aB[b] = lbase + B_BASE + row * 256 + ((ch ^ ((row & 3) | (row & 8))) << 4) + (((b ^ pp) & 1) << 3);
  }
  f32x4 acc[MB][4];
  bf16x8 Af[MB][2];
  s16x4 Bq[2][4][2][2];
  auto rdA = [&](int m, int ks) { Af[m][ks] = ldA(aA[ks], m * 2048); SB(); };
  auto rdB = [&](int set, int i) { Bq[set][i >> 2][(i >> 1) & 1][i & 1] = ldB(aB[i >> 2], ((i >> 1) & 1) * 8192 + (i & 1) * 1024); SB(); };
  auto mma = [&](int set, int m, int i) {
    const int b = (m & 1) ? 3 - (i >> 1) : (i >> 1), ks = ((i >> 1) ^ i) & 1;
    s16x8 bv = __builtin_shufflevector(Bq[set][b][ks][0], Bq[set][b][ks][1], 0, 1, 2, 3, 4, 5, 6, 7);
    acc[m][b] = __builtin_amdgcn_mfma_f32_16x16x32_bf16(__builtin_bit_cast(bf16x8, bv), Af[m][ks], acc[m][b], 0, 0, 0);
    SB();
  };
  int buf = 0;
  auto ktile = [&](int set) {
    int da = 0, db = 0;
#pragma unroll
    for (int m = 0; m < MB - 4; ++m)
#pragma unroll
      for (int i = 0; i < 8; ++i) {
        mma(set, m, i);
        if (m == 0 && i < 6) rdA(2 + (i >> 1), i & 1);
        if (m > 0 && i < 2) rdA(m + 4, i);
        if (m == 0 && i == 6) { db = buf == 2 ? -2 * B_BYTES : B_BYTES; asm volatile("" : "+s"(db)); SB(); }
        if (m == 1 && i >= 3 && i < 7) { aB[i - 3] += db; SB(); }
        if (m == 2 && i == 3) { da = buf == 2 ? -2 * A_BYTES : A_BYTES; asm volatile("" : "+s"(da)); SB(); }
        if (m == 2 && i == 5) { buf = buf == 2 ? -1 : buf; asm volatile("" : "+s"(buf)); SB(); }
        if (m == 2 && i == 6) { ++buf; asm volatile("" : "+s"(buf)); SB(); }
        if (m == MB - 5 && (i == 3 || i == 4)) { aA[i - 3] += da; SB(); }
      }
    __builtin_amdgcn_s_waitcnt(0xc07f);
    SB();
    BAR();
    SB();
#pragma unroll
    for (int m = MB - 4; m < MB; ++m)
#pragma unroll
      for (int i = 0; i < 8; ++i) {
        const int j = (m - (MB - 4)) * 8 + i;
        mma(set, m, i);
        if (j < 16) rdB(set ^ 1, j);
        if (j >= 16 && j < 20) rdA((j - 16) >> 1, j & 1);
      }
  };
#pragma unroll
  for (int m = 0; m < MB; ++m)
#pragma unroll
    for (int b = 0; b < 4; ++b) { acc[m][b] = (f32x4){0.f, 0.f, 0.f, 0.f}; asm volatile("" : "+v"(acc[m][b])); }
  SB();
  const int id = ((T.grp - 1) * 16 + T.ct) * 3 + wid;
  if (MB == 9 && T.grp != 0 && wid < 3 && id < 256) {
    const int bi = id >> 5, pc = id & 31, kt = bi < 7 ? 11 + 8 * bi : NT - 1;
    u32x4 v;
    __builtin_amdgcn_s_sleep(24);
    asm volatile("global_load_dwordx4 %0, %1, off\n\ts_waitcnt vmcnt(0)" : "=&v"(v)
                 : "v"((const char*)(C.x + (size_t)(8 * pc + (lane >> 3)) * K + kt * BK) + (((lane & 7) ^ (((pc & 1) << 2) | (lane >> 4))) << 4)) : "memory");
    store16(C.ws + WS_XS + (size_t)C.rank * XS_STRIDE + kt * XT_BYTES + pc * 1024 + lane * 16, v);
    WAITV(0);
    if (lane == 0) __hip_atomic_fetch_add((GAS unsigned*)(size_t)(C.ws + WS_XC + bi * 128), 1u, __ATOMIC_RELAXED, SYS);
  }
  auto first = [&]() {
#pragma unroll
    for (int i = 0; i < 16; ++i) rdB(0, i);
#pragma unroll
    for (int i = 0; i < 4; ++i) rdA(i >> 1, i & 1);
  };
  {
    if (MB == 9 && T.grp != 0) { BAR(); BAR(); }
    BAR();
    first();
#pragma unroll 1
    for (int kt = 0; kt < NT; kt += 2) { ktile(0); ktile(1); }
  }
  float am = 0.f;
#pragma unroll
  for (int m = 0; m < MB; ++m)
#pragma unroll
    for (int b = 0; b < 4; ++b) {
      am = __builtin_fmaxf(__builtin_fmaxf(am, __builtin_fabsf(acc[m][b][0])), __builtin_fabsf(acc[m][b][1]));
      am = __builtin_fmaxf(__builtin_fmaxf(am, __builtin_fabsf(acc[m][b][2])), __builtin_fabsf(acc[m][b][3]));
    }
  const unsigned mu = wave_max_u32(__float_as_uint(am));
  const unsigned call = xw[1];
  if (lane == 0)
    st64((u64*)(C.ws + WS_G1) + (blockIdx.x & 7) * 128 + (blockIdx.x >> 3) * 4 + wid, call, mu);
  u8* ob = C.pout + T.ct * BN + wc * 64 + 16 * g;
  u8* op = ob + (size_t)(T.src * M + (MB == 9 ? BM : 0) + wr * 128 + c) * NS;
  if (MB == 8) {
    u32x4 pk[8][2];
#pragma unroll
    for (int m = 0; m < 8; ++m)
#pragma unroll
      for (int h = 0; h < 2; ++h)
        pk[m][h] = (u32x4){pkbf(acc[m][2 * h][0], acc[m][2 * h][1]), pkbf(acc[m][2 * h][2], acc[m][2 * h][3]),
                           pkbf(acc[m][2 * h + 1][0], acc[m][2 * h + 1][1]), pkbf(acc[m][2 * h + 1][2], acc[m][2 * h + 1][3])};
    const int slot = (call & 1) * (NR * 64) + (C.rank * 16 + T.ct) * 4 + wid;
    u8* ep = C.pws + WS_ES + (size_t)slot * 16384 + lane * 16;
    int ne = 0;
#pragma unroll
    for (int m = 0; m < 8; ++m)
      if (ne == m && (int)(__builtin_amdgcn_s_memrealtime() - tb) < ECUT &&
          __builtin_amdgcn_readfirstlane(__hip_atomic_load(&xw[2], __ATOMIC_RELAXED, __HIP_MEMORY_SCOPE_WORKGROUP)) == 0) {
#pragma unroll
        for (int h = 0; h < 2; ++h) {
          store16(ep + m * 2048 + h * 1024, pk[m][h]);
          WAITV(1);
          __builtin_amdgcn_s_sleep(8);
        }
        ne = m + 1;
      }
    WAITV(0);
    if (lane == 0) st32<SYS>(C.pws + WS_EF + (size_t)slot * 128, call * 16u + ne);
    BAR();
    const float inv = 448.0f / __uint_as_float(xw[0]);
#pragma unroll
    for (int m = 7; m >= 0; --m)
      if (m >= ne) { store_bf16_row(op + (size_t)m * 16 * NS, pk[m][0], pk[m][1], inv, g & 1); SB(); }
  } else {
    u8* oe = ob + (size_t)(C.rank * M + T.grp * 32 + wr * 16 + c) * NS;
    BAR();
    const float inv = 448.0f / __uint_as_float(xw[0]);
#pragma unroll
    for (int m = 0; m < MB; ++m) {
      unsigned w[4];
#pragma unroll
      for (int b = 0; b < 4; ++b) w[b] = q4(acc[m][b][0], acc[m][b][1], acc[m][b][2], acc[m][b][3], inv);
      store_row16(m < 8 ? op + (size_t)m * 16 * NS : oe, w[0], w[1], w[2], w[3], g & 1);
      SB();
    }
  }
  if (MB == 9) asm volatile(".Lgaq_m9:\n\ts_nop 0");
  if (MB == 8) {
    asm volatile(".Lgaq_m8:");
    WAITV(0);
    BAR();
  }
}

struct Early { u32x4 v[16]; int ne; };
#define LD8(o, p) asm volatile("global_load_dwordx4 %0, %8, off sc0 sc1\n\tglobal_load_dwordx4 %1, %8, off offset:1024 sc0 sc1\n\t" \
               "global_load_dwordx4 %2, %8, off offset:2048 sc0 sc1\n\tglobal_load_dwordx4 %3, %8, off offset:3072 sc0 sc1\n\t" \
               "global_load_dwordx4 %4, %9, off sc0 sc1\n\tglobal_load_dwordx4 %5, %9, off offset:1024 sc0 sc1\n\t" \
               "global_load_dwordx4 %6, %9, off offset:2048 sc0 sc1\n\tglobal_load_dwordx4 %7, %9, off offset:3072 sc0 sc1\n\ts_waitcnt vmcnt(0)" \
               : "=&v"(E.v[o]), "=&v"(E.v[o + 1]), "=&v"(E.v[o + 2]), "=&v"(E.v[o + 3]), "=&v"(E.v[o + 4]), "=&v"(E.v[o + 5]), "=&v"(E.v[o + 6]), "=&v"(E.v[o + 7]) \
               : "v"(p), "v"((p) + 4096) : "memory")
DI void early_fetch(const Ctx& C, Early& E, const int f, const int w, const int lane, const unsigned call) {
  const int slot = (call & 1) * (NR * 64) + f * 4 + w;
  const unsigned* fl = (const unsigned*)(C.ws + WS_EF + (size_t)slot * 128);
  unsigned spins = 0;
  const u64 t0 = __builtin_amdgcn_s_memrealtime();
  unsigned v;
  while ((v = (unsigned)__builtin_amdgcn_readfirstlane((int)ld32<SYS>(fl))) < call * 16u) {
    if (give_up((unsigned*)(C.ws + WS_TMO), call, t0, spins)) { v = call * 16u + 8u; break; }
    __builtin_amdgcn_s_sleep(2);
  }
  E.ne = (int)(v - call * 16u);
  const u8* ep = C.ws + WS_ES + (size_t)slot * 16384 + lane * 16;
  if (E.ne > 0) LD8(0, ep);
  if (E.ne > 4) LD8(8, ep + 8192);
}
DI void early_store(const Ctx& C, const Early& E, const int f, const int w, const int lane, const float inv) {
  u8* op = C.out + (size_t)((f >> 4) * M + (w >> 1) * 128 + (lane & 15)) * NS + (f & 15) * BN + (w & 1) * 64 + (lane & 48);
#pragma unroll
  for (int m = 0; m < 8; ++m)
    if (m < E.ne) store_bf16_row(op + (size_t)m * 16 * NS, E.v[2 * m], E.v[2 * m + 1], inv, (lane >> 4) & 1);
}

__global__ __launch_bounds__(512, 2) void gemm_a2a_q(const Params P) {
  extern __shared__ __attribute__((aligned(16))) char smem[];
  LAS char* lds = (LAS char*)smem;
  const int tid = threadIdx.x, lane = tid & 63;
  const int wid = __builtin_amdgcn_readfirstlane(tid >> 6);
  const int bid = blockIdx.x;
  const int xcd = bid & 7, idx = bid >> 3;
  unsigned cjunk = 0;
  if (idx == 31) {
    u64 pc; int len;
    asm volatile("s_getpc_b64 %0\n.Lgaq_pc:\n\ts_mov_b32 %1, .Lgaq_m0-.Lgaq_pc\n\ts_max_i32 %1, %1, .Lgaq_m8-.Lgaq_pc\n\t"
                 "s_max_i32 %1, %1, .Lgaq_m9-.Lgaq_pc\n\ts_max_i32 %1, %1, 0" : "=s"(pc), "=s"(len) :: "scc");
    asm volatile("global_load_dword %0, %1, off" : "+v"(cjunk) : "v"(pc + (unsigned)min(tid * 128, len)) : "memory");
  }
  const int pj = (idx < 2 ? wid + 1 : idx < 16 ? 1 + (idx - 2) % 7 : 0) & 7;
  Ctx C = {P.x, P.w, P.ws[0], P.out[0], P.ws[pj], P.out[pj], P.rank};
  asm volatile("" : "+s"(C.x), "+s"(C.w), "+s"(C.ws), "+s"(C.out), "+s"(C.pws), "+s"(C.pout), "+s"(C.rank));
  if (idx == 31) asm volatile("s_waitcnt vmcnt(0)" :: "v"(cjunk) : "memory");
  if (idx >= 16 && wid == 3) {
    const int u = idx - 16;
    const u8* tp = u >= 14 ? (const u8*)C.x : P.ws[1 + u % 7] + (u < 7 ? WS_XS + (size_t)C.rank * XS_STRIDE : (size_t)WS_XF);
    unsigned junk;
    asm volatile("global_load_dword %0, %1, off sc0 sc1\n\tglobal_load_dword %0, %2, off nt\n\ts_waitcnt vmcnt(0)" : "=&v"(junk)
                 : "v"(tp), "v"((const u8*)C.w + ((size_t)(u + 16 * (lane & 3)) << 21)) : "memory");
  }
  const int rank = C.rank;
  unsigned* const tmo = (unsigned*)(C.ws + WS_TMO);
  unsigned* const cnt = (unsigned*)(C.ws + WS_CNT + (size_t)bid * 128);
  LAS unsigned* xw = (LAS unsigned*)(lds + X_BASE);

  if (idx < 2) {
    const int cw = xcd * 2 + idx;
    if (wid < 7) {
      const char* sp = (const char*)(C.x + (size_t)(BM + 16 * cw + (lane >> 3)) * K);
      const char* sp0 = sp + (((lane & 7) ^ (lane >> 4)) << 4);
      const char* sp1 = sp + 8 * K * 2 + (((lane & 7) ^ (4 | (lane >> 4))) << 4);
      u8* dp = C.pws + WS_XS + (size_t)rank * XS_STRIDE + cw * 2048 + lane * 16;
      unsigned* fp = (unsigned*)(C.pws + WS_XF + (rank * 16 + cw) * 128);
      u32x4 r0[8], r1[8];
#pragma unroll
      for (int s = 0; s < 8; ++s) { r0[s] = (u32x4){0u, 0u, 0u, 0u}; r1[s] = (u32x4){0u, 0u, 0u, 0u}; }
#define CLOAD(s) asm volatile("global_load_dwordx4 %0, %2, off offset:%4\n\tglobal_load_dwordx4 %1, %3, off offset:%4" \
                              : "+v"(r0[s]), "+v"(r1[s]) : "v"(sp0), "v"(sp1), "n"((s) * 128) : "memory")
#define CSTORE(s) do { asm volatile("global_store_dwordx4 %0, %1, off sc0 sc1\n\tglobal_store_dwordx4 %0, %2, off offset:1024 sc0 sc1\n\ts_nop 1" \
                                    :: "v"(dp), "v"(r0[s]), "v"(r1[s]) : "memory"); dp += XT_BYTES; } while (0)
#define CFLAG() do { ++fv; asm volatile("s_mov_b64 exec, 1\n\tglobal_store_dword %0, %1, off sc0 sc1\n\ts_mov_b64 exec, -1" :: "v"(fp), "v"(fv) : "memory"); } while (0)
      CLOAD(0); CLOAD(1); CLOAD(2);
      WAITV(4); CSTORE(0);
      WAITV(4); CSTORE(1);
#pragma unroll
      for (int s = 3; s < 8; ++s) CLOAD(s);
      sp0 += 1024; sp1 += 1024;
      CLOAD(0); CLOAD(1);
      BAR();
      unsigned fv = (xw[1] - 1u) * NT;
      WAITV(16); CFLAG();
      CSTORE(2); CLOAD(2); WAITV(19); CFLAG(); BAR();
      WAITV(10);
#pragma unroll
      for (int t = 3; t < 8; ++t) { CSTORE(t); CLOAD(t); WAITV(7); CFLAG(); BAR(); }
      for (int k8 = 8; k8 < NT - 8; k8 += 8) {
        sp0 += 1024; sp1 += 1024;
#pragma unroll
        for (int i = 0; i < 8; ++i) { CSTORE(i); CLOAD(i); WAITV(7); CFLAG(); BAR(); }
      }
#pragma unroll
      for (int i = 0; i < 8; ++i) { CSTORE(i); WAITV(3); CFLAG(); BAR(); }
      WAITV(0);
      asm volatile(".Lgaq_m0:");
      CFLAG();
    } else {
      unsigned ticket = 0;
      if (lane == 0) ticket = draw(cnt);
      const unsigned call = (unsigned)__builtin_amdgcn_readfirstlane((int)ticket) + 1u;
      if (lane == 0) xw[1] = call;
      WAITL0();
      BAR();
      for (int k = 2; k < NT; ++k) BAR();
      if (idx == 0) {
        const u8* gp = C.ws + WS_G1 + xcd * 1024 + lane * 16;
        u64* dstg = (u64*)(P.ws[lane & 7] + WS_G2) + (call & 1) * (NR * 8) + rank * 8 + xcd;
        asm volatile("" : "+v"(dstg));
        unsigned spins = 0;
        const u64 t0 = __builtin_amdgcn_s_memrealtime();
        for (;;) {
          u32x4 a;
          asm volatile("global_load_dwordx4 %0, %1, off sc0 sc1\n\ts_waitcnt vmcnt(0)" : "=&v"(a) : "v"(gp) : "memory");
          if (__all(lane < 4 || (a[1] == call && a[3] == call))) {
            const unsigned mx = wave_max_u32(lane < 4 ? 0u : max(a[0], a[2]));
            if (lane < NR) st64(dstg, call, mx);
            break;
          }
          if (give_up(tmo, call, t0, spins)) break;
          __builtin_amdgcn_s_sleep(1);
        }
      }
    }
    return;
  }

  const bool tall = idx >= 16;
  Tile T;
  if (!tall) { T.src = rank; T.dst = (rank + pj) & 7; T.ct = xcd * 2 + (idx - 2) / 7; T.grp = 0; }
  else { const int u = idx - 16; T.grp = (xcd >> 2) * 4 + (u >> 2); T.src = (rank + T.grp) & 7; T.dst = rank; T.ct = (xcd & 3) * 4 + (u & 3); }

  if (wid < 4) {
    if (tall) consumer<9>(C, T, lds, wid, lane); else consumer<8>(C, T, lds, wid, lane);
    return;
  }
  const int f = bid - 128;
  const bool incoming = tall && (f >> 4) != rank;
  const bool remote = T.grp != 0;
  Early E;
  if (wid < 6) {
    if (tall) loader<true>(C, T, lds, wid - 4, lane); else loader<false>(C, T, lds, wid - 4, lane);
  } else if (wid == 6) {
    const u64 tb = __builtin_amdgcn_s_memrealtime();
    unsigned warm = 0;
    if (remote) {
      {
        const int l = lane & 7;
        const u8* tp = l < 2 ? C.ws + WS_XS + (size_t)T.src * XS_STRIDE + l * (XS_STRIDE - 4)
                     : l < 4 ? C.ws + WS_ES + (size_t)((l & 1) * (NR * 64) + f * 4) * 16384
                     : l < 6 ? C.out + (size_t)((l & 1) ? T.src * M + BM : (f >> 4) * M) * NS : C.out + (size_t)rank * M * NS;
        asm volatile("global_load_dword %0, %1, off sc0 sc1" : "+v"(warm) : "v"(tp) : "memory"); }
      __builtin_amdgcn_s_sleep(24);
#pragma unroll
      for (int i = 0; i < 5; ++i)
        asm volatile("global_load_dword %0, %1, off" : "+v"(warm) : "v"((const char*)C.x + (size_t)min((T.grp - 1) * 16 + T.ct + 112 * i, M - 1) * (K * 2) + lane * 128) : "memory");
    }
    if (remote) { BAR(); BAR(); }
    BAR();
    const int hold = tall ? (remote ? QB : 0) : !(T.ct & 1) ? QA0 : ((T.ct & 2) && T.ct != 15) ? QA3 : QA1;
    int tl = (int)(__builtin_amdgcn_s_memrealtime() - tb);
    for (int k = 1; k <= NT; ++k) {
      if (tl >= QT1 && tl < QT2) {
        tl += hold;
        while ((int)(__builtin_amdgcn_s_memrealtime() - tb) < tl) __builtin_amdgcn_s_sleep(1);
      }
      BAR();
      tl = max(tl, (int)(__builtin_amdgcn_s_memrealtime() - tb) - 40);
    }
    asm volatile("s_waitcnt vmcnt(0)" :: "v"(warm) : "memory");
  }
  if (wid < 7) {
    if (incoming) {
      const unsigned call = xw[1];
      early_fetch(C, E, f, wid - 4, lane, call);
      const bool dead = ld32<AGT>(tmo) == call;
      BAR();
      early_store(C, E, f, wid - 4, lane, dead ? __uint_as_float(0x7fc00000u) : 448.0f / __uint_as_float(xw[0]));
    }
    return;
  }
  unsigned ticket = 0, spins = 0;
  if (lane == 0) ticket = draw(cnt);
  const unsigned call = (unsigned)__builtin_amdgcn_readfirstlane((int)ticket) + 1u;
  if (lane == 0) { xw[1] = call; xw[2] = 0u; }
  WAITL0();
  if (remote) {
    const unsigned* fp = (const unsigned*)(C.ws + WS_XF + (T.src * 16 + (lane & 15)) * 128);
    const unsigned base = (call - 1u) * NT;
    unsigned ready = 0;
    for (int k = -3; k < NT; ++k) {
      const unsigned need = (unsigned)min(k + 4, NT);
      if (ready < (k < 0 ? need : min(need + 2u, (unsigned)NT))) {
        const u64 t0 = __builtin_amdgcn_s_memrealtime();
        for (;;) {
          const unsigned d = ld32<SYS>(fp) - base;
          ready = NT - wave_max_u32(d > (unsigned)NT ? (unsigned)NT : NT - d);
          if (ready >= need) break;
          if (give_up(tmo, call, t0, spins)) { ready = NT; break; }
          __builtin_amdgcn_s_sleep(1);
        }
      }
      BAR();
    }
  } else {
    const u8* cp = C.ws + WS_XC + (lane & 7) * 128;
    unsigned cv = 0;
    for (int k = -1; k < 5; ++k) BAR();
    if (!tall) asm volatile("global_load_dword %0, %1, off sc0 sc1" : "+v"(cv) : "v"(cp) : "memory");
    BAR(); BAR();
    if (!tall) {
      asm volatile("s_waitcnt vmcnt(0)" : "+v"(cv) :: "memory");
      const u64 t0 = __builtin_amdgcn_s_memrealtime();
      while (!__all(cv - call * 32u < 0x80000000u)) {
        if (give_up(tmo, call, t0, spins)) break;
        __builtin_amdgcn_s_sleep(1);
        cv = ld32<SYS>(cp);
      }
    }
    for (int k = 7; k < NT; ++k) BAR();
  }
  if (incoming) early_fetch(C, E, f, 3, lane, call);
  if (!tall) {
    unsigned junk;
    asm volatile("global_load_dword %0, %1, off sc0 sc1\n\ts_waitcnt vmcnt(0)" : "=&v"(junk) : "v"(C.pout + (size_t)(rank * M + (lane & 1) * (BM - 1)) * NS + T.ct * BN) : "memory");
  }
  unsigned mx;
  {
    const u64* gp = (const u64*)(C.ws + WS_G2) + (call & 1) * (NR * 8) + lane;
    const u64 t0 = __builtin_amdgcn_s_memrealtime();
    for (;;) {
      u64 a; unsigned tm;
      asm volatile("global_load_dwordx2 %0, %2, off sc0 sc1\n\tglobal_load_dword %1, %3, off sc1\n\ts_waitcnt vmcnt(0)"
                   : "=&v"(a), "=&v"(tm) : "v"(gp), "v"(tmo) : "memory");
      if (tm == call) { mx = 0x7fc00000u; break; }
      if (__all((unsigned)(a >> 32) == call)) { mx = wave_max_u32((unsigned)a); break; }
      if (__builtin_amdgcn_s_memrealtime() - t0 > DEADLINE_TICKS) st32<AGT>(tmo, call);
      __builtin_amdgcn_s_sleep(2);
    }
    if (lane == 0) { xw[0] = mx; __hip_atomic_store(&xw[2], 1u, __ATOMIC_RELAXED, __HIP_MEMORY_SCOPE_WORKGROUP); }
    WAITL0();
  }
  BAR();
  if (!tall) {
    BAR();
    if (lane == 0) st32<SYS>(C.pws + WS_DONE + (size_t)(rank * 16 + T.ct) * 128, call);
  } else if (incoming) {
    early_store(C, E, f, 3, lane, 448.0f / __uint_as_float(mx));
    const unsigned* fl = (const unsigned*)(C.ws + WS_DONE + (size_t)f * 128);
    const u64 t1 = __builtin_amdgcn_s_memrealtime();
    while (ld32<SYS>(fl) < call) {
      if (give_up(tmo, call, t1, spins)) { if (lane == 0) *(GAS u8*)(size_t)(C.out + (size_t)rank * M * NS) = 0x7f; break; }
      __builtin_amdgcn_s_sleep(1);
    }
  }
}

extern "C" void kernel_launch(int rank, int nranks, ncclComm_t comm, hipStream_t stream,
                              void* const* d_in, const int* in_sizes, int n_in,
                              void* d_out, int out_size, const int* in_dtypes, int out_dtype,
                              void* const* d_out_all, void* const* d_ws_all, size_t ws_bytes) {
  (void)comm; (void)in_sizes; (void)n_in; (void)d_out; (void)out_size; (void)in_dtypes; (void)out_dtype;
  if (nranks != NR || ws_bytes < WS_END || (unsigned)rank >= (unsigned)NR) { fprintf(stderr, "unsupported configuration\n"); return; }
  static bool init = false;
  if (!init) {
    HIP_CHECK(hipFuncSetAttribute((const void*)gemm_a2a_q, hipFuncAttributeMaxDynamicSharedMemorySize, LDS_BYTES));
    init = true;
  }
  Params P;
  P.x = (const u16*)d_in[0]; P.w = (const u16*)d_in[1];
  for (int j = 0; j < NR; ++j) { P.out[j] = (u8*)d_out_all[(rank + j) % NR]; P.ws[j] = (u8*)d_ws_all[(rank + j) % NR]; }
  P.rank = rank; P.pad = 0;
  hipLaunchKernelGGL(gemm_a2a_q, dim3(NWG), dim3(512), LDS_BYTES, stream, P);
  HIP_CHECK(hipGetLastError());
}
```

```cpp
#include <hip/hip_runtime.h>
#include <rccl/rccl.h>
#include <cstdio>

#define HIP_CHECK(expr) do { hipError_t _e = (expr); if (_e != hipSuccess) { \
    fprintf(stderr, "HIP error %s at %s:%d\n", hipGetErrorString(_e), __FILE__, __LINE__); \
    return; } } while (0)

#define LAS __attribute__((address_space(3)))
#define GAS __attribute__((address_space(1)))
typedef short s16x4 __attribute__((ext_vector_type(4)));
typedef short s16x8 __attribute__((ext_vector_type(8)));
typedef __bf16 bf16x8 __attribute__((ext_vector_type(8)));
typedef float f32x4 __attribute__((ext_vector_type(4)));
typedef unsigned u32x4 __attribute__((ext_vector_type(4)));
typedef unsigned short u16;
typedef unsigned long long u64;
typedef unsigned char u8;

constexpr int NR = 8;
constexpr int M = 512, N = 16384, K = 4096;
constexpr int NS = N / NR;
constexpr int BM = 256, BN = 128, BK = 64, NT = K / BK;
constexpr int NWG = 256;
constexpr int XT_BYTES = BM * BK * 2;
constexpr int A_BYTES = 288 * BK * 2, B_BYTES = BK * BN * 2;
constexpr int B_BASE = 3 * A_BYTES;
constexpr int X_BASE = B_BASE + 3 * B_BYTES;
constexpr int LDS_BYTES = X_BASE + 256;
constexpr int QT1 = 1570, QT2 = 3670, QA0 = 71, QA1 = 92, QA3 = 126, QB = 131, ECUT = 6100;

constexpr size_t WS_CNT = 0;
constexpr size_t WS_TMO = WS_CNT + (size_t)NWG * 128;
constexpr size_t WS_G1 = WS_TMO + 256;
constexpr size_t WS_G2 = WS_G1 + 8 * 128 * 8;
constexpr size_t WS_DONE = WS_G2 + 2 * NR * 8 * 8;
constexpr size_t WS_XF = WS_DONE + (size_t)NR * 16 * 128;
constexpr size_t WS_EF = WS_XF + (size_t)NR * 16 * 128;
constexpr size_t WS_XC = WS_EF + (size_t)2 * NR * 64 * 128;
constexpr size_t WS_XS = 1u << 20;
constexpr size_t XS_STRIDE = (size_t)NT * XT_BYTES;
constexpr size_t WS_ES = WS_XS + (size_t)NR * XS_STRIDE;
constexpr size_t WS_END = WS_ES + (size_t)2 * NR * 64 * 16384;
static_assert(WS_XC + 8 * 128 <= WS_XS, "layout");
constexpr u64 DEADLINE_TICKS = 40000000ull;

struct Params {
  const u16* x; const u16* w;
  int rank; int pad;
  u8* ws[NR];
  u8* out[NR];
};
struct Ctx { const u16* x; const u16* w; u8* ws; u8* out; u8* pws; u8* pout; int rank; };
struct Tile { int src, dst, ct, grp; };

#define SB() __builtin_amdgcn_sched_barrier(0)
#define BAR() do { asm volatile("" ::: "memory"); __builtin_amdgcn_s_barrier(); asm volatile("" ::: "memory"); } while (0)
#define WAITV(n) asm volatile("s_waitcnt vmcnt(%0)" :: "n"(n) : "memory")
#define WAITL0() asm volatile("s_waitcnt lgkmcnt(0)" ::: "memory")
#define DI __device__ __forceinline__

#define SYS __HIP_MEMORY_SCOPE_SYSTEM
#define AGT __HIP_MEMORY_SCOPE_AGENT
template <int SC> DI unsigned ld32(const void* p) { return __hip_atomic_load((const GAS unsigned*)(size_t)p, __ATOMIC_RELAXED, SC); }
template <int SC> DI void st32(void* p, unsigned v) { __hip_atomic_store((GAS unsigned*)(size_t)p, v, __ATOMIC_RELAXED, SC); }
DI void st64(void* p, unsigned tag, unsigned v) { __hip_atomic_store((GAS u64*)(size_t)p, ((u64)tag << 32) | v, __ATOMIC_RELAXED, SYS); }
DI unsigned draw(void* p) { return __hip_atomic_fetch_add((GAS unsigned*)(size_t)p, 1u, __ATOMIC_RELAXED, AGT); }
DI bf16x8 ldA(unsigned addr, int off) { return *(LAS bf16x8*)(size_t)(addr + off); }
DI s16x4 ldB(unsigned addr, int off) { return __builtin_amdgcn_ds_read_tr16_b64_v4i16((LAS s16x4*)(size_t)(addr + off)); }
DI unsigned wave_max_u32(unsigned v) {
  v = max(v, (unsigned)__builtin_amdgcn_update_dpp(0, (int)v, 0x111, 0xf, 0xf, false));
  v = max(v, (unsigned)__builtin_amdgcn_update_dpp(0, (int)v, 0x112, 0xf, 0xf, false));
  v = max(v, (unsigned)__builtin_amdgcn_update_dpp(0, (int)v, 0x114, 0xf, 0xf, false));
  v = max(v, (unsigned)__builtin_amdgcn_update_dpp(0, (int)v, 0x118, 0xf, 0xf, false));
  const unsigned a = __builtin_amdgcn_readlane(v, 15), b = __builtin_amdgcn_readlane(v, 31);
  const unsigned c = __builtin_amdgcn_readlane(v, 47), d = __builtin_amdgcn_readlane(v, 63);
  return max(max(a, b), max(c, d));
}
DI bool give_up(unsigned* tmo, unsigned call, u64 t0, unsigned& spins) {
  if ((++spins & 7u) != 0u) return false;
  if (ld32<AGT>(tmo) == call || __builtin_amdgcn_s_memrealtime() - t0 > DEADLINE_TICKS) {
    st32<AGT>(tmo, call);
    return true;
  }
  return false;
}
DI unsigned pkbf(float a, float b) {
  typedef float f32x2 __attribute__((ext_vector_type(2)));
  typedef __bf16 bf16x2 __attribute__((ext_vector_type(2)));
  return __builtin_bit_cast(unsigned, __builtin_convertvector((f32x2){a, b}, bf16x2));
}
DI unsigned q4(float a, float b, float c, float d, float inv) {
  const int t = __builtin_amdgcn_cvt_pk_fp8_f32(a * inv, b * inv, 0, false);
  return (unsigned)__builtin_amdgcn_cvt_pk_fp8_f32(c * inv, d * inv, t, true);
}
DI void store16(u8* p, const u32x4 v) { asm volatile("global_store_dwordx4 %0, %1, off sc0 sc1\n\ts_nop 1" :: "v"(p), "v"(v) : "memory"); }
DI void store_row16(u8* p, unsigned w0, unsigned w1, unsigned w2, unsigned w3, bool odd) {
  store16(p, (u32x4){odd ? w1 : w0, odd ? w0 : w1, odd ? w3 : w2, odd ? w2 : w3});
}

DI unsigned q4b(unsigned lo, unsigned hi, float inv) {
  return q4(__uint_as_float(lo << 16), __uint_as_float(lo & 0xffff0000u), __uint_as_float(hi << 16), __uint_as_float(hi & 0xffff0000u), inv);
}
DI void store_bf16_row(u8* p, const u32x4 a, const u32x4 b, float inv, bool odd) {
  store_row16(p, q4b(a[0], a[1], inv), q4b(a[2], a[3], inv), q4b(b[0], b[1], inv), q4b(b[2], b[3], inv), odd);
}

template <bool TALL>
DI void loader(const Ctx& C, const Tile T, LAS char* lds, const int p, const int lane) {
  const bool remote = TALL && T.grp != 0;
  const void* base = remote ? (const void*)(C.ws + WS_XS + (size_t)T.src * XS_STRIDE) : (const void*)(C.x + (size_t)(TALL ? BM : 0) * K);
  __amdgpu_buffer_rsrc_t rA = __builtin_amdgcn_make_buffer_rsrc((void*)base, 0, BM * K * 2, 0x00020000);
  __amdgpu_buffer_rsrc_t rE = __builtin_amdgcn_make_buffer_rsrc((void*)(C.x + (size_t)T.grp * 32 * K), 0, 32 * K * 2, 0x00020000);
  __amdgpu_buffer_rsrc_t rB = __builtin_amdgcn_make_buffer_rsrc((void*)(C.w + (size_t)T.dst * NS + T.ct * BN), 0, (K - 1) * N * 2 + BN * 2, 0x00020000);
  const unsigned pstr = remote ? 1024u : 8u * K * 2, kstr = remote ? (unsigned)XT_BYTES : BK * 2u;
  const unsigned voL = (unsigned)(((lane >> 3) + 8 * p) * (K * 2) + (((lane & 7) ^ ((p << 2) | (lane >> 4))) << 4));
  const unsigned voA = remote ? (unsigned)lane * 16u + p * 1024u : voL;
  const unsigned voB = (unsigned)(((lane >> 4) + 8 * p) * (N * 2) + (((lane & 15) ^ ((lane >> 4) | (p << 3))) << 4));
  __amdgpu_buffer_rsrc_t rX = __builtin_amdgcn_make_buffer_rsrc((void*)(C.ws + WS_XS + (size_t)C.rank * XS_STRIDE), 0, BM * K * 2, 0x00020000);
  auto issueA = [&](int kt, int buf) {
    if (!TALL && kt > 8 && ((kt & 7) == 3 || kt == NT - 1)) {
#pragma unroll
      for (int j = 0; j < 16; ++j)
        __builtin_amdgcn_raw_ptr_buffer_load_lds(rX, (LAS void*)(lds + buf * A_BYTES + (p + 2 * j + 2 * (j >> 3)) * 1024), 16, (unsigned)lane * 16u + p * 1024u, kt * XT_BYTES + 2 * j * 1024, 0, 17);
      return;
    }
#pragma unroll
    for (int j = 0; j < 16; ++j)
      __builtin_amdgcn_raw_ptr_buffer_load_lds(rA, (LAS void*)(lds + buf * A_BYTES + (p + 2 * j + 2 * (j >> 3)) * 1024), 16, voA, kt * kstr + 2 * j * pstr, 0, TALL ? 17 : 0);
    if (TALL) {
#pragma unroll
      for (int j = 0; j < 2; ++j)
        __builtin_amdgcn_raw_ptr_buffer_load_lds(rE, (LAS void*)(lds + buf * A_BYTES + (16 + p + 18 * j) * 1024), 16, voL, kt * (BK * 2) + j * 16 * K * 2, 0, 0);
    }
  };
  auto issueB = [&](int kt, int buf) {
#pragma unroll
    for (int j = 0; j < 8; ++j) {
      const int pc = (j & 1) + 4 * (j >> 1);
      __builtin_amdgcn_raw_ptr_buffer_load_lds(rB, (LAS void*)(lds + B_BASE + buf * B_BYTES + (pc + 2 * p) * 1024), 16, voB, (kt * BK + pc * 4) * (N * 2), 0, TALL ? 0 : 2);
    }
  };
  constexpr int NA = TALL ? 18 : 16;
  if (remote) {
    __builtin_amdgcn_s_sleep(24);
    issueB(0, 0); issueB(1, 1);
    BAR();
    issueA(0, 0);
    BAR();
    issueA(1, 1);
    WAITV(NA);
  } else {
    issueB(0, 0); issueA(0, 0); issueB(1, 1); issueA(1, 1);
    WAITV(NA + 8);
  }
  BAR();
  int buf = 2;
  for (int kt = 0; kt < NT - 2; ++kt) {
    issueB(kt + 2, buf); issueA(kt + 2, buf);
    WAITV(NA + 8);
    buf = (buf == 2) ? 0 : buf + 1;
    BAR();
  }
  WAITV(0);
  BAR(); BAR();
}

template <int MB>
DI void consumer(const Ctx& C, const Tile T, LAS char* lds, const int wid, const int lane) {
  const u64 tb = __builtin_amdgcn_s_memrealtime();
  const unsigned lbase = (unsigned)(size_t)lds;
  LAS unsigned* xw = (LAS unsigned*)(lds + X_BASE);
  const int wr = wid >> 1, wc = wid & 1;
  const int g = lane >> 4, c = lane & 15, q = (lane & 15) >> 2, pp = lane & 3;
  unsigned aA[2], aB[4];
#pragma unroll
  for (int ks = 0; ks < 2; ++ks) aA[ks] = lbase + (wr * 144 + c) * 128 + (((4 * ks + g) ^ (c >> 1)) << 4);
#pragma unroll
  for (int b = 0; b < 4; ++b) {
    const int row = 8 * g + q, ch = 8 * wc + 2 * pp + (b >> 1);
    aB[b] = lbase + B_BASE + row * 256 + ((ch ^ ((row & 3) | (row & 8))) << 4) + (((b ^ pp) & 1) << 3);
  }
  f32x4 acc[MB][4];
  bf16x8 Af[MB][2];
  s16x4 Bq[2][4][2][2];
  auto rdA = [&](int m, int ks) { Af[m][ks] = ldA(aA[ks], m * 2048); SB(); };
  auto rdB = [&](int set, int i) { Bq[set][i >> 2][(i >> 1) & 1][i & 1] = ldB(aB[i >> 2], ((i >> 1) & 1) * 8192 + (i & 1) * 1024); SB(); };
  auto mma = [&](int set, int m, int i) {
    const int b = (m & 1) ? 3 - (i >> 1) : (i >> 1), ks = ((i >> 1) ^ i) & 1;
    s16x8 bv = __builtin_shufflevector(Bq[set][b][ks][0], Bq[set][b][ks][1], 0, 1, 2, 3, 4, 5, 6, 7);
    acc[m][b] = __builtin_amdgcn_mfma_f32_16x16x32_bf16(__builtin_bit_cast(bf16x8, bv), Af[m][ks], acc[m][b], 0, 0, 0);
    SB();
  };
  int buf = 0;
  auto ktile = [&](int set) {
    int da = 0, db = 0;
#pragma unroll
    for (int m = 0; m < MB - 4; ++m)
#pragma unroll
      for (int i = 0; i < 8; ++i) {
        mma(set, m, i);
        if (m == 0 && i < 6) rdA(2 + (i >> 1), i & 1);
        if (m > 0 && i < 2) rdA(m + 4, i);
        if (m == 0 && i == 6) { db = buf == 2 ? -2 * B_BYTES : B_BYTES; asm volatile("" : "+s"(db)); SB(); }
        if (m == 1 && i >= 3 && i < 7) { aB[i - 3] += db; SB(); }
        if (m == 2 && i == 3) { da = buf == 2 ? -2 * A_BYTES : A_BYTES; asm volatile("" : "+s"(da)); SB(); }
        if (m == 2 && i == 5) { buf = buf == 2 ? -1 : buf; asm volatile("" : "+s"(buf)); SB(); }
        if (m == 2 && i == 6) { ++buf; asm volatile("" : "+s"(buf)); SB(); }
        if (m == MB - 5 && (i == 3 || i == 4)) { aA[i - 3] += da; SB(); }
      }
    __builtin_amdgcn_s_waitcnt(0xc07f);
    SB();
    BAR();
    SB();
#pragma unroll
    for (int m = MB - 4; m < MB; ++m)
#pragma unroll
      for (int i = 0; i < 8; ++i) {
        const int j = (m - (MB - 4)) * 8 + i;
        mma(set, m, i);
        if (j < 16) rdB(set ^ 1, j);
        if (j >= 16 && j < 20) rdA((j - 16) >> 1, j & 1);
      }
  };
#pragma unroll
  for (int m = 0; m < MB; ++m)
#pragma unroll
    for (int b = 0; b < 4; ++b) { acc[m][b] = (f32x4){0.f, 0.f, 0.f, 0.f}; asm volatile("" : "+v"(acc[m][b])); }
  SB();
  const int id = ((T.grp - 1) * 16 + T.ct) * 3 + wid;
  if (MB == 9 && T.grp != 0 && wid < 3 && id < 256) {
    const int bi = id >> 5, pc = id & 31, kt = bi < 7 ? 11 + 8 * bi : NT - 1;
    u32x4 v;
    __builtin_amdgcn_s_sleep(24);
    asm volatile("global_load_dwordx4 %0, %1, off\n\ts_waitcnt vmcnt(0)" : "=&v"(v)
                 : "v"((const char*)(C.x + (size_t)(8 * pc + (lane >> 3)) * K + kt * BK) + (((lane & 7) ^ (((pc & 1) << 2) | (lane >> 4))) << 4)) : "memory");
    store16(C.ws + WS_XS + (size_t)C.rank * XS_STRIDE + kt * XT_BYTES + pc * 1024 + lane * 16, v);
    WAITV(0);
    if (lane == 0) __hip_atomic_fetch_add((GAS unsigned*)(size_t)(C.ws + WS_XC + bi * 128), 1u, __ATOMIC_RELAXED, SYS);
  }
  auto first = [&]() {
#pragma unroll
    for (int i = 0; i < 16; ++i) rdB(0, i);
#pragma unroll
    for (int i = 0; i < 4; ++i) rdA(i >> 1, i & 1);
  };
  {
    if (MB == 9 && T.grp != 0) { BAR(); BAR(); }
    BAR();
    first();
#pragma unroll 1
    for (int kt = 0; kt < NT; kt += 2) { ktile(0); ktile(1); }
  }
  float am = 0.f;
#pragma unroll
  for (int m = 0; m < MB; ++m)
#pragma unroll
    for (int b = 0; b < 4; ++b) {
      am = __builtin_fmaxf(__builtin_fmaxf(am, __builtin_fabsf(acc[m][b][0])), __builtin_fabsf(acc[m][b][1]));
      am = __builtin_fmaxf(__builtin_fmaxf(am, __builtin_fabsf(acc[m][b][2])), __builtin_fabsf(acc[m][b][3]));
    }
  const unsigned mu = wave_max_u32(__float_as_uint(am));
  const unsigned call = xw[1];
  if (lane == 0)
    st64((u64*)(C.ws + WS_G1) + (blockIdx.x & 7) * 128 + (blockIdx.x >> 3) * 4 + wid, call, mu);
  u8* ob = C.pout + T.ct * BN + wc * 64 + 16 * g;
  u8* op = ob + (size_t)(T.src * M + (MB == 9 ? BM : 0) + wr * 128 + c) * NS;
  if (MB == 8) {
    u32x4 pk[8][2];
#pragma unroll
    for (int m = 0; m < 8; ++m)
#pragma unroll
      for (int h = 0; h < 2; ++h)
        pk[m][h] = (u32x4){pkbf(acc[m][2 * h][0], acc[m][2 * h][1]), pkbf(acc[m][2 * h][2], acc[m][2 * h][3]),
                           pkbf(acc[m][2 * h + 1][0], acc[m][2 * h + 1][1]), pkbf(acc[m][2 * h + 1][2], acc[m][2 * h + 1][3])};
    const int slot = (call & 1) * (NR * 64) + (C.rank * 16 + T.ct) * 4 + wid;
    u8* ep = C.pws + WS_ES + (size_t)slot * 16384 + lane * 16;
    int ne = 0;
#pragma unroll
    for (int m = 0; m < 8; ++m)
      if (ne == m && (int)(__builtin_amdgcn_s_memrealtime() - tb) < ECUT &&
          __builtin_amdgcn_readfirstlane(__hip_atomic_load(&xw[2], __ATOMIC_RELAXED, __HIP_MEMORY_SCOPE_WORKGROUP)) == 0) {
#pragma unroll
        for (int h = 0; h < 2; ++h) {
          store16(ep + m * 2048 + h * 1024, pk[m][h]);
          WAITV(1);
          __builtin_amdgcn_s_sleep(8);
        }
        ne = m + 1;
      }
    WAITV(0);
    if (lane == 0) st32<SYS>(C.pws + WS_EF + (size_t)slot * 128, call * 16u + ne);
    BAR();
    const float inv = 448.0f / __uint_as_float(xw[0]);
#pragma unroll
    for (int m = 7; m >= 0; --m)
      if (m >= ne) { store_bf16_row(op + (size_t)m * 16 * NS, pk[m][0], pk[m][1], inv, g & 1); SB(); }
  } else {
    u8* oe = ob + (size_t)(C.rank * M + T.grp * 32 + wr * 16 + c) * NS;
    BAR();
    const float inv = 448.0f / __uint_as_float(xw[0]);
#pragma unroll
    for (int m = 0; m < MB; ++m) {
      unsigned w[4];
#pragma unroll
      for (int b = 0; b < 4; ++b) w[b] = q4(acc[m][b][0], acc[m][b][1], acc[m][b][2], acc[m][b][3], inv);
      store_row16(m < 8 ? op + (size_t)m * 16 * NS : oe, w[0], w[1], w[2], w[3], g & 1);
      SB();
    }
  }
  if (MB == 9) asm volatile(".Lgaq_m9:\n\ts_nop 0");
  if (MB == 8) {
    asm volatile(".Lgaq_m8:");
    WAITV(0);
    BAR();
  }
}

struct Early { u32x4 v[16]; int ne; };
#define LD8(o, p) asm volatile("global_load_dwordx4 %0, %8, off sc0 sc1\n\tglobal_load_dwordx4 %1, %8, off offset:1024 sc0 sc1\n\t" \
               "global_load_dwordx4 %2, %8, off offset:2048 sc0 sc1\n\tglobal_load_dwordx4 %3, %8, off offset:3072 sc0 sc1\n\t" \
               "global_load_dwordx4 %4, %9, off sc0 sc1\n\tglobal_load_dwordx4 %5, %9, off offset:1024 sc0 sc1\n\t" \
               "global_load_dwordx4 %6, %9, off offset:2048 sc0 sc1\n\tglobal_load_dwordx4 %7, %9, off offset:3072 sc0 sc1\n\ts_waitcnt vmcnt(0)" \
               : "=&v"(E.v[o]), "=&v"(E.v[o + 1]), "=&v"(E.v[o + 2]), "=&v"(E.v[o + 3]), "=&v"(E.v[o + 4]), "=&v"(E.v[o + 5]), "=&v"(E.v[o + 6]), "=&v"(E.v[o + 7]) \
               : "v"(p), "v"((p) + 4096) : "memory")
DI void early_fetch(const Ctx& C, Early& E, const int f, const int w, const int lane, const unsigned call) {
  const int slot = (call & 1) * (NR * 64) + f * 4 + w;
  const unsigned* fl = (const unsigned*)(C.ws + WS_EF + (size_t)slot * 128);
  unsigned spins = 0;
  const u64 t0 = __builtin_amdgcn_s_memrealtime();
  unsigned v;
  while ((v = (unsigned)__builtin_amdgcn_readfirstlane((int)ld32<SYS>(fl))) < call * 16u) {
    if (give_up((unsigned*)(C.ws + WS_TMO), call, t0, spins)) { v = call * 16u + 8u; break; }
    __builtin_amdgcn_s_sleep(2);
  }
  E.ne = (int)(v - call * 16u);
  const u8* ep = C.ws + WS_ES + (size_t)slot * 16384 + lane * 16;
  if (E.ne > 0) LD8(0, ep);
  if (E.ne > 4) LD8(8, ep + 8192);
}
DI void early_store(const Ctx& C, const Early& E, const int f, const int w, const int lane, const float inv) {
  u8* op = C.out + (size_t)((f >> 4) * M + (w >> 1) * 128 + (lane & 15)) * NS + (f & 15) * BN + (w & 1) * 64 + (lane & 48);
#pragma unroll
  for (int m = 0; m < 8; ++m)
    if (m < E.ne) store_bf16_row(op + (size_t)m * 16 * NS, E.v[2 * m], E.v[2 * m + 1], inv, (lane >> 4) & 1);
}

__global__ __launch_bounds__(512, 2) void gemm_a2a_q(const Params P) {
  extern __shared__ __attribute__((aligned(16))) char smem[];
  LAS char* lds = (LAS char*)smem;
  const int tid = threadIdx.x, lane = tid & 63;
  const int wid = __builtin_amdgcn_readfirstlane(tid >> 6);
  const int bid = blockIdx.x;
  const int xcd = bid & 7, idx = bid >> 3;
  unsigned cjunk = 0;
  if (idx == 31) {
    u64 pc; int len;
    asm volatile("s_getpc_b64 %0\n.Lgaq_pc:\n\ts_mov_b32 %1, .Lgaq_m0-.Lgaq_pc\n\ts_max_i32 %1, %1, .Lgaq_m8-.Lgaq_pc\n\t"
                 "s_max_i32 %1, %1, .Lgaq_m9-.Lgaq_pc\n\ts_max_i32 %1, %1, 0" : "=s"(pc), "=s"(len) :: "scc");
    asm volatile("global_load_dword %0, %1, off" : "+v"(cjunk) : "v"(pc + (unsigned)min(tid * 128, len)) : "memory");
  }
  const int pj = (idx < 2 ? wid + 1 : idx < 16 ? 1 + (idx - 2) % 7 : 0) & 7;
  Ctx C = {P.x, P.w, P.ws[0], P.out[0], P.ws[pj], P.out[pj], P.rank};
  asm volatile("" : "+s"(C.x), "+s"(C.w), "+s"(C.ws), "+s"(C.out), "+s"(C.pws), "+s"(C.pout), "+s"(C.rank));
  if (idx == 31) asm volatile("s_waitcnt vmcnt(0)" :: "v"(cjunk) : "memory");
  if (idx >= 16 && wid == 3) {
    const int u = idx - 16;
    const u8* tp = u >= 14 ? (const u8*)C.x : P.ws[1 + u % 7] + (u < 7 ? WS_XS + (size_t)C.rank * XS_STRIDE : (size_t)WS_XF);
    unsigned junk;
    asm volatile("global_load_dword %0, %1, off sc0 sc1\n\tglobal_load_dword %0, %2, off nt\n\ts_waitcnt vmcnt(0)" : "=&v"(junk)
                 : "v"(tp), "v"((const u8*)C.w + ((size_t)(u + 16 * (lane & 3)) << 21)) : "memory");
  }
  const int rank = C.rank;
  unsigned* const tmo = (unsigned*)(C.ws + WS_TMO);
  unsigned* const cnt = (unsigned*)(C.ws + WS_CNT + (size_t)bid * 128);
  LAS unsigned* xw = (LAS unsigned*)(lds + X_BASE);

  if (idx < 2) {
    const int cw = xcd * 2 + idx;
    if (wid < 7) {
      const char* sp = (const char*)(C.x + (size_t)(BM + 16 * cw + (lane >> 3)) * K);
      const char* sp0 = sp + (((lane & 7) ^ (lane >> 4)) << 4);
      const char* sp1 = sp + 8 * K * 2 + (((lane & 7) ^ (4 | (lane >> 4))) << 4);
      u8* dp = C.pws + WS_XS + (size_t)rank * XS_STRIDE + cw * 2048 + lane * 16;
      unsigned* fp = (unsigned*)(C.pws + WS_XF + (rank * 16 + cw) * 128);
      u32x4 r0[8], r1[8];
#pragma unroll
      for (int s = 0; s < 8; ++s) { r0[s] = (u32x4){0u, 0u, 0u, 0u}; r1[s] = (u32x4){0u, 0u, 0u, 0u}; }
#define CLOAD(s) asm volatile("global_load_dwordx4 %0, %2, off offset:%4\n\tglobal_load_dwordx4 %1, %3, off offset:%4" \
                              : "+v"(r0[s]), "+v"(r1[s]) : "v"(sp0), "v"(sp1), "n"((s) * 128) : "memory")
#define CSTORE(s) do { asm volatile("global_store_dwordx4 %0, %1, off sc0 sc1\n\tglobal_store_dwordx4 %0, %2, off offset:1024 sc0 sc1\n\ts_nop 1" \
                                    :: "v"(dp), "v"(r0[s]), "v"(r1[s]) : "memory"); dp += XT_BYTES; } while (0)
#define CFLAG() do { ++fv; asm volatile("s_mov_b64 exec, 1\n\tglobal_store_dword %0, %1, off sc0 sc1\n\ts_mov_b64 exec, -1" :: "v"(fp), "v"(fv) : "memory"); } while (0)
      CLOAD(0); CLOAD(1); CLOAD(2);
      WAITV(4); CSTORE(0);
      WAITV(4); CSTORE(1);
#pragma unroll
      for (int s = 3; s < 8; ++s) CLOAD(s);
      sp0 += 1024; sp1 += 1024;
      CLOAD(0); CLOAD(1);
      BAR();
      unsigned fv = (xw[1] - 1u) * NT;
      WAITV(16); CFLAG();
      CSTORE(2); CLOAD(2); WAITV(19); CFLAG(); BAR();
      WAITV(10);
#pragma unroll
      for (int t = 3; t < 8; ++t) { CSTORE(t); CLOAD(t); WAITV(7); CFLAG(); BAR(); }
      for (int k8 = 8; k8 < NT - 8; k8 += 8) {
        sp0 += 1024; sp1 += 1024;
#pragma unroll
        for (int i = 0; i < 8; ++i) { CSTORE(i); CLOAD(i); WAITV(7); CFLAG(); BAR(); }
      }
#pragma unroll
      for (int i = 0; i < 8; ++i) { CSTORE(i); WAITV(3); CFLAG(); BAR(); }
      WAITV(0);
      asm volatile(".Lgaq_m0:");
      CFLAG();
    } else {
      unsigned ticket = 0;
      if (lane == 0) ticket = draw(cnt);
      const unsigned call = (unsigned)__builtin_amdgcn_readfirstlane((int)ticket) + 1u;
      if (lane == 0) xw[1] = call;
      WAITL0();
      BAR();
      for (int k = 2; k < NT; ++k) BAR();
      if (idx == 0) {
        const u8* gp = C.ws + WS_G1 + xcd * 1024 + lane * 16;
        u64* dstg = (u64*)(P.ws[lane & 7] + WS_G2) + (call & 1) * (NR * 8) + rank * 8 + xcd;
        asm volatile("" : "+v"(dstg));
        unsigned spins = 0;
        const u64 t0 = __builtin_amdgcn_s_memrealtime();
        for (;;) {
          u32x4 a;
          asm volatile("global_load_dwordx4 %0, %1, off sc0 sc1\n\ts_waitcnt vmcnt(0)" : "=&v"(a) : "v"(gp) : "memory");
          if (__all(lane < 4 || (a[1] == call && a[3] == call))) {
            const unsigned mx = wave_max_u32(lane < 4 ? 0u : max(a[0], a[2]));
            if (lane < NR) st64(dstg, call, mx);
            break;
          }
          if (give_up(tmo, call, t0, spins)) break;
          __builtin_amdgcn_s_sleep(1);
        }
      }
    }
    return;
  }

  const bool tall = idx >= 16;
  Tile T;
  if (!tall) { T.src = rank; T.dst = (rank + pj) & 7; T.ct = xcd * 2 + (idx - 2) / 7; T.grp = 0; }
  else { const int u = idx - 16; T.grp = (xcd >> 2) * 4 + (u >> 2); T.src = (rank + T.grp) & 7; T.dst = rank; T.ct = (xcd & 3) * 4 + (u & 3); }

  if (wid < 4) {
    if (tall) consumer<9>(C, T, lds, wid, lane); else consumer<8>(C, T, lds, wid, lane);
    return;
  }
  const int f = bid - 128;
  const bool incoming = tall && (f >> 4) != rank;
  const bool remote = T.grp != 0;
  Early E;
  if (wid < 6) {
    if (tall) loader<true>(C, T, lds, wid - 4, lane); else loader<false>(C, T, lds, wid - 4, lane);
  } else if (wid == 6) {
    const u64 tb = __builtin_amdgcn_s_memrealtime();
    unsigned warm = 0;
    if (remote) {
      {
        const int l = lane & 7;
        const u8* tp = l < 2 ? C.ws + WS_XS + (size_t)T.src * XS_STRIDE + l * (XS_STRIDE - 4)
                     : l < 4 ? C.ws + WS_ES + (size_t)((l & 1) * (NR * 64) + f * 4) * 16384
                     : l < 6 ? C.out + (size_t)((l & 1) ? T.src * M + BM : (f >> 4) * M) * NS : C.out + (size_t)rank * M * NS;
        asm volatile("global_load_dword %0, %1, off sc0 sc1" : "+v"(warm) : "v"(tp) : "memory"); }
      __builtin_amdgcn_s_sleep(24);
#pragma unroll
      for (int i = 0; i < 5; ++i)
        asm volatile("global_load_dword %0, %1, off" : "+v"(warm) : "v"((const char*)C.x + (size_t)min((T.grp - 1) * 16 + T.ct + 112 * i, M - 1) * (K * 2) + lane * 128) : "memory");
    }
    if (remote) { BAR(); BAR(); }
    BAR();
    const int hold = tall ? (remote ? QB : 0) : !(T.ct & 1) ? QA0 : ((T.ct & 2) && T.ct != 15) ? QA3 : QA1;
    int tl = (int)(__builtin_amdgcn_s_memrealtime() - tb);
    for (int k = 1; k <= NT; ++k) {
      if (tl >= QT1 && tl < QT2) {
        tl += hold;
        while ((int)(__builtin_amdgcn_s_memrealtime() - tb) < tl) __builtin_amdgcn_s_sleep(1);
      }
      BAR();
      tl = max(tl, (int)(__builtin_amdgcn_s_memrealtime() - tb) - 40);
    }
    asm volatile("s_waitcnt vmcnt(0)" :: "v"(warm) : "memory");
  }
  if (wid < 7) {
    if (incoming) {
      const unsigned call = xw[1];
      early_fetch(C, E, f, wid - 4, lane, call);
      const bool dead = ld32<AGT>(tmo) == call;
      BAR();
      early_store(C, E, f, wid - 4, lane, dead ? __uint_as_float(0x7fc00000u) : 448.0f / __uint_as_float(xw[0]));
    }
    return;
  }
  unsigned ticket = 0, spins = 0;
  if (lane == 0) ticket = draw(cnt);
  const unsigned call = (unsigned)__builtin_amdgcn_readfirstlane((int)ticket) + 1u;
  if (lane == 0) { xw[1] = call; xw[2] = 0u; }
  WAITL0();
  if (remote) {
    const unsigned* fp = (const unsigned*)(C.ws + WS_XF + (T.src * 16 + (lane & 15)) * 128);
    const unsigned base = (call - 1u) * NT;
    unsigned ready = 0;
    for (int k = -3; k < NT; ++k) {
      const unsigned need = (unsigned)min(k + 4, NT);
      if (ready < (k < 0 ? need : min(need + 2u, (unsigned)NT))) {
        const u64 t0 = __builtin_amdgcn_s_memrealtime();
        for (;;) {
          const unsigned d = ld32<SYS>(fp) - base;
          ready = NT - wave_max_u32(d > (unsigned)NT ? (unsigned)NT : NT - d);
          if (ready >= need) break;
          if (give_up(tmo, call, t0, spins)) { ready = NT; break; }
          __builtin_amdgcn_s_sleep(1);
        }
      }
      BAR();
    }
  } else {
    const u8* cp = C.ws + WS_XC + (lane & 7) * 128;
    unsigned cv = 0;
    for (int k = -1; k < 5; ++k) BAR();
    if (!tall) asm volatile("global_load_dword %0, %1, off sc0 sc1" : "+v"(cv) : "v"(cp) : "memory");
    BAR(); BAR();
    if (!tall) {
      asm volatile("s_waitcnt vmcnt(0)" : "+v"(cv) :: "memory");
      const u64 t0 = __builtin_amdgcn_s_memrealtime();
      while (!__all(cv - call * 32u < 0x80000000u)) {
        if (give_up(tmo, call, t0, spins)) break;
        __builtin_amdgcn_s_sleep(1);
        cv = ld32<SYS>(cp);
      }
    }
    for (int k = 7; k < NT; ++k) BAR();
  }
  if (incoming) early_fetch(C, E, f, 3, lane, call);
  if (!tall) {
    unsigned junk;
    asm volatile("global_load_dword %0, %1, off sc0 sc1\n\ts_waitcnt vmcnt(0)" : "=&v"(junk) : "v"(C.pout + (size_t)(rank * M + (lane & 1) * (BM - 1)) * NS + T.ct * BN) : "memory");
  }
  unsigned mx;
  {
    const u64* gp = (const u64*)(C.ws + WS_G2) + (call & 1) * (NR * 8) + lane;
    const u64 t0 = __builtin_amdgcn_s_memrealtime();
    for (;;) {
      u64 a; unsigned tm;
      asm volatile("global_load_dwordx2 %0, %2, off sc0 sc1\n\tglobal_load_dword %1, %3, off sc1\n\ts_waitcnt vmcnt(0)"
                   : "=&v"(a), "=&v"(tm) : "v"(gp), "v"(tmo) : "memory");
      if (tm == call) { mx = 0x7fc00000u; break; }
      if (__all((unsigned)(a >> 32) == call)) { mx = wave_max_u32((unsigned)a); break; }
      if (__builtin_amdgcn_s_memrealtime() - t0 > DEADLINE_TICKS) st32<AGT>(tmo, call);
      __builtin_amdgcn_s_sleep(2);
    }
    if (lane == 0) { xw[0] = mx; __hip_atomic_store(&xw[2], 1u, __ATOMIC_RELAXED, __HIP_MEMORY_SCOPE_WORKGROUP); }
    WAITL0();
  }
  BAR();
  if (!tall) {
    BAR();
    if (lane == 0) st32<SYS>(C.pws + WS_DONE + (size_t)(rank * 16 + T.ct) * 128, call);
  } else if (incoming) {
    early_store(C, E, f, 3, lane, 448.0f / __uint_as_float(mx));
    const unsigned* fl = (const unsigned*)(C.ws + WS_DONE + (size_t)f * 128);
    const u64 t1 = __builtin_amdgcn_s_memrealtime();
    while (ld32<SYS>(fl) < call) {
      if (give_up(tmo, call, t1, spins)) { if (lane == 0) *(GAS u8*)(size_t)(C.out + (size_t)rank * M * NS) = 0x7f; break; }
      __builtin_amdgcn_s_sleep(1);
    }
  }
}

extern "C" void kernel_launch(int rank, int nranks, ncclComm_t comm, hipStream_t stream,
                              void* const* d_in, const int* in_sizes, int n_in,
                              void* d_out, int out_size, const int* in_dtypes, int out_dtype,
                              void* const* d_out_all, void* const* d_ws_all, size_t ws_bytes) {
  (void)comm; (void)in_sizes; (void)n_in; (void)d_out; (void)out_size; (void)in_dtypes; (void)out_dtype;
  if (nranks != NR || ws_bytes < WS_END || (unsigned)rank >= (unsigned)NR) { fprintf(stderr, "unsupported configuration\n"); return; }
  static bool init = false;
  if (!init) {
    HIP_CHECK(hipFuncSetAttribute((const void*)gemm_a2a_q, hipFuncAttributeMaxDynamicSharedMemorySize, LDS_BYTES));
    init = true;
  }
  Params P;
  P.x = (const u16*)d_in[0]; P.w = (const u16*)d_in[1];
  for (int j = 0; j < NR; ++j) { P.out[j] = (u8*)d_out_all[(rank + j) % NR]; P.ws[j] = (u8*)d_ws_all[(rank + j) % NR]; }
  P.rank = rank; P.pad = 0;
  hipLaunchKernelGGL(gemm_a2a_q, dim3(NWG), dim3(512), LDS_BYTES, stream, P);
  HIP_CHECK(hipGetLastError());
}
```

```cpp
#include <hip/hip_runtime.h>
#include <rccl/rccl.h>
#include <cstdio>

#define HIP_CHECK(expr) do { hipError_t _e = (expr); if (_e != hipSuccess) { \
    fprintf(stderr, "HIP error %s at %s:%d\n", hipGetErrorString(_e), __FILE__, __LINE__); \
    return; } } while (0)

#define LAS __attribute__((address_space(3)))
#define GAS __attribute__((address_space(1)))
typedef short s16x4 __attribute__((ext_vector_type(4)));
typedef short s16x8 __attribute__((ext_vector_type(8)));
typedef __bf16 bf16x8 __attribute__((ext_vector_type(8)));
typedef float f32x4 __attribute__((ext_vector_type(4)));
typedef unsigned u32x4 __attribute__((ext_vector_type(4)));
typedef unsigned short u16;
typedef unsigned long long u64;
typedef unsigned char u8;

constexpr int NR = 8;
constexpr int M = 512, N = 16384, K = 4096;
constexpr int NS = N / NR;
constexpr int BM = 256, BN = 128, BK = 64, NT = K / BK;
constexpr int NWG = 256;
constexpr int XT_BYTES = BM * BK * 2;
constexpr int A_BYTES = 288 * BK * 2, B_BYTES = BK * BN * 2;
constexpr int B_BASE = 3 * A_BYTES;
constexpr int X_BASE = B_BASE + 3 * B_BYTES;
constexpr int LDS_BYTES = X_BASE + 256;
constexpr int QT1 = 1570, QT2 = 3670, QA0 = 71, QA1 = 92, QA3 = 126, QB = 131, ECUT = 6100;

constexpr size_t WS_CNT = 0;
constexpr size_t WS_TMO = WS_CNT + (size_t)NWG * 128;
constexpr size_t WS_G1 = WS_TMO + 256;
constexpr size_t WS_G2 = WS_G1 + 8 * 128 * 8;
constexpr size_t WS_DONE = WS_G2 + 2 * NR * 8 * 8;
constexpr size_t WS_XF = WS_DONE + (size_t)NR * 16 * 128;
constexpr size_t WS_EF = WS_XF + (size_t)NR * 16 * 128;
constexpr size_t WS_XC = WS_EF + (size_t)2 * NR * 64 * 128;
constexpr size_t WS_XS = 1u << 20;
constexpr size_t XS_STRIDE = (size_t)NT * XT_BYTES;
constexpr size_t WS_ES = WS_XS + (size_t)NR * XS_STRIDE;
constexpr size_t WS_END = WS_ES + (size_t)2 * NR * 64 * 16384;
static_assert(WS_XC + 8 * 128 <= WS_XS, "layout");
constexpr u64 DEADLINE_TICKS = 40000000ull;

struct Params {
  const u16* x; const u16* w;
  int rank; int pad;
  u8* ws[NR];
  u8* out[NR];
};
struct Ctx { const u16* x; const u16* w; u8* ws; u8* out; u8* pws; u8* pout; int rank; };
struct Tile { int src, dst, ct, grp; };

#define SB() __builtin_amdgcn_sched_barrier(0)
#define BAR() do { asm volatile("" ::: "memory"); __builtin_amdgcn_s_barrier(); asm volatile("" ::: "memory"); } while (0)
#define WAITV(n) asm volatile("s_waitcnt vmcnt(%0)" :: "n"(n) : "memory")
#define WAITL0() asm volatile("s_waitcnt lgkmcnt(0)" ::: "memory")
#define DI __device__ __forceinline__

#define SYS __HIP_MEMORY_SCOPE_SYSTEM
#define AGT __HIP_MEMORY_SCOPE_AGENT
template <int SC> DI unsigned ld32(const void* p) { return __hip_atomic_load((const GAS unsigned*)(size_t)p, __ATOMIC_RELAXED, SC); }
template <int SC> DI void st32(void* p, unsigned v) { __hip_atomic_store((GAS unsigned*)(size_t)p, v, __ATOMIC_RELAXED, SC); }
DI void st64(void* p, unsigned tag, unsigned v) { __hip_atomic_store((GAS u64*)(size_t)p, ((u64)tag << 32) | v, __ATOMIC_RELAXED, SYS); }
DI unsigned draw(void* p) { return __hip_atomic_fetch_add((GAS unsigned*)(size_t)p, 1u, __ATOMIC_RELAXED, AGT); }
DI bf16x8 ldA(unsigned addr, int off) { return *(LAS bf16x8*)(size_t)(addr + off); }
DI s16x4 ldB(unsigned addr, int off) { return __builtin_amdgcn_ds_read_tr16_b64_v4i16((LAS s16x4*)(size_t)(addr + off)); }
DI unsigned wave_max_u32(unsigned v) {
  v = max(v, (unsigned)__builtin_amdgcn_update_dpp(0, (int)v, 0x111, 0xf, 0xf, false));
  v = max(v, (unsigned)__builtin_amdgcn_update_dpp(0, (int)v, 0x112, 0xf, 0xf, false));
  v = max(v, (unsigned)__builtin_amdgcn_update_dpp(0, (int)v, 0x114, 0xf, 0xf, false));
  v = max(v, (unsigned)__builtin_amdgcn_update_dpp(0, (int)v, 0x118, 0xf, 0xf, false));
  const unsigned a = __builtin_amdgcn_readlane(v, 15), b = __builtin_amdgcn_readlane(v, 31);
  const unsigned c = __builtin_amdgcn_readlane(v, 47), d = __builtin_amdgcn_readlane(v, 63);
  return max(max(a, b), max(c, d));
}
DI bool give_up(unsigned* tmo, unsigned call, u64 t0, unsigned& spins) {
  if ((++spins & 7u) != 0u) return false;
  if (ld32<AGT>(tmo) == call || __builtin_amdgcn_s_memrealtime() - t0 > DEADLINE_TICKS) {
    st32<AGT>(tmo, call);
    return true;
  }
  return false;
}
DI unsigned pkbf(float a, float b) {
  typedef float f32x2 __attribute__((ext_vector_type(2)));
  typedef __bf16 bf16x2 __attribute__((ext_vector_type(2)));
  return __builtin_bit_cast(unsigned, __builtin_convertvector((f32x2){a, b}, bf16x2));
}
DI unsigned q4(float a, float b, float c, float d, float inv) {
  const int t = __builtin_amdgcn_cvt_pk_fp8_f32(a * inv, b * inv, 0, false);
  return (unsigned)__builtin_amdgcn_cvt_pk_fp8_f32(c * inv, d * inv, t, true);
}
DI void store16(u8* p, const u32x4 v) { asm volatile("global_store_dwordx4 %0, %1, off sc0 sc1\n\ts_nop 1" :: "v"(p), "v"(v) : "memory"); }
DI void store_row16(u8* p, unsigned w0, unsigned w1, unsigned w2, unsigned w3, bool odd) {
  store16(p, (u32x4){odd ? w1 : w0, odd ? w0 : w1, odd ? w3 : w2, odd ? w2 : w3});
}

DI unsigned q4b(unsigned lo, unsigned hi, float inv) {
  return q4(__uint_as_float(lo << 16), __uint_as_float(lo & 0xffff0000u), __uint_as_float(hi << 16), __uint_as_float(hi & 0xffff0000u), inv);
}
DI void store_bf16_row(u8* p, const u32x4 a, const u32x4 b, float inv, bool odd) {
  store_row16(p, q4b(a[0], a[1], inv), q4b(a[2], a[3], inv), q4b(b[0], b[1], inv), q4b(b[2], b[3], inv), odd);
}

DI int slow_ktile(const Ctx& C, int i) {
  const int ph = (int)((size_t)C.x >> 7), k63 = (63 - ph) & 63;
  return i < 7 ? 10 + ((1 - ph) & 7) + 8 * i : k63 >= 10 ? k63 : NT;
}

template <bool TALL>
DI void loader(const Ctx& C, const Tile T, LAS char* lds, const int p, const int lane) {
  const bool remote = TALL && T.grp != 0;
  const void* base = remote ? (const void*)(C.ws + WS_XS + (size_t)T.src * XS_STRIDE) : (const void*)(C.x + (size_t)(TALL ? BM : 0) * K);
  __amdgpu_buffer_rsrc_t rA = __builtin_amdgcn_make_buffer_rsrc((void*)base, 0, BM * K * 2, 0x00020000);
  __amdgpu_buffer_rsrc_t rE = __builtin_amdgcn_make_buffer_rsrc((void*)(C.x + (size_t)T.grp * 32 * K), 0, 32 * K * 2, 0x00020000);
  __amdgpu_buffer_rsrc_t rB = __builtin_amdgcn_make_buffer_rsrc((void*)(C.w + (size_t)T.dst * NS + T.ct * BN), 0, (K - 1) * N * 2 + BN * 2, 0x00020000);
  const unsigned pstr = remote ? 1024u : 8u * K * 2, kstr = remote ? (unsigned)XT_BYTES : BK * 2u;
  const unsigned voL = (unsigned)(((lane >> 3) + 8 * p) * (K * 2) + (((lane & 7) ^ ((p << 2) | (lane >> 4))) << 4));
  const unsigned voA = remote ? (unsigned)lane * 16u + p * 1024u : voL;
  const unsigned voB = (unsigned)(((lane >> 4) + 8 * p) * (N * 2) + (((lane & 15) ^ ((lane >> 4) | (p << 3))) << 4));
  __amdgpu_buffer_rsrc_t rX = __builtin_amdgcn_make_buffer_rsrc((void*)(C.ws + WS_XS + (size_t)C.rank * XS_STRIDE), 0, BM * K * 2, 0x00020000);
  const int s0 = slow_ktile(C, 0), s7 = slow_ktile(C, 7);
  auto issueA = [&](int kt, int buf) {
    if (!TALL && kt >= 10 && (((kt - s0) & 7) == 0 || kt == s7)) {
#pragma unroll
      for (int j = 0; j < 16; ++j)
        __builtin_amdgcn_raw_ptr_buffer_load_lds(rX, (LAS void*)(lds + buf * A_BYTES + (p + 2 * j + 2 * (j >> 3)) * 1024), 16, (unsigned)lane * 16u + p * 1024u, kt * XT_BYTES + 2 * j * 1024, 0, 17);
      return;
    }
#pragma unroll
    for (int j = 0; j < 16; ++j)
      __builtin_amdgcn_raw_ptr_buffer_load_lds(rA, (LAS void*)(lds + buf * A_BYTES + (p + 2 * j + 2 * (j >> 3)) * 1024), 16, voA, kt * kstr + 2 * j * pstr, 0, TALL ? 17 : 0);
    if (TALL) {
#pragma unroll
      for (int j = 0; j < 2; ++j)
        __builtin_amdgcn_raw_ptr_buffer_load_lds(rE, (LAS void*)(lds + buf * A_BYTES + (16 + p + 18 * j) * 1024), 16, voL, kt * (BK * 2) + j * 16 * K * 2, 0, 0);
    }
  };
  auto issueB = [&](int kt, int buf) {
#pragma unroll
    for (int j = 0; j < 8; ++j) {
      const int pc = (j & 1) + 4 * (j >> 1);
      __builtin_amdgcn_raw_ptr_buffer_load_lds(rB, (LAS void*)(lds + B_BASE + buf * B_BYTES + (pc + 2 * p) * 1024), 16, voB, (kt * BK + pc * 4) * (N * 2), 0, TALL ? 0 : 2);
    }
  };
  constexpr int NA = TALL ? 18 : 16;
  if (remote) {
    __builtin_amdgcn_s_sleep(24);
    issueB(0, 0); issueB(1, 1);
    BAR();
    issueA(0, 0);
    BAR();
    issueA(1, 1);
    WAITV(NA);
  } else {
    issueB(0, 0); issueA(0, 0); issueB(1, 1); issueA(1, 1);
    WAITV(NA + 8);
  }
  BAR();
  int buf = 2;
  for (int kt = 0; kt < NT - 2; ++kt) {
    issueB(kt + 2, buf); issueA(kt + 2, buf);
    WAITV(NA + 8);
    buf = (buf == 2) ? 0 : buf + 1;
    BAR();
  }
  WAITV(0);
  BAR(); BAR();
}

template <int MB>
DI void consumer(const Ctx& C, const Tile T, LAS char* lds, const int wid, const int lane) {
  const u64 tb = __builtin_amdgcn_s_memrealtime();
  const unsigned lbase = (unsigned)(size_t)lds;
  LAS unsigned* xw = (LAS unsigned*)(lds + X_BASE);
  const int wr = wid >> 1, wc = wid & 1;
  const int g = lane >> 4, c = lane & 15, q = (lane & 15) >> 2, pp = lane & 3;
  unsigned aA[2], aB[4];
#pragma unroll
  for (int ks = 0; ks < 2; ++ks) aA[ks] = lbase + (wr * 144 + c) * 128 + (((4 * ks + g) ^ (c >> 1)) << 4);
#pragma unroll
  for (int b = 0; b < 4; ++b) {
    const int row = 8 * g + q, ch = 8 * wc + 2 * pp + (b >> 1);
    aB[b] = lbase + B_BASE + row * 256 + ((ch ^ ((row & 3) | (row & 8))) << 4) + (((b ^ pp) & 1) << 3);
  }
  f32x4 acc[MB][4];
  bf16x8 Af[MB][2];
  s16x4 Bq[2][4][2][2];
  auto rdA = [&](int m, int ks) { Af[m][ks] = ldA(aA[ks], m * 2048); SB(); };
  auto rdB = [&](int set, int i) { Bq[set][i >> 2][(i >> 1) & 1][i & 1] = ldB(aB[i >> 2], ((i >> 1) & 1) * 8192 + (i & 1) * 1024); SB(); };
  auto mma = [&](int set, int m, int i) {
    const int b = (m & 1) ? 3 - (i >> 1) : (i >> 1), ks = ((i >> 1) ^ i) & 1;
    s16x8 bv = __builtin_shufflevector(Bq[set][b][ks][0], Bq[set][b][ks][1], 0, 1, 2, 3, 4, 5, 6, 7);
    acc[m][b] = __builtin_amdgcn_mfma_f32_16x16x32_bf16(__builtin_bit_cast(bf16x8, bv), Af[m][ks], acc[m][b], 0, 0, 0);
    SB();
  };
  int buf = 0;
  auto ktile = [&](int set) {
    int da = 0, db = 0;
#pragma unroll
    for (int m = 0; m < MB - 4; ++m)
#pragma unroll
      for (int i = 0; i < 8; ++i) {
        mma(set, m, i);
        if (m == 0 && i < 6) rdA(2 + (i >> 1), i & 1);
        if (m > 0 && i < 2) rdA(m + 4, i);
        if (m == 0 && i == 6) { db = buf == 2 ? -2 * B_BYTES : B_BYTES; asm volatile("" : "+s"(db)); SB(); }
        if (m == 1 && i >= 3 && i < 7) { aB[i - 3] += db; SB(); }
        if (m == 2 && i == 3) { da = buf == 2 ? -2 * A_BYTES : A_BYTES; asm volatile("" : "+s"(da)); SB(); }
        if (m == 2 && i == 5) { buf = buf == 2 ? -1 : buf; asm volatile("" : "+s"(buf)); SB(); }
        if (m == 2 && i == 6) { ++buf; asm volatile("" : "+s"(buf)); SB(); }
        if (m == MB - 5 && (i == 3 || i == 4)) { aA[i - 3] += da; SB(); }
      }
    __builtin_amdgcn_s_waitcnt(0xc07f);
    SB();
    BAR();
    SB();
#pragma unroll
    for (int m = MB - 4; m < MB; ++m)
#pragma unroll
      for (int i = 0; i < 8; ++i) {
        const int j = (m - (MB - 4)) * 8 + i;
        mma(set, m, i);
        if (j < 16) rdB(set ^ 1, j);
        if (j >= 16 && j < 20) rdA((j - 16) >> 1, j & 1);
      }
  };
#pragma unroll
  for (int m = 0; m < MB; ++m)
#pragma unroll
    for (int b = 0; b < 4; ++b) { acc[m][b] = (f32x4){0.f, 0.f, 0.f, 0.f}; asm volatile("" : "+v"(acc[m][b])); }
  SB();
  const int id = ((T.grp - 1) * 16 + T.ct) * 3 + wid;
  if (MB == 9 && T.grp != 0 && wid < 3 && id < 256) {
    const int bi = id >> 5, pc = id & 31, kt = min(slow_ktile(C, bi), NT - 1);
    u32x4 v;
    __builtin_amdgcn_s_sleep(24);
    asm volatile("global_load_dwordx4 %0, %1, off\n\ts_waitcnt vmcnt(0)" : "=&v"(v)
                 : "v"((const char*)(C.x + (size_t)(8 * pc + (lane >> 3)) * K + kt * BK) + (((lane & 7) ^ (((pc & 1) << 2) | (lane >> 4))) << 4)) : "memory");
    store16(C.ws + WS_XS + (size_t)C.rank * XS_STRIDE + kt * XT_BYTES + pc * 1024 + lane * 16, v);
    WAITV(0);
    if (lane == 0) __hip_atomic_fetch_add((GAS unsigned*)(size_t)(C.ws + WS_XC + bi * 128), 1u, __ATOMIC_RELAXED, SYS);
  }
  if (MB == 9 && T.grp != 0) { BAR(); BAR(); }
  BAR();
#pragma unroll
  for (int i = 0; i < 16; ++i) rdB(0, i);
#pragma unroll
  for (int i = 0; i < 4; ++i) rdA(i >> 1, i & 1);
#pragma unroll 1
  for (int kt = 0; kt < NT; kt += 2) { ktile(0); ktile(1); }
  float am = 0.f;
#pragma unroll
  for (int m = 0; m < MB; ++m)
#pragma unroll
    for (int b = 0; b < 4; ++b) {
      am = __builtin_fmaxf(__builtin_fmaxf(am, __builtin_fabsf(acc[m][b][0])), __builtin_fabsf(acc[m][b][1]));
      am = __builtin_fmaxf(__builtin_fmaxf(am, __builtin_fabsf(acc[m][b][2])), __builtin_fabsf(acc[m][b][3]));
    }
  const unsigned mu = wave_max_u32(__float_as_uint(am));
  const unsigned call = xw[1];
  if (lane == 0)
    st64((u64*)(C.ws + WS_G1) + (blockIdx.x & 7) * 128 + (blockIdx.x >> 3) * 4 + wid, call, mu);
  u8* ob = C.pout + T.ct * BN + wc * 64 + 16 * g;
  u8* op = ob + (size_t)(T.src * M + (MB == 9 ? BM : 0) + wr * 128 + c) * NS;
  if (MB == 8) {
    u32x4 pk[8][2];
#pragma unroll
    for (int m = 0; m < 8; ++m)
#pragma unroll
      for (int h = 0; h < 2; ++h)
        pk[m][h] = (u32x4){pkbf(acc[m][2 * h][0], acc[m][2 * h][1]), pkbf(acc[m][2 * h][2], acc[m][2 * h][3]),
                           pkbf(acc[m][2 * h + 1][0], acc[m][2 * h + 1][1]), pkbf(acc[m][2 * h + 1][2], acc[m][2 * h + 1][3])};
    const int slot = (call & 1) * (NR * 64) + (C.rank * 16 + T.ct) * 4 + wid;
    u8* ep = C.pws + WS_ES + (size_t)slot * 16384 + lane * 16;
    int ne = 0;
#pragma unroll
    for (int m = 0; m < 8; ++m)
      if (ne == m && (int)(__builtin_amdgcn_s_memrealtime() - tb) < ECUT &&
          __builtin_amdgcn_readfirstlane(__hip_atomic_load(&xw[2], __ATOMIC_RELAXED, __HIP_MEMORY_SCOPE_WORKGROUP)) == 0) {
#pragma unroll
        for (int h = 0; h < 2; ++h) {
          store16(ep + m * 2048 + h * 1024, pk[m][h]);
          WAITV(1);
          __builtin_amdgcn_s_sleep(8);
        }
        ne = m + 1;
      }
    WAITV(0);
    if (lane == 0) st32<SYS>(C.pws + WS_EF + (size_t)slot * 128, call * 16u + ne);
    BAR();
    const float inv = 448.0f / __uint_as_float(xw[0]);
#pragma unroll
    for (int m = 7; m >= 0; --m)
      if (m >= ne) { store_bf16_row(op + (size_t)m * 16 * NS, pk[m][0], pk[m][1], inv, g & 1); SB(); }
  } else {
    u8* oe = ob + (size_t)(C.rank * M + T.grp * 32 + wr * 16 + c) * NS;
    BAR();
    const float inv = 448.0f / __uint_as_float(xw[0]);
#pragma unroll
    for (int m = 0; m < MB; ++m) {
      unsigned w[4];
#pragma unroll
      for (int b = 0; b < 4; ++b) w[b] = q4(acc[m][b][0], acc[m][b][1], acc[m][b][2], acc[m][b][3], inv);
      store_row16(m < 8 ? op + (size_t)m * 16 * NS : oe, w[0], w[1], w[2], w[3], g & 1);
      SB();
    }
  }
  if (MB == 9) asm volatile(".Lgaq_m9:\n\ts_nop 0");
  if (MB == 8) {
    asm volatile(".Lgaq_m8:");
    WAITV(0);
    BAR();
  }
}

struct Early { u32x4 v[16]; int ne; };
#define LD8(o, p) asm volatile("global_load_dwordx4 %0, %8, off sc0 sc1\n\tglobal_load_dwordx4 %1, %8, off offset:1024 sc0 sc1\n\t" \
               "global_load_dwordx4 %2, %8, off offset:2048 sc0 sc1\n\tglobal_load_dwordx4 %3, %8, off offset:3072 sc0 sc1\n\t" \
               "global_load_dwordx4 %4, %9, off sc0 sc1\n\tglobal_load_dwordx4 %5, %9, off offset:1024 sc0 sc1\n\t" \
               "global_load_dwordx4 %6, %9, off offset:2048 sc0 sc1\n\tglobal_load_dwordx4 %7, %9, off offset:3072 sc0 sc1\n\ts_waitcnt vmcnt(0)" \
               : "=&v"(E.v[o]), "=&v"(E.v[o + 1]), "=&v"(E.v[o + 2]), "=&v"(E.v[o + 3]), "=&v"(E.v[o + 4]), "=&v"(E.v[o + 5]), "=&v"(E.v[o + 6]), "=&v"(E.v[o + 7]) \
               : "v"(p), "v"((p) + 4096) : "memory")
DI void early_fetch(const Ctx& C, Early& E, const int f, const int w, const int lane, const unsigned call) {
  const int slot = (call & 1) * (NR * 64) + f * 4 + w;
  const unsigned* fl = (const unsigned*)(C.ws + WS_EF + (size_t)slot * 128);
  unsigned spins = 0;
  const u64 t0 = __builtin_amdgcn_s_memrealtime();
  unsigned v;
  while ((v = (unsigned)__builtin_amdgcn_readfirstlane((int)ld32<SYS>(fl))) < call * 16u) {
    if (give_up((unsigned*)(C.ws + WS_TMO), call, t0, spins)) { v = call * 16u + 8u; break; }
    __builtin_amdgcn_s_sleep(2);
  }
  E.ne = (int)(v - call * 16u);
  const u8* ep = C.ws + WS_ES + (size_t)slot * 16384 + lane * 16;
  if (E.ne > 0) LD8(0, ep);
  if (E.ne > 4) LD8(8, ep + 8192);
}
DI void early_store(const Ctx& C, const Early& E, const int f, const int w, const int lane, const float inv) {
  u8* op = C.out + (size_t)((f >> 4) * M + (w >> 1) * 128 + (lane & 15)) * NS + (f & 15) * BN + (w & 1) * 64 + (lane & 48);
#pragma unroll
  for (int m = 0; m < 8; ++m)
    if (m < E.ne) store_bf16_row(op + (size_t)m * 16 * NS, E.v[2 * m], E.v[2 * m + 1], inv, (lane >> 4) & 1);
}

__global__ __launch_bounds__(512, 2) void gemm_a2a_q(const Params P) {
  extern __shared__ __attribute__((aligned(16))) char smem[];
  LAS char* lds = (LAS char*)smem;
  const int tid = threadIdx.x, lane = tid & 63;
  const int wid = __builtin_amdgcn_readfirstlane(tid >> 6);
  const int bid = blockIdx.x;
  const int xcd = bid & 7, idx = bid >> 3;
  unsigned cjunk = 0;
  if (idx == 31) {
    u64 pc; int len;
    asm volatile("s_getpc_b64 %0\n.Lgaq_pc:\n\ts_mov_b32 %1, .Lgaq_m0-.Lgaq_pc\n\ts_max_i32 %1, %1, .Lgaq_m8-.Lgaq_pc\n\t"
                 "s_max_i32 %1, %1, .Lgaq_m9-.Lgaq_pc\n\ts_max_i32 %1, %1, 0" : "=s"(pc), "=s"(len) :: "scc");
    asm volatile("global_load_dword %0, %1, off" : "+v"(cjunk) : "v"(pc + (unsigned)min(tid * 128, len)) : "memory");
  }
  const int pj = (idx < 2 ? wid + 1 : idx < 16 ? 1 + (idx - 2) % 7 : 0) & 7;
  Ctx C = {P.x, P.w, P.ws[0], P.out[0], P.ws[pj], P.out[pj], P.rank};
  asm volatile("" : "+s"(C.x), "+s"(C.w), "+s"(C.ws), "+s"(C.out), "+s"(C.pws), "+s"(C.pout), "+s"(C.rank));
  if (idx == 31) asm volatile("s_waitcnt vmcnt(0)" :: "v"(cjunk) : "memory");
  if (idx >= 16 && wid == 3) {
    const int u = idx - 16;
    const u8* tp = u >= 14 ? (const u8*)C.x : P.ws[1 + u % 7] + (u < 7 ? WS_XS + (size_t)C.rank * XS_STRIDE : (size_t)WS_XF);
    unsigned junk;
    asm volatile("global_load_dword %0, %1, off sc0 sc1\n\tglobal_load_dword %0, %2, off nt\n\ts_waitcnt vmcnt(0)" : "=&v"(junk)
                 : "v"(tp), "v"((const u8*)C.w + ((size_t)(u + 16 * (lane & 3)) << 21)) : "memory");
  }
  const int rank = C.rank;
  unsigned* const tmo = (unsigned*)(C.ws + WS_TMO);
  unsigned* const cnt = (unsigned*)(C.ws + WS_CNT + (size_t)bid * 128);
  LAS unsigned* xw = (LAS unsigned*)(lds + X_BASE);

  if (idx < 2) {
    const int cw = xcd * 2 + idx;
    if (wid < 7) {
      const char* sp = (const char*)(C.x + (size_t)(BM + 16 * cw + (lane >> 3)) * K);
      const char* sp0 = sp + (((lane & 7) ^ (lane >> 4)) << 4);
      const char* sp1 = sp + 8 * K * 2 + (((lane & 7) ^ (4 | (lane >> 4))) << 4);
      u8* dp = C.pws + WS_XS + (size_t)rank * XS_STRIDE + cw * 2048 + lane * 16;
      unsigned* fp = (unsigned*)(C.pws + WS_XF + (rank * 16 + cw) * 128);
      u32x4 r0[8], r1[8];
#pragma unroll
      for (int s = 0; s < 8; ++s) { r0[s] = (u32x4){0u, 0u, 0u, 0u}; r1[s] = (u32x4){0u, 0u, 0u, 0u}; }
#define CLOAD(s) asm volatile("global_load_dwordx4 %0, %2, off offset:%4\n\tglobal_load_dwordx4 %1, %3, off offset:%4" \
                              : "+v"(r0[s]), "+v"(r1[s]) : "v"(sp0), "v"(sp1), "n"((s) * 128) : "memory")
#define CSTORE(s) do { asm volatile("global_store_dwordx4 %0, %1, off sc0 sc1\n\tglobal_store_dwordx4 %0, %2, off offset:1024 sc0 sc1\n\ts_nop 1" \
                                    :: "v"(dp), "v"(r0[s]), "v"(r1[s]) : "memory"); dp += XT_BYTES; } while (0)
#define CFLAG() do { ++fv; asm volatile("s_mov_b64 exec, 1\n\tglobal_store_dword %0, %1, off sc0 sc1\n\ts_mov_b64 exec, -1" :: "v"(fp), "v"(fv) : "memory"); } while (0)
      CLOAD(0); CLOAD(1); CLOAD(2);
      WAITV(4); CSTORE(0);
      WAITV(4); CSTORE(1);
#pragma unroll
      for (int s = 3; s < 8; ++s) CLOAD(s);
      sp0 += 1024; sp1 += 1024;
      CLOAD(0); CLOAD(1);
      BAR();
      unsigned fv = (xw[1] - 1u) * NT;
      WAITV(16); CFLAG();
      CSTORE(2); CLOAD(2); WAITV(19); CFLAG(); BAR();
      WAITV(10);
#pragma unroll
      for (int t = 3; t < 8; ++t) { CSTORE(t); CLOAD(t); WAITV(7); CFLAG(); BAR(); }
      for (int k8 = 8; k8 < NT - 8; k8 += 8) {
        sp0 += 1024; sp1 += 1024;
#pragma unroll
        for (int i = 0; i < 8; ++i) { CSTORE(i); CLOAD(i); WAITV(7); CFLAG(); BAR(); }
      }
#pragma unroll
      for (int i = 0; i < 8; ++i) { CSTORE(i); WAITV(3); CFLAG(); BAR(); }
      WAITV(0);
      asm volatile(".Lgaq_m0:");
      CFLAG();
    } else {
      unsigned ticket = 0;
      if (lane == 0) ticket = draw(cnt);
      const unsigned call = (unsigned)__builtin_amdgcn_readfirstlane((int)ticket) + 1u;
      if (lane == 0) xw[1] = call;
      WAITL0();
      BAR();
      for (int k = 2; k < NT; ++k) BAR();
      if (idx == 0) {
        const u8* gp = C.ws + WS_G1 + xcd * 1024 + lane * 16;
        u64* dstg = (u64*)(P.ws[lane & 7] + WS_G2) + (call & 1) * (NR * 8) + rank * 8 + xcd;
        asm volatile("" : "+v"(dstg));
        unsigned spins = 0;
        const u64 t0 = __builtin_amdgcn_s_memrealtime();
        for (;;) {
          u32x4 a;
          asm volatile("global_load_dwordx4 %0, %1, off sc0 sc1\n\ts_waitcnt vmcnt(0)" : "=&v"(a) : "v"(gp) : "memory");
          if (__all(lane < 4 || (a[1] == call && a[3] == call))) {
            const unsigned mx = wave_max_u32(lane < 4 ? 0u : max(a[0], a[2]));
            if (lane < NR) st64(dstg, call, mx);
            break;
          }
          if (give_up(tmo, call, t0, spins)) break;
          __builtin_amdgcn_s_sleep(1);
        }
      }
    }
    return;
  }

  const bool tall = idx >= 16;
  Tile T;
  if (!tall) { T.src = rank; T.dst = (rank + pj) & 7; T.ct = xcd * 2 + (idx - 2) / 7; T.grp = 0; }
  else { const int u = idx - 16; T.grp = (xcd >> 2) * 4 + (u >> 2); T.src = (rank + T.grp) & 7; T.dst = rank; T.ct = (xcd & 3) * 4 + (u & 3); }

  if (wid < 4) {
    if (tall) consumer<9>(C, T, lds, wid, lane); else consumer<8>(C, T, lds, wid, lane);
    return;
  }
  const int f = bid - 128;
  const bool incoming = tall && (f >> 4) != rank;
  const bool remote = T.grp != 0;
  Early E;
  if (wid < 6) {
    if (tall) loader<true>(C, T, lds, wid - 4, lane); else loader<false>(C, T, lds, wid - 4, lane);
  } else if (wid == 6) {
    const u64 tb = __builtin_amdgcn_s_memrealtime();
    unsigned warm = 0;
    if (remote) {
      {
        const int l = lane & 7;
        const u8* tp = l < 2 ? C.ws + WS_XS + (size_t)T.src * XS_STRIDE + l * (XS_STRIDE - 4)
                     : l < 4 ? C.ws + WS_ES + (size_t)((l & 1) * (NR * 64) + f * 4) * 16384
                     : l < 6 ? C.out + (size_t)((l & 1) ? T.src * M + BM : (f >> 4) * M) * NS : C.out + (size_t)rank * M * NS;
        asm volatile("global_load_dword %0, %1, off sc0 sc1" : "+v"(warm) : "v"(tp) : "memory"); }
      __builtin_amdgcn_s_sleep(24);
#pragma unroll
      for (int i = 0; i < 5; ++i)
        asm volatile("global_load_dword %0, %1, off" : "+v"(warm) : "v"((const char*)C.x + (size_t)min((T.grp - 1) * 16 + T.ct + 112 * i, M - 1) * (K * 2) + lane * 128) : "memory");
    }
    if (remote) { BAR(); BAR(); }
    BAR();
    const int hold = tall ? (remote ? QB : 0) : !(T.ct & 1) ? QA0 : ((T.ct & 2) && T.ct != 15) ? QA3 : QA1;
    int tl = (int)(__builtin_amdgcn_s_memrealtime() - tb);
    for (int k = 1; k <= NT; ++k) {
      if (tl >= QT1 && tl < QT2) {
        tl += hold;
        while ((int)(__builtin_amdgcn_s_memrealtime() - tb) < tl) __builtin_amdgcn_s_sleep(1);
      }
      BAR();
      tl = max(tl, (int)(__builtin_amdgcn_s_memrealtime() - tb) - 40);
    }
    asm volatile("s_waitcnt vmcnt(0)" :: "v"(warm) : "memory");
  }
  if (wid < 7) {
    if (incoming) {
      const unsigned call = xw[1];
      early_fetch(C, E, f, wid - 4, lane, call);
      const bool dead = ld32<AGT>(tmo) == call;
      BAR();
      early_store(C, E, f, wid - 4, lane, dead ? __uint_as_float(0x7fc00000u) : 448.0f / __uint_as_float(xw[0]));
    }
    return;
  }
  unsigned ticket = 0, spins = 0;
  if (lane == 0) ticket = draw(cnt);
  const unsigned call = (unsigned)__builtin_amdgcn_readfirstlane((int)ticket) + 1u;
  if (lane == 0) { xw[1] = call; xw[2] = 0u; }
  WAITL0();
  if (remote) {
    const unsigned* fp = (const unsigned*)(C.ws + WS_XF + (T.src * 16 + (lane & 15)) * 128);
    const unsigned base = (call - 1u) * NT;
    unsigned ready = 0;
    for (int k = -3; k < NT; ++k) {
      const unsigned need = (unsigned)min(k + 4, NT);
      if (ready < (k < 0 ? need : min(need + 2u, (unsigned)NT))) {
        const u64 t0 = __builtin_amdgcn_s_memrealtime();
        for (;;) {
          const unsigned d = ld32<SYS>(fp) - base;
          ready = NT - wave_max_u32(d > (unsigned)NT ? (unsigned)NT : NT - d);
          if (ready >= need) break;
          if (give_up(tmo, call, t0, spins)) { ready = NT; break; }
          __builtin_amdgcn_s_sleep(1);
        }
      }
      BAR();
    }
  } else {
    const u8* cp = C.ws + WS_XC + (lane & 7) * 128;
    unsigned cv = 0;
    for (int k = -1; k < 5; ++k) BAR();
    if (!tall) asm volatile("global_load_dword %0, %1, off sc0 sc1" : "+v"(cv) : "v"(cp) : "memory");
    BAR(); BAR();
    if (!tall) {
      asm volatile("s_waitcnt vmcnt(0)" : "+v"(cv) :: "memory");
      const u64 t0 = __builtin_amdgcn_s_memrealtime();
      while (!__all(cv - call * 32u < 0x80000000u)) {
        if (give_up(tmo, call, t0, spins)) break;
        __builtin_amdgcn_s_sleep(1);
        cv = ld32<SYS>(cp);
      }
    }
    for (int k = 7; k < NT; ++k) BAR();
  }
  if (incoming) early_fetch(C, E, f, 3, lane, call);
  if (!tall) {
    unsigned junk;
    asm volatile("global_load_dword %0, %1, off sc0 sc1\n\ts_waitcnt vmcnt(0)" : "=&v"(junk) : "v"(C.pout + (size_t)(rank * M + (lane & 1) * (BM - 1)) * NS + T.ct * BN) : "memory");
  }
  unsigned mx;
  {
    const u64* gp = (const u64*)(C.ws + WS_G2) + (call & 1) * (NR * 8) + lane;
    const u64 t0 = __builtin_amdgcn_s_memrealtime();
    for (;;) {
      u64 a; unsigned tm;
      asm volatile("global_load_dwordx2 %0, %2, off sc0 sc1\n\tglobal_load_dword %1, %3, off sc1\n\ts_waitcnt vmcnt(0)"
                   : "=&v"(a), "=&v"(tm) : "v"(gp), "v"(tmo) : "memory");
      if (tm == call) { mx = 0x7fc00000u; break; }
      if (__all((unsigned)(a >> 32) == call)) { mx = wave_max_u32((unsigned)a); break; }
      if (__builtin_amdgcn_s_memrealtime() - t0 > DEADLINE_TICKS) st32<AGT>(tmo, call);
      __builtin_amdgcn_s_sleep(2);
    }
    if (lane == 0) { xw[0] = mx; __hip_atomic_store(&xw[2], 1u, __ATOMIC_RELAXED, __HIP_MEMORY_SCOPE_WORKGROUP); }
    WAITL0();
  }
  BAR();
  if (!tall) {
    BAR();
    if (lane == 0) st32<SYS>(C.pws + WS_DONE + (size_t)(rank * 16 + T.ct) * 128, call);
  } else if (incoming) {
    early_store(C, E, f, 3, lane, 448.0f / __uint_as_float(mx));
    const unsigned* fl = (const unsigned*)(C.ws + WS_DONE + (size_t)f * 128);
    const u64 t1 = __builtin_amdgcn_s_memrealtime();
    while (ld32<SYS>(fl) < call) {
      if (give_up(tmo, call, t1, spins)) { if (lane == 0) *(GAS u8*)(size_t)(C.out + (size_t)rank * M * NS) = 0x7f; break; }
      __builtin_amdgcn_s_sleep(1);
    }
  }
}

extern "C" void kernel_launch(int rank, int nranks, ncclComm_t comm, hipStream_t stream,
                              void* const* d_in, const int* in_sizes, int n_in,
                              void* d_out, int out_size, const int* in_dtypes, int out_dtype,
                              void* const* d_out_all, void* const* d_ws_all, size_t ws_bytes) {
  (void)comm; (void)in_sizes; (void)n_in; (void)d_out; (void)out_size; (void)in_dtypes; (void)out_dtype;
  if (nranks != NR || ws_bytes < WS_END || (unsigned)rank >= (unsigned)NR) { fprintf(stderr, "unsupported configuration\n"); return; }
  static bool init = false;
  if (!init) {
    HIP_CHECK(hipFuncSetAttribute((const void*)gemm_a2a_q, hipFuncAttributeMaxDynamicSharedMemorySize, LDS_BYTES));
    init = true;
  }
  Params P;
  P.x = (const u16*)d_in[0]; P.w = (const u16*)d_in[1];
  for (int j = 0; j < NR; ++j) { P.out[j] = (u8*)d_out_all[(rank + j) % NR]; P.ws[j] = (u8*)d_ws_all[(rank + j) % NR]; }
  P.rank = rank; P.pad = 0;
  hipLaunchKernelGGL(gemm_a2a_q, dim3(NWG), dim3(512), LDS_BYTES, stream, P);
  HIP_CHECK(hipGetLastError());
}
```
